```python
import numpy as np

kernel_name = "gpu_gemm_ar_m4096_k4096_n4096_w8a8_dqsilu"


ABSMAX_RTOL = 0.001

COMPUTE_DTYPE = "int8"
EPILOGUE = "dequant_silu"
IN_QUANT = "int8"
OUT_QUANT = None
IN_DTYPES = {"x": "int8", "w_mat": "int8",
             "scale_x": "float32", "scale_w": "float32"}
WORLD_SIZE = 8

def setup_inputs():

    x = np.random.randint(-127, 128, size=(4096, 4096)).astype(np.int8)
    w_mat = np.random.randint(-127, 128, size=(4096, 4096)).astype(np.int8)
    scale_x = (0.02 * (1.0 + np.random.rand(1))).astype(np.float32)
    scale_w = (0.02 * (1.0 + np.random.rand(1))).astype(np.float32)
    return {"x": x, "w_mat": w_mat, "scale_x": scale_x, "scale_w": scale_w}

def reference(x, w_mat, scale_x, scale_w):
    acc = x.astype(np.float64) @ w_mat.astype(np.float64)
    y = (acc.astype(np.float32) * (scale_x * scale_w)).astype(np.float32)
    return (y / (1.0 + np.exp(-np.clip(y, -60.0, 60.0)))).astype(np.float32)
```

```cpp
#include <hip/hip_runtime.h>
#include <rccl/rccl.h>
#include <cstdio>
#include <cstdint>
#include <cstring>

#define HIP_CHECK(expr) do { hipError_t _e = (expr); if (_e != hipSuccess) { \
    fprintf(stderr, "HIP error %s at %s:%d\n", hipGetErrorString(_e), __FILE__, __LINE__); \
    return; } } while (0)

namespace {

typedef unsigned u32x4 __attribute__((ext_vector_type(4)));
typedef int i32x4 __attribute__((ext_vector_type(4)));
typedef float f32x4 __attribute__((ext_vector_type(4)));

constexpr int NR = 8;
constexpr int MM = 4096, NN = 4096;
constexpr int KSH = 512;
constexpr int NCOMM = 16, NGEMM = 240, GRID = NCOMM + NGEMM, THREADS = 512;
constexpr int MT = 18;
constexpr int NBI = 15;
constexpr int XFR = NBI * MT;
constexpr int WFR = 256;
constexpr unsigned STEP_BYTES = (XFR + WFR) * 1024u;
constexpr unsigned SLOT_BYTES = 64u * STEP_BYTES;
constexpr unsigned CNT_OFF = 0;
constexpr unsigned TMO_OFF = 16384;
constexpr unsigned FLAG_OFF = 65536;
constexpr unsigned DATA_OFF = 1u << 20;
constexpr size_t WS_NEEDED = (size_t)DATA_OFF + 2ull * SLOT_BYTES;
constexpr int STAGE_FR = MT + 16;
constexpr int STAGE_BYTES = STAGE_FR * 1024;
constexpr int RING = 3;
constexpr int LDS_BYTES = RING * STAGE_BYTES;
constexpr unsigned long long DEADLINE_TICKS = 50000000ull;
constexpr int LAGF = 2;

struct Args {
    const char* x;
    const char* w;
    const float* sx;
    const float* sw;
    float* out;
    char* ws[NR];
    int rank;
    int pad;
    unsigned order[NGEMM];
};

__host__ __device__ inline int rot_of_unit(int v) {
    return v < 64 ? (v * 7) >> 6 : (((v - 64) >> 2) * 7) >> 4;
}

__device__ __forceinline__ void st16_sys(unsigned off, u32x4 d, const char* base, int imm) {
    asm volatile("global_store_dwordx4 %0, %1, %2 offset:%3 sc0 sc1\n\ts_nop 1"
                 :: "v"(off), "v"(d), "s"(base), "i"(imm) : "memory");
}
__device__ __forceinline__ void ld16(u32x4& d, unsigned off, const char* base) {
    asm volatile("global_load_dwordx4 %0, %1, %2" : "=v"(d) : "v"(off), "s"(base) : "memory");
}
__device__ __forceinline__ void ld4(unsigned& d, unsigned off, const char* base) {
    asm volatile("global_load_dword %0, %1, %2" : "=v"(d) : "v"(off), "s"(base) : "memory");
}
template <int N> __device__ __forceinline__ void wait_vm() {
    asm volatile("s_waitcnt vmcnt(%0)" :: "i"(N) : "memory");
}
__device__ __forceinline__ void glds16_sys(unsigned voff, const char* sbase, unsigned ldsaddr) {
    unsigned keep;
    asm volatile("s_mov_b32 %0, m0\n\ts_mov_b32 m0, %3\n\ts_nop 0\n\t"
                 "global_load_lds_dwordx4 %1, %2 sc0 sc1\n\ts_mov_b32 m0, %0"
                 : "=&s"(keep) : "v"(voff), "s"(sbase), "s"(ldsaddr) : "memory");
}

template <bool IS_W>
__device__ __forceinline__ void comm_wave(const Args& a, int v, int lane, unsigned seq) {
    constexpr int NL = IS_W ? 16 : 4;
    const int vr = IS_W ? v - 64 : v;
    const int rot = rot_of_unit(v);
    const char* base[NR];
    base[0] = a.ws[7];
#pragma unroll
    for (int p = 1; p < NR; ++p) { int j = rot + p - 1; if (j >= 7) j -= 7; base[p] = a.ws[j]; }
    const int i = lane & 15, g = lane >> 4;
    unsigned soff, sstep;
    if (IS_W) { soff = (unsigned)(16 * g) * NN + (unsigned)vr * 64u + 4u * i; sstep = 64u * NN; }
    else      { soff = (unsigned)(64 * vr + i) * KSH + 16u * g; sstep = 64u; }
    const char* src = IS_W ? a.w : a.x;
    unsigned doff = DATA_OFF + (seq & 1u) * SLOT_BYTES + (unsigned)a.rank * 8u * STEP_BYTES
                  + (IS_W ? (unsigned)(XFR + 4 * vr) : (unsigned)(4 * vr)) * 1024u + lane * 16u;
    unsigned foff = FLAG_OFF + ((unsigned)a.rank * 8u * 128u + (unsigned)v) * 4u;
    unsigned rawW[16];
    u32x4 rawX[4];
    auto issue_raw = [&](unsigned off) {
        if constexpr (IS_W) {
#pragma unroll
            for (int j = 0; j < 16; ++j) ld4(rawW[j], off, src + (size_t)j * NN);
        } else {
#pragma unroll
            for (int f = 0; f < 4; ++f) ld16(rawX[f], off, src + (size_t)f * 16 * KSH);
        }
    };
    issue_raw(soff);
    wait_vm<0>();
    for (int s = 0; s < 8; ++s) {
        u32x4 F[4];
        if constexpr (IS_W) {
#pragma unroll
            for (int j = 0; j < 16; ++j) asm volatile("" : "+v"(rawW[j]));
#pragma unroll
            for (int q4 = 0; q4 < 4; ++q4) {
                unsigned r0 = rawW[4 * q4], r1 = rawW[4 * q4 + 1], r2 = rawW[4 * q4 + 2], r3 = rawW[4 * q4 + 3];
                unsigned t0 = __builtin_amdgcn_perm(r1, r0, 0x05010400u), t1 = __builtin_amdgcn_perm(r1, r0, 0x07030602u);
                unsigned t2 = __builtin_amdgcn_perm(r3, r2, 0x05010400u), t3 = __builtin_amdgcn_perm(r3, r2, 0x07030602u);
                F[0][q4] = __builtin_amdgcn_perm(t2, t0, 0x05040100u);
                F[1][q4] = __builtin_amdgcn_perm(t2, t0, 0x07060302u);
                F[2][q4] = __builtin_amdgcn_perm(t3, t1, 0x05040100u);
                F[3][q4] = __builtin_amdgcn_perm(t3, t1, 0x07060302u);
            }
#pragma unroll
            for (int f = 0; f < 4; ++f) asm volatile("" : "+v"(F[f]));
        } else {
#pragma unroll
            for (int f = 0; f < 4; ++f) { asm volatile("" : "+v"(rawX[f])); F[f] = rawX[f]; asm volatile("" : "+v"(F[f])); }
        }
        if (s < 7) soff += sstep;
        issue_raw(soff);
#pragma unroll
        for (int p = 0; p < NR; ++p) {
#pragma unroll
            for (int f = 0; f < 4; ++f) st16_sys(doff, F[f], base[p], f * 1024);
            if (p >= LAGF) {
                wait_vm<4 * LAGF>();
                if (lane == 0) __hip_atomic_store((unsigned*)(base[p - LAGF] + foff), seq, __ATOMIC_RELAXED, __HIP_MEMORY_SCOPE_SYSTEM);
            } else if (s > 0) {
                wait_vm<4 * LAGF + NL>();
                if (lane == 0) __hip_atomic_store((unsigned*)(base[NR - LAGF + p] + foff - 512u), seq, __ATOMIC_RELAXED, __HIP_MEMORY_SCOPE_SYSTEM);
            }
        }
        doff += STEP_BYTES;
        foff += 512u;
        wait_vm<32>();
    }
    wait_vm<0>();
#pragma unroll
    for (int p = NR - LAGF; p < NR; ++p)
        if (lane == 0) __hip_atomic_store((unsigned*)(base[p] + foff - 512u), seq, __ATOMIC_RELAXED, __HIP_MEMORY_SCOPE_SYSTEM);
}

__device__ __forceinline__ void poll_step(const char* myws, unsigned flagidx4, int q, unsigned seq, int lane) {
    const unsigned* f = (const unsigned*)(myws + FLAG_OFF + (unsigned)q * 512u + flagidx4);
    unsigned* tmo = (unsigned*)(myws + TMO_OFF);
    unsigned long long t0 = __builtin_amdgcn_s_memrealtime();
    while (true) {
        unsigned val = __hip_atomic_load(f, __ATOMIC_RELAXED, __HIP_MEMORY_SCOPE_SYSTEM);
        if (__all(val >= seq)) break;
        unsigned tv = __hip_atomic_load(tmo, __ATOMIC_RELAXED, __HIP_MEMORY_SCOPE_AGENT);
        if (__builtin_amdgcn_readfirstlane(tv) == seq || __builtin_amdgcn_s_memrealtime() - t0 > DEADLINE_TICKS) {
            if (lane == 0) __hip_atomic_store(tmo, seq, __ATOMIC_RELAXED, __HIP_MEMORY_SCOPE_AGENT);
            break;
        }
        __builtin_amdgcn_s_sleep(2);
    }
}

__device__ __forceinline__ void gemm_wg(const Args& a, char* smem, int gidx, int wave, int lane, unsigned seq) {
    const int xcd = gidx & 7, idx = gidx >> 3;
    const int I = idx % NBI, J = xcd * 2 + idx / NBI;
    const int wr = wave >> 2, wc = wave & 3;
    const char* myws = a.ws[7];
    const char* data = myws + DATA_OFF + (size_t)(seq & 1u) * SLOT_BYTES;
    const unsigned order = a.order[gidx];
    const unsigned lds0 = (unsigned)(uintptr_t)(__attribute__((address_space(3))) char*)smem;

    const char* sbase[5];
    unsigned ldst[5];
#pragma unroll
    for (int i = 0; i < 5; ++i) {
        int f = 5 * wave + i; if (f > STAGE_FR - 1) f = STAGE_FR - 1;
        sbase[i] = data + (size_t)(f < MT ? MT * I + f : XFR + 16 * J + (f - MT)) * 1024u;
        ldst[i] = lds0 + f * 1024u;
    }
    unsigned flagidx4;
    {
        const int u0 = (MT * I) >> 2;
        int u1 = (MT * I + MT - 1); if (u1 > 255) u1 = 255; u1 >>= 2;
        int e = lane & 15, fi;
        if (e < 5) { fi = u0 + e; if (fi > u1) fi = u1; }
        else if (e < 9) fi = 64 + 4 * J + (e - 5);
        else fi = u0;
        flagidx4 = (unsigned)fi * 4u;
    }
    auto qof = [&](int t) { return (int)((order >> (4 * (t & 7))) & 7u) * 8 + (t >> 3); };
    auto issue = [&](int t, int slot) {
        unsigned voff = (unsigned)qof(t) * STEP_BYTES + lane * 16u;
#pragma unroll
        for (int i = 0; i < 5; ++i) glds16_sys(voff, sbase[i], ldst[i] + slot * STAGE_BYTES);
    };

    i32x4 acc[9][4];
#pragma unroll
    for (int m = 0; m < 9; ++m)
#pragma unroll
        for (int n = 0; n < 4; ++n) acc[m][n] = (i32x4){0, 0, 0, 0};

    if (wave == 7) { poll_step(myws, flagidx4, qof(0), seq, lane); poll_step(myws, flagidx4, qof(1), seq, lane); }
    asm volatile("" ::: "memory");
    __builtin_amdgcn_s_barrier();
    asm volatile("" ::: "memory");
    if (wave < 7) { issue(0, 0); issue(1, 1); }

    int slot = 0;
    for (int t = 0; t < 64; ++t) {
        if (wave < 7) { if (t < 63) wait_vm<5>(); else wait_vm<0>(); }
        else if (t + 2 < 64) poll_step(myws, flagidx4, qof(t + 2), seq, lane);
        asm volatile("" ::: "memory");
        __builtin_amdgcn_s_barrier();
        asm volatile("" ::: "memory");
        if (wave < 7 && t + 2 < 64) { int s2 = slot + 2; if (s2 >= RING) s2 -= RING; issue(t + 2, s2); }
        const char* sa = smem + slot * STAGE_BYTES + wr * 9 * 1024 + lane * 16;
        const char* sb = smem + slot * STAGE_BYTES + MT * 1024 + wc * 4096 + lane * 16;
        i32x4 bf[4];
#pragma unroll
        for (int n = 0; n < 4; ++n) bf[n] = *(const i32x4*)(sb + n * 1024);
#pragma unroll
        for (int m = 0; m < 9; ++m) {
            i32x4 af = *(const i32x4*)(sa + m * 1024);
#pragma unroll
            for (int n = 0; n < 4; ++n) acc[m][n] = __builtin_amdgcn_mfma_i32_16x16x64_i8(af, bf[n], acc[m][n], 0, 0, 0);
        }
        asm volatile("s_waitcnt lgkmcnt(0)" ::: "memory");
        if (++slot == RING) slot = 0;
    }

    const unsigned tv = __hip_atomic_load((const unsigned*)(myws + TMO_OFF), __ATOMIC_RELAXED, __HIP_MEMORY_SCOPE_AGENT);
    const bool bad = __builtin_amdgcn_readfirstlane(tv) == seq;
    const float scale = a.sx[0] * a.sw[0];
    float* outp = a.out + (size_t)(I * (MT * 16) + wr * 144 + (lane >> 4) * 4) * NN + J * 256 + wc * 64 + (lane & 15) * 4;
#pragma unroll
    for (int m = 0; m < 9; ++m) {
        if (MT * I + wr * 9 + m < 256) {
#pragma unroll
            for (int r = 0; r < 4; ++r) {
                f32x4 o;
#pragma unroll
                for (int n = 0; n < 4; ++n) {
                    float y = (float)acc[m][n][r] * scale;
                    float c = __builtin_amdgcn_fmed3f(y, -60.0f, 60.0f);
                    float e = __builtin_amdgcn_exp2f(c * -1.4426950408889634f);
                    o[n] = y * __builtin_amdgcn_rcpf(1.0f + e);
                }
                if (bad) o = (f32x4){__builtin_nanf(""), __builtin_nanf(""), __builtin_nanf(""), __builtin_nanf("")};
                *(f32x4*)(outp + (size_t)(m * 16 + r) * NN) = o;
            }
        }
    }
}

__global__ void __launch_bounds__(THREADS, 2) fused_gemm_ag(Args a) {
    extern __shared__ __attribute__((aligned(16))) char smem[];
    const int b = blockIdx.x;
    const int wave = __builtin_amdgcn_readfirstlane(threadIdx.x >> 6);
    const int lane = threadIdx.x & 63;
    unsigned old = 0;
    if (lane == 0)
        old = __hip_atomic_fetch_add((unsigned*)(a.ws[7] + CNT_OFF) + b * 8 + wave, 1u, __ATOMIC_RELAXED, __HIP_MEMORY_SCOPE_AGENT);
    const unsigned seq = __builtin_amdgcn_readfirstlane(old) + 1u;
    if (b < NCOMM) {
        const int v = (wave >> 2) * 64 + b * 4 + (wave & 3);
        if (wave < 4) comm_wave<false>(a, v, lane, seq);
        else comm_wave<true>(a, v, lane, seq);
    } else {
        gemm_wg(a, smem, b - NCOMM, wave, lane, seq);
    }
}

}

extern "C" void kernel_launch(int rank, int nranks, ncclComm_t comm, hipStream_t stream,
                              void* const* d_in, const int* in_sizes, int n_in,
                              void* d_out, int out_size,
                              const int* in_dtypes, int out_dtype,
                              void* const* d_out_all,
                              void* const* d_ws_all, size_t ws_bytes) {
    (void)comm; (void)in_sizes; (void)n_in; (void)in_dtypes; (void)out_dtype; (void)d_out_all;
    if (nranks != NR || (unsigned)rank >= (unsigned)NR || ws_bytes < WS_NEEDED || out_size != MM * NN) {
        fprintf(stderr, "kernel_launch: unexpected configuration\n");
        return;
    }
    HIP_CHECK(hipFuncSetAttribute((const void*)fused_gemm_ag, hipFuncAttributeMaxDynamicSharedMemorySize, LDS_BYTES));
    Args a;
    memset(&a, 0, sizeof a);
    a.x = (const char*)d_in[0];
    a.w = (const char*)d_in[1];
    a.sx = (const float*)d_in[2];
    a.sw = (const float*)d_in[3];
    a.out = (float*)d_out;
    for (int j = 0; j < NR; ++j) a.ws[j] = (char*)d_ws_all[(rank + 1 + j) % NR];
    a.rank = rank;
    for (int g = 0; g < NGEMM; ++g) {
        const int xcd = g & 7, idx = g >> 3, I = idx % NBI, J = xcd * 2 + idx / NBI;
        const int u0 = (MT * I) >> 2;
        int u1 = MT * I + MT - 1; if (u1 > 255) u1 = 255; u1 >>= 2;
        int key[7];
        for (int d = 0; d < 7; ++d) {
            int mx = 0;
            for (int u = u0; u <= u1; ++u) { int s = (d - rot_of_unit(u) + 7) % 7; if (s > mx) mx = s; }
            for (int u = 64 + 4 * J; u < 64 + 4 * J + 4; ++u) { int s = (d - rot_of_unit(u) + 7) % 7; if (s > mx) mx = s; }
            key[d] = mx * 8 + d;
        }
        unsigned ord = (unsigned)rank;
        int used = 0;
        for (int k = 1; k < 8; ++k) {
            int best = -1;
            for (int d = 0; d < 7; ++d) if (!((used >> d) & 1) && (best < 0 || key[d] < key[best])) best = d;
            used |= 1 << best;
            ord |= (unsigned)((rank - 1 - best + 2 * NR) % NR) << (4 * k);
        }
        a.order[g] = ord;
    }
    hipLaunchKernelGGL(fused_gemm_ag, dim3(GRID), dim3(THREADS), LDS_BYTES, stream, a);
    HIP_CHECK(hipGetLastError());
}
```

```cpp
#include <hip/hip_runtime.h>
#include <rccl/rccl.h>
#include <cstdio>
#include <cstdint>
#include <cstring>

#define HIP_CHECK(expr) do { hipError_t _e = (expr); if (_e != hipSuccess) { \
    fprintf(stderr, "HIP error %s at %s:%d\n", hipGetErrorString(_e), __FILE__, __LINE__); \
    return; } } while (0)

namespace {

typedef unsigned u32x4 __attribute__((ext_vector_type(4)));
typedef int i32x4 __attribute__((ext_vector_type(4)));
typedef float f32x4 __attribute__((ext_vector_type(4)));

constexpr int NR = 8;
constexpr int MM = 4096, NN = 4096;
constexpr int KSH = 512;
constexpr int NCOMM = 16, NGEMM = 240, GRID = NCOMM + NGEMM, THREADS = 512;
#define CW_ 1
constexpr int CW = CW_;
constexpr int TW = NCOMM * CW;
constexpr int UPH = 64 / TW;
constexpr int MT = 18;
constexpr int NBI = 15;
constexpr int XFR = NBI * MT;
constexpr int WFR = 256;
constexpr unsigned STEP_BYTES = (XFR + WFR) * 1024u;
constexpr unsigned SLOT_BYTES = 64u * STEP_BYTES;
constexpr unsigned CNT_OFF = 0;
constexpr unsigned TMO_OFF = 16384;
#define FSTRIDE 128
constexpr unsigned FLAG_STRIDE = FSTRIDE;
constexpr unsigned FLAG_OFF = 1u << 20;
constexpr unsigned DATA_OFF = 2u << 20;
constexpr size_t WS_NEEDED = (size_t)DATA_OFF + 2ull * SLOT_BYTES;
constexpr int STAGE_FR = MT + 16;
constexpr int STAGE_BYTES = STAGE_FR * 1024;
constexpr int RING = 4;
constexpr int CTRL_LDS = RING * STAGE_BYTES;
constexpr int FB_LDS = CTRL_LDS + 64;
constexpr int FB_BYTES = 67 * 64;
constexpr int QT_LDS = FB_LDS + FB_BYTES;
constexpr int QT_BYTES = 68 * 4;
constexpr int LDS_BYTES = QT_LDS + QT_BYTES;
constexpr unsigned long long DEADLINE_TICKS = 50000000ull;
#define LINGER 60
#define POLL_BITS "sc0 sc1"
#define PLANES 64
#define PSLEEP 4
#define LF_ 3
constexpr int LF = LF_;

struct Args {
    const char* x;
    const char* w;
    const float* sx;
    const float* sw;
    float* out;
    char* ws[NR];
    int rank;
    int mode;
    unsigned order[NGEMM];
};

__host__ __device__ inline int unit_at(int pos, int r) { return pos < 64 ? 64 + ((pos - 8 * r) & 63) : ((pos - 8 * r) & 63); }
__host__ __device__ inline int pos_of(int v, int r) { return v >= 64 ? ((v + 8 * r) & 63) : 64 + ((v + 8 * r) & 63); }

__device__ __forceinline__ void st16_sys(unsigned off, u32x4 d, const char* base, int imm) {
    asm volatile("global_store_dwordx4 %0, %1, %2 offset:%3 sc0 sc1\n\ts_nop 1"
                 :: "v"(off), "v"(d), "s"(base), "i"(imm) : "memory");
}
__device__ __forceinline__ void st4_sys(uint64_t addr, unsigned val) {
    asm volatile("global_store_dword %0, %1, off sc0 sc1" :: "v"(addr), "v"(val) : "memory");
}
template <int N> __device__ __forceinline__ void wait_vm() {
    asm volatile("s_waitcnt vmcnt(%0)" :: "i"(N) : "memory");
}
#define GLDS_BITS " sc0 sc1"
__device__ __forceinline__ void glds16_sys(unsigned voff, const char* sbase, unsigned ldsaddr) {
    unsigned keep;
    asm volatile("s_mov_b32 %0, m0\n\ts_mov_b32 m0, %3\n\ts_nop 0\n\t"
                 "global_load_lds_dwordx4 %1, %2" GLDS_BITS "\n\ts_mov_b32 m0, %0"
                 : "=&s"(keep) : "v"(voff), "s"(sbase), "s"(ldsaddr) : "memory");
}
__device__ __forceinline__ void glds4_sys(unsigned voff, const char* sbase, unsigned ldsaddr) {
    unsigned keep;
    asm volatile("s_mov_b32 %0, m0\n\ts_mov_b32 m0, %3\n\ts_nop 0\n\t"
                 "global_load_lds_dword %1, %2 " POLL_BITS "\n\ts_mov_b32 m0, %0"
                 : "=&s"(keep) : "v"(voff), "s"(sbase), "s"(ldsaddr) : "memory");
}
#define WG_BARRIER() do { asm volatile("" ::: "memory"); __builtin_amdgcn_s_waitcnt(0xC07F); __builtin_amdgcn_s_barrier(); asm volatile("" ::: "memory"); } while (0)

struct RawW { unsigned d[16]; };
struct RawX { u32x4 q[4]; };

__device__ __forceinline__ void raw_issue(RawW& r, const Args& a, int v, int s, int lane) {
    const unsigned off = (unsigned)(64 * s + 16 * (lane >> 4)) * NN + (unsigned)(v - 64) * 64u + 4u * (lane & 15);
#pragma unroll
    for (int j = 0; j < 16; ++j)
        asm volatile("global_load_dword %0, %1, %2" : "=v"(r.d[j]) : "v"(off + (unsigned)j * NN), "s"(a.w) : "memory");
}
__device__ __forceinline__ void raw_issue(RawX& r, const Args& a, int v, int s, int lane) {
    const unsigned off = (unsigned)(64 * v + (lane & 15)) * KSH + 64u * s + 16u * (lane >> 4);
#pragma unroll
    for (int f = 0; f < 4; ++f)
        asm volatile("global_load_dwordx4 %0, %1, %2" : "=v"(r.q[f]) : "v"(off + (unsigned)f * 16u * KSH), "s"(a.x) : "memory");
}
__device__ __forceinline__ void raw_pin(RawW& r) {
#pragma unroll
    for (int j = 0; j < 16; ++j) asm volatile("" : "+v"(r.d[j]));
}
__device__ __forceinline__ void raw_pin(RawX& r) {
#pragma unroll
    for (int f = 0; f < 4; ++f) asm volatile("" : "+v"(r.q[f]));
}
__device__ __forceinline__ void raw_format(u32x4 (&F)[4], const RawW& r) {
#pragma unroll
    for (int q4 = 0; q4 < 4; ++q4) {
        const unsigned r0 = r.d[4 * q4], r1 = r.d[4 * q4 + 1], r2 = r.d[4 * q4 + 2], r3 = r.d[4 * q4 + 3];
        const unsigned t0 = __builtin_amdgcn_perm(r1, r0, 0x05010400u), t1 = __builtin_amdgcn_perm(r1, r0, 0x07030602u);
        const unsigned t2 = __builtin_amdgcn_perm(r3, r2, 0x05010400u), t3 = __builtin_amdgcn_perm(r3, r2, 0x07030602u);
        F[0][q4] = __builtin_amdgcn_perm(t2, t0, 0x05040100u);
        F[1][q4] = __builtin_amdgcn_perm(t2, t0, 0x07060302u);
        F[2][q4] = __builtin_amdgcn_perm(t3, t1, 0x05040100u);
        F[3][q4] = __builtin_amdgcn_perm(t3, t1, 0x07060302u);
    }
#pragma unroll
    for (int f = 0; f < 4; ++f) asm volatile("" : "+v"(F[f]));
}
__device__ __forceinline__ void raw_format(u32x4 (&F)[4], const RawX& r) {
#pragma unroll
    for (int f = 0; f < 4; ++f) { F[f] = r.q[f]; asm volatile("" : "+v"(F[f])); }
}
__device__ __forceinline__ unsigned unit_doff(int rank, int s, int v, unsigned seq) {
    return DATA_OFF + (seq & 1u) * SLOT_BYTES + (unsigned)(rank * 8 + s) * STEP_BYTES
         + (unsigned)(v >= 64 ? XFR + 4 * (v - 64) : 4 * v) * 1024u;
}
__device__ __forceinline__ unsigned unit_foff(int rank, int s, int v) {
    return FLAG_OFF + (unsigned)((rank * 8 + s) * 128 + v) * FLAG_STRIDE;
}

template <int J> struct IC { static constexpr int value = J; };
template <bool B> struct BC { static constexpr bool value = B; };
template <int J, int N, class Fn> __device__ __forceinline__ void static_for(Fn&& fn) {
    if constexpr (J < N) { fn(IC<J>{}); static_for<J + 1, N>(fn); }
}

typedef __attribute__((address_space(3))) int lds_int;

__device__ __forceinline__ void comm_wave(const Args& a, int gw, int lane, unsigned seq, volatile lds_int* done) {
    const char* base[7];
#pragma unroll
    for (int p = 0; p < 7; ++p) base[p] = a.ws[p];
    RawW rw; RawX rx;
    raw_issue(rw, a, unit_at(gw, a.rank), 0, lane);
    wait_vm<0>();
    raw_pin(rw);
    for (int s = 0; s < 8; ++s) {
        static_for<0, 2 * UPH>([&](auto jc) {
            constexpr int j = decltype(jc)::value;
            constexpr bool cur_w = j < UPH, next_w = (j + 1 < UPH) || (j + 1 == 2 * UPH);
            const int v = unit_at(j * TW + gw, a.rank);
            u32x4 F[4];
            if constexpr (cur_w) raw_format(F, rw); else raw_format(F, rx);
            const unsigned doff = unit_doff(a.rank, s, v, seq) + lane * 16u;
            const int sn = (j + 1 == 2 * UPH) ? (s < 7 ? s + 1 : 7) : s;
            const int vn = unit_at(((j + 1) % (2 * UPH)) * TW + gw, a.rank);
            if constexpr (next_w) raw_issue(rw, a, vn, sn, lane); else raw_issue(rx, a, vn, sn, lane);
#pragma unroll
            for (int f = 0; f < 4; ++f) {
#pragma unroll
                for (int p = 0; p < 7; ++p) st16_sys(doff, F[f], base[p], f * 1024);
                if (f == LF - 1) {
                    wait_vm<7 * LF + 4>();
                    if (lane == 0) done[0] = s * 2 * UPH + j;
                } else if (f >= LF) wait_vm<7 * LF>();
            }
            wait_vm<28>();
            if constexpr (next_w) raw_pin(rw); else raw_pin(rx);
        });
    }
    wait_vm<0>();
    if (lane == 0) done[0] = 8 * 2 * UPH;
}

__device__ __forceinline__ void flag_wave(const Args& a, int gw0, int lane, unsigned seq, volatile lds_int* done) {
    uint64_t fl = (uint64_t)a.ws[lane < 7 ? lane : 0];
    asm volatile("" : "+v"(fl));
    constexpr int NU = 16 * UPH;
    int sent[CW];
#pragma unroll
    for (int w = 0; w < CW; ++w) sent[w] = 0;
    const unsigned long long t0 = __builtin_amdgcn_s_memrealtime();
    while (true) {
        bool all = true;
#pragma unroll
        for (int w = 0; w < CW; ++w) {
            const int d = __builtin_amdgcn_readfirstlane(done[w * 16]);
            while (sent[w] < d) {
                const int k = sent[w]++;
                const int s = k / (2 * UPH), j = k % (2 * UPH);
                const int v = unit_at(j * TW + gw0 + w, a.rank);
                if (lane < 7) st4_sys(fl + unit_foff(a.rank, s, v), seq);
            }
            all = all && sent[w] == NU;
        }
        if (all || __builtin_amdgcn_s_memrealtime() - t0 > DEADLINE_TICKS) break;
        __builtin_amdgcn_s_sleep(1);
    }
}

__device__ __forceinline__ void gemm_wg(const Args& a, char* smem, int gidx, int wave, int lane, unsigned seq) {
    const int xcd = gidx & 7, idx = gidx >> 3;
    const int I = idx % NBI, J = xcd * 2 + idx / NBI;
    const int wr = wave >> 2, wc = wave & 3;
    char* myws = a.ws[7];

    volatile lds_int* ctl_lds = (volatile lds_int*)((__attribute__((address_space(3))) char*)smem + CTRL_LDS);
    volatile lds_int* fb_lds = (volatile lds_int*)((__attribute__((address_space(3))) char*)smem + FB_LDS);
    for (int i = threadIdx.x; i < FB_BYTES / 4; i += THREADS) fb_lds[i] = 0;
    if (threadIdx.x == 0) ctl_lds[0] = 0;

    {
        const int u = gidx * 8 + wave;
        if (u < 1024) {
            u32x4 F[4];
            const int s = u >> 7, v = u & 127;
            if (v >= 64) { RawW r; raw_issue(r, a, v, s, lane); wait_vm<0>(); raw_pin(r); raw_format(F, r); }
            else         { RawX r; raw_issue(r, a, v, s, lane); wait_vm<0>(); raw_pin(r); raw_format(F, r); }
            const unsigned doff = unit_doff(a.rank, s, v, seq) + lane * 16u;
#pragma unroll
            for (int f = 0; f < 4; ++f) st16_sys(doff, F[f], myws, f * 1024);
            wait_vm<0>();
            if (lane == 0) __hip_atomic_store((unsigned*)(myws + unit_foff(a.rank, s, v)), seq, __ATOMIC_RELAXED, __HIP_MEMORY_SCOPE_SYSTEM);
        }
    }

    const char* data = myws + DATA_OFF + (size_t)(seq & 1u) * SLOT_BYTES;
    const unsigned lds0 = (unsigned)(uintptr_t)(__attribute__((address_space(3))) char*)smem;
    volatile lds_int* qt_lds = (volatile lds_int*)((__attribute__((address_space(3))) char*)smem + QT_LDS);
    if (wave == 0) {
        const unsigned order = a.order[gidx];
        const int tt = lane - 8;
        const int s = lane < 8 ? lane : tt / 7, k = lane < 8 ? 0 : 1 + tt % 7;
        const int q = (int)((order >> (4 * k)) & 7u) * 8 + s;
        qt_lds[lane] = q;
        if (lane == 63) { qt_lds[64] = q; qt_lds[65] = q; qt_lds[66] = q; qt_lds[67] = q; }
    }
    const bool g1 = wave >= 4;
    const bool poller = wave == 3;
    const int dv = wave < 3 ? wave : wave - 1;
    const char* sbase[5];
    unsigned ldst[5];
#pragma unroll
    for (int i = 0; i < 5; ++i) {
        int f = 5 * dv + i; if (f > STAGE_FR - 1) f = STAGE_FR - 1;
        sbase[i] = data + (size_t)(f < MT ? MT * I + f : XFR + 16 * J + (f - MT)) * 1024u;
        ldst[i] = lds0 + f * 1024u;
    }
    const unsigned lane16 = lane * 16u;
    unsigned flagidx4;
    {
        const int u0 = (MT * I) >> 2;
        int u1 = (MT * I + MT - 1); if (u1 > 255) u1 = 255; u1 >>= 2;
        int e = lane & 15, fi;
        if (e < 5) { fi = u0 + e; if (fi > u1) fi = u1; }
        else if (e < 9) fi = 64 + 4 * J + (e - 5);
        else fi = u0;
        flagidx4 = FLAG_OFF + (unsigned)fi * FLAG_STRIDE;
    }
    auto refresh = [&](int lo) {
        const unsigned q = (unsigned)qt_lds[lo + (lane >> 4)];
        if (lane < PLANES) glds4_sys(q * (128u * FLAG_STRIDE) + flagidx4, myws, lds0 + FB_LDS + (unsigned)lo * 64u);
    };
    auto mirror = [&](int u) { return (unsigned)fb_lds[u * 16 + lane]; };
    auto voff_of = [&](int u) { return (unsigned)qt_lds[u] * STEP_BYTES + lane16; };
    auto upto_from = [&](int u, unsigned long long ok) {
        const unsigned long long nok = ~ok;
        int n = u + (nok ? (int)(__builtin_ctzll(nok) >> 4) : 4);
        return n > 64 ? 64 : n;
    };
    auto ready_upto = [&](int u, unsigned v) {
        return upto_from(u, __ballot(v >= seq));
    };
    auto ready1 = [&](unsigned v) {
        return (__ballot(v >= seq) & 0xFFFFull) == 0xFFFFull;
    };
    auto issue = [&](int t, unsigned voff) {
        const unsigned so = (unsigned)(t & (RING - 1)) * STAGE_BYTES;
#pragma unroll
        for (int i = 0; i < 5; ++i) glds16_sys(voff, sbase[i], ldst[i] + so);
    };
    i32x4 acc[9][4];
#pragma unroll
    for (int m = 0; m < 9; ++m)
#pragma unroll
        for (int n = 0; n < 4; ++n) acc[m][n] = (i32x4){0, 0, 0, 0};
    i32x4 Af[9], Bf[4];
    int iss = 0;
#define SB() __builtin_amdgcn_sched_barrier(0)
#define LIKELY(x) __builtin_expect(!!(x), 1)
#define UNLIKELY(x) __builtin_expect(!!(x), 0)

    auto loader_block = [&](int u, int lim) {
        while (true) {
            int r = ready_upto(iss, mirror(iss));
            if (r > lim) r = lim;
            for (; iss < r; ++iss) issue(iss, voff_of(iss));
            if (iss > u) break;
            __builtin_amdgcn_s_sleep(1);
        }
    };
    auto loader_linger = [&](int lim) {
        const unsigned long long t0 = __builtin_amdgcn_s_memrealtime();
        while (iss < lim && iss < 64 && __builtin_amdgcn_s_memrealtime() - t0 < LINGER) {
            if (ready1(mirror(iss))) { issue(iss, voff_of(iss)); ++iss; }
            else __builtin_amdgcn_s_sleep(1);
        }
    };
    auto poller_block = [&](int u) {
        unsigned* tmo = (unsigned*)(myws + TMO_OFF);
        const unsigned long long t0 = __builtin_amdgcn_s_memrealtime();
        for (unsigned spins = 1;; ++spins) {
            refresh(iss);
            wait_vm<0>();
            iss = ready_upto(iss, mirror(iss));
            if (iss > u) break;
            __builtin_amdgcn_s_sleep(PSLEEP);
            if ((spins & 15u) == 0u) {
                const unsigned tv = __hip_atomic_load(tmo, __ATOMIC_RELAXED, __HIP_MEMORY_SCOPE_AGENT);
                if (__builtin_amdgcn_readfirstlane(tv) == seq || __builtin_amdgcn_s_memrealtime() - t0 > DEADLINE_TICKS) {
                    if (lane == 0) __hip_atomic_store(tmo, seq, __ATOMIC_RELAXED, __HIP_MEMORY_SCOPE_AGENT);
                    wait_vm<0>();
                    for (int i = lane; i < FB_BYTES / 4; i += 64) fb_lds[i] = 0x7fffffff;
                    iss = 64;
                    break;
                }
            }
        }
    };
    auto poller_linger = [&](int lim) {
        const unsigned long long t0 = __builtin_amdgcn_s_memrealtime();
        while (iss < lim && iss < 64 && __builtin_amdgcn_s_memrealtime() - t0 < LINGER) {
            refresh(iss);
            wait_vm<0>();
            iss = ready_upto(iss, mirror(iss));
        }
    };
    auto wait_landed = [&](int u) {
        const int ahead = iss - u - 1;
        wait_vm<10>();
        if (UNLIKELY(ahead < 2)) { wait_vm<5>(); if (ahead < 1) wait_vm<0>(); }
    };
    auto qraw = [&](int u) { return (unsigned)qt_lds[u]; };
    auto read_frags = [&](int t, auto dma, unsigned voff) {
        const char* st = smem + (t & (RING - 1)) * STAGE_BYTES + lane16;
        const char* sa = st + wr * 9 * 1024;
        const char* sb = st + MT * 1024 + wc * 4096;
        const unsigned so = (unsigned)(iss & (RING - 1)) * STAGE_BYTES;
        constexpr bool D = decltype(dma)::value;
        if constexpr (D) { glds16_sys(voff, sbase[0], ldst[0] + so); SB(); }
#pragma unroll
        for (int n = 0; n < 4; ++n) Bf[n] = *(const i32x4*)(sb + n * 1024);
        if constexpr (D) { SB(); glds16_sys(voff, sbase[1], ldst[1] + so); SB(); }
#pragma unroll
        for (int m = 0; m < 3; ++m) Af[m] = *(const i32x4*)(sa + m * 1024);
        if constexpr (D) { SB(); glds16_sys(voff, sbase[2], ldst[2] + so); SB(); }
#pragma unroll
        for (int m = 3; m < 6; ++m) Af[m] = *(const i32x4*)(sa + m * 1024);
        if constexpr (D) { SB(); glds16_sys(voff, sbase[3], ldst[3] + so); SB(); }
#pragma unroll
        for (int m = 6; m < 9; ++m) Af[m] = *(const i32x4*)(sa + m * 1024);
        if constexpr (D) { SB(); glds16_sys(voff, sbase[4], ldst[4] + so); }
        SB();
    };
    auto mma_rows = [&](int m0, int m1) {
#pragma unroll
        for (int m = m0; m < m1; ++m)
#pragma unroll
            for (int n = 0; n < 4; ++n) {
                acc[m][n] = __builtin_amdgcn_mfma_i32_16x16x64_i8(Af[m], Bf[n], acc[m][n], 0, 0, 0);
            }
    };
    auto mma_phase = [&](auto&& h1, auto&& h2, auto&& h3) {
        __builtin_amdgcn_s_setprio(1);
        mma_rows(0, 3);
        SB();
        h1();
        SB();
        mma_rows(3, 4);
        SB();
        h2();
        SB();
        mma_rows(4, 5);
        SB();
        h3();
        SB();
        mma_rows(5, 9);
        __builtin_amdgcn_s_setprio(0);
    };
#define STAMP0()
#define STAMP(i)
    typedef BC<true> True_;
    typedef BC<false> False_;
    WG_BARRIER();
    if (poller) {
        poller_block(0);
        poller_linger(RING);
        WG_BARRIER();
        STAMP0();
        for (int t = 0; t < 64; ++t) {
            read_frags(t, False_{}, 0u);
            STAMP(0);
            WG_BARRIER();
            STAMP(1);
            const unsigned mv = mirror(iss);
            SB();
            unsigned long long ok;
            mma_phase([&]() { ok = __ballot(mv >= seq); },
                      [&]() { iss = upto_from(iss, ok); },
                      [&]() { if (iss < 64 && iss <= t + 1 + RING) refresh(iss); });
            SB();
            STAMP(2);
            if (UNLIKELY(iss <= t + 1 && t < 63)) { poller_block(t + 1); poller_linger(t + RING); }
            STAMP(3);
            WG_BARRIER();
            STAMP(4);
        }
    } else if (!g1) {
        loader_block(0, RING);
        loader_linger(RING);
        wait_landed(0);
        unsigned voff = voff_of(iss);
        bool more = iss < RING && ready_upto(iss, mirror(iss)) > iss;
        WG_BARRIER();
        STAMP0();
        for (int t = 0; t < 64; ++t) {
            if (LIKELY(more)) { read_frags(t, True_{}, voff); ++iss; } else read_frags(t, False_{}, voff);
            STAMP(0);
            WG_BARRIER();
            STAMP(1);
            const unsigned mv = mirror(iss), qv = qraw(iss);
            SB();
            unsigned long long ok;
            mma_phase([&]() { ok = __ballot(mv >= seq); },
                      [&]() { more = (iss < t + 1 + RING) & (iss < 64) & ((ok & 0xFFFFull) == 0xFFFFull); },
                      [&]() { voff = __umul24(qv, STEP_BYTES) + lane16; });
            SB();
            STAMP(2);
            if (LIKELY(t < 63)) {
                if (UNLIKELY(iss <= t + 1)) {
                    loader_block(t + 1, t + RING);
                    loader_linger(t + RING);
                    more = iss < t + 1 + RING && iss < 64 && ready1(mirror(iss));
                    voff = voff_of(iss);
                }
                wait_landed(t + 1);
            }
            STAMP(3);
            WG_BARRIER();
            STAMP(4);
        }
    } else {
        loader_block(0, RING);
        loader_linger(RING);
        wait_landed(0);
        WG_BARRIER();
        unsigned voff = voff_of(iss);
        bool more = iss < RING && ready_upto(iss, mirror(iss)) > iss;
        WG_BARRIER();
        STAMP0();
        for (int t = 0; t < 64; ++t) {
            if (LIKELY(more)) { read_frags(t, True_{}, voff); ++iss; } else read_frags(t, False_{}, voff);
            STAMP(0);
            if (LIKELY(t < 63)) {
                if (UNLIKELY(iss <= t + 1)) { loader_block(t + 1, t + RING); loader_linger(t + RING); }
                wait_landed(t + 1);
            }
            STAMP(3);
            WG_BARRIER();
            STAMP(1);
            const unsigned mv = mirror(iss), qv = qraw(iss);
            SB();
            unsigned long long ok;
            mma_phase([&]() { ok = __ballot(mv >= seq); },
                      [&]() { more = (iss < t + 1 + RING) & (iss < 64) & ((ok & 0xFFFFull) == 0xFFFFull); },
                      [&]() { voff = __umul24(qv, STEP_BYTES) + lane16; });
            SB();
            STAMP(2);
            if (LIKELY(t < 63)) WG_BARRIER();
            STAMP(4);
        }
    }

    const unsigned tv = __hip_atomic_load((const unsigned*)(myws + TMO_OFF), __ATOMIC_RELAXED, __HIP_MEMORY_SCOPE_AGENT);
    const bool bad = __builtin_amdgcn_readfirstlane(tv) == seq;
    const float scale = a.sx[0] * a.sw[0];
    float* outp = a.out + (size_t)(I * (MT * 16) + wr * 144 + (lane >> 4) * 4) * NN + J * 256 + wc * 64 + (lane & 15) * 4;
#pragma unroll
    for (int m = 0; m < 9; ++m) {
        if (MT * I + wr * 9 + m < 256) {
#pragma unroll
            for (int r = 0; r < 4; ++r) {
                f32x4 o;
#pragma unroll
                for (int n = 0; n < 4; ++n) {
                    float y = (float)acc[m][n][r] * scale;
                    float c = __builtin_amdgcn_fmed3f(y, -60.0f, 60.0f);
                    float e = __builtin_amdgcn_exp2f(c * -1.4426950408889634f);
                    o[n] = y * __builtin_amdgcn_rcpf(1.0f + e);
                }
                if (bad) o = (f32x4){__builtin_nanf(""), __builtin_nanf(""), __builtin_nanf(""), __builtin_nanf("")};
                *(f32x4*)(outp + (size_t)(m * 16 + r) * NN) = o;
            }
        }
    }
}

__global__ void __launch_bounds__(THREADS, 2) fused_gemm_ag(Args a) {
    extern __shared__ __attribute__((aligned(16))) char smem[];
    const int b = blockIdx.x;
    const int wave = __builtin_amdgcn_readfirstlane(threadIdx.x >> 6);
    const int lane = threadIdx.x & 63;
    if (b < NCOMM && wave >= CW && wave != 7) return;
    unsigned old = 0;
    if (lane == 0)
        old = __hip_atomic_fetch_add((unsigned*)(a.ws[7] + CNT_OFF) + b * 8 + wave, 1u, __ATOMIC_RELAXED, __HIP_MEMORY_SCOPE_AGENT);
    const unsigned seq = __builtin_amdgcn_readfirstlane(old) + 1u;
    if (b < NCOMM) {
        volatile lds_int* done = (volatile lds_int*)((__attribute__((address_space(3))) char*)smem);
        if (wave < CW && lane == 0) done[wave * 16] = 0;
        WG_BARRIER();
        if (wave < CW) comm_wave(a, b * CW + wave, lane, seq, done + wave * 16);
        else flag_wave(a, b * CW, lane, seq, done);
    } else {
        gemm_wg(a, smem, b - NCOMM, wave, lane, seq);
    }
}

}

extern "C" void kernel_launch(int rank, int nranks, ncclComm_t comm, hipStream_t stream,
                              void* const* d_in, const int* in_sizes, int n_in,
                              void* d_out, int out_size,
                              const int* in_dtypes, int out_dtype,
                              void* const* d_out_all,
                              void* const* d_ws_all, size_t ws_bytes) {
    (void)comm; (void)in_sizes; (void)n_in; (void)in_dtypes; (void)out_dtype; (void)d_out_all;
    if (nranks != NR || (unsigned)rank >= (unsigned)NR || ws_bytes < WS_NEEDED || out_size != MM * NN) {
        fprintf(stderr, "kernel_launch: unexpected configuration\n");
        return;
    }
    HIP_CHECK(hipFuncSetAttribute((const void*)fused_gemm_ag, hipFuncAttributeMaxDynamicSharedMemorySize, LDS_BYTES));
    Args a;
    memset(&a, 0, sizeof a);
    a.x = (const char*)d_in[0];
    a.w = (const char*)d_in[1];
    a.sx = (const float*)d_in[2];
    a.sw = (const float*)d_in[3];
    a.out = (float*)d_out;
    for (int j = 0; j < NR; ++j) a.ws[j] = (char*)d_ws_all[(rank + 1 + j) % NR];
    a.rank = rank;
    for (int g = 0; g < NGEMM; ++g) {
        const int xcd = g & 7, idx = g >> 3, I = idx % NBI, J = xcd * 2 + idx / NBI;
        const int u0 = (MT * I) >> 2;
        int u1 = MT * I + MT - 1; if (u1 > 255) u1 = 255; u1 >>= 2;
        int key[NR];
        for (int r = 0; r < NR; ++r) {
            int mx = 0;
            for (int u = u0; u <= u1; ++u) { int p = pos_of(u, r); if (p > mx) mx = p; }
            for (int u = 64 + 4 * J; u < 64 + 4 * J + 4; ++u) { int p = pos_of(u, r); if (p > mx) mx = p; }
            key[r] = mx * 8 + r;
        }
        unsigned ord = (unsigned)rank;
        int used = 1 << rank;
        for (int k = 1; k < 8; ++k) {
            int best = -1;
            for (int r = 0; r < NR; ++r) if (!((used >> r) & 1) && (best < 0 || key[r] < key[best])) best = r;
            used |= 1 << best;
            ord |= (unsigned)best << (4 * k);
        }
        a.order[g] = ord;
    }
    hipLaunchKernelGGL(fused_gemm_ag, dim3(GRID), dim3(THREADS), LDS_BYTES, stream, a);
    HIP_CHECK(hipGetLastError());
}
```

```cpp
#include <hip/hip_runtime.h>
#include <rccl/rccl.h>
#include <cstdio>
#include <cstdint>
#include <cstring>

#define HIP_CHECK(expr) do { hipError_t _e = (expr); if (_e != hipSuccess) { \
    fprintf(stderr, "HIP error %s at %s:%d\n", hipGetErrorString(_e), __FILE__, __LINE__); \
    return; } } while (0)

namespace {

typedef unsigned u32x4 __attribute__((ext_vector_type(4)));
typedef int i32x4 __attribute__((ext_vector_type(4)));
typedef float f32x4 __attribute__((ext_vector_type(4)));
typedef __attribute__((address_space(1))) unsigned gu32;

constexpr int NR = 8;
constexpr int MM = 4096, NN = 4096;
constexpr int KSH = 512;
constexpr int NCOMM = 16, NGEMM = 240, GRID = NCOMM + NGEMM, THREADS = 512;
#define CW_ 1
constexpr int CW = CW_;
constexpr int TW = NCOMM * CW;
constexpr int UPH = 64 / TW;
constexpr int MT = 18;
constexpr int NBI = 15;
constexpr int XFR = NBI * MT;
constexpr int WFR = 256;
constexpr unsigned STEP_BYTES = (XFR + WFR) * 1024u;
constexpr unsigned SLOT_BYTES = 64u * STEP_BYTES;
constexpr unsigned CNT_OFF = 0;
constexpr unsigned TMO_OFF = 16384;
#define FSTRIDE 128
constexpr unsigned FLAG_STRIDE = FSTRIDE;
constexpr unsigned FLAG_OFF = 1u << 20;
constexpr unsigned DATA_OFF = 2u << 20;
constexpr size_t WS_NEEDED = (size_t)DATA_OFF + 2ull * SLOT_BYTES;
constexpr int STAGE_FR = MT + 16;
constexpr int STAGE_BYTES = STAGE_FR * 1024;
constexpr int RING = 4;
constexpr int CTRL_LDS = RING * STAGE_BYTES;
constexpr int PB_LDS = CTRL_LDS + 64;
constexpr int QT_LDS = PB_LDS + 256;
constexpr int QT_BYTES = 68 * 4;
constexpr int LDS_BYTES = QT_LDS + QT_BYTES;
constexpr unsigned long long DEADLINE_TICKS = 50000000ull;
#define LINGER 60
#define POLL_BITS "sc0 sc1"
#define EPI_BITS "sc1"
#define PSLEEP 4
#define LF_ 3
constexpr int LF = LF_;

struct Args {
    const char* x;
    const char* w;
    const float* sx;
    const float* sw;
    float* out;
    char* ws[NR];
    int rank;
    int mode;
    unsigned order[NGEMM];
};

__host__ __device__ inline int unit_at(int pos, int r) { return pos < 64 ? 64 + ((pos - 8 * r) & 63) : ((pos - 8 * r) & 63); }
__host__ __device__ inline int pos_of(int v, int r) { return v >= 64 ? ((v + 8 * r) & 63) : 64 + ((v + 8 * r) & 63); }

__device__ __forceinline__ void st16_sys(unsigned off, u32x4 d, const char* base, int imm) {
    asm volatile("global_store_dwordx4 %0, %1, %2 offset:%3 sc0 sc1\n\ts_nop 1"
                 :: "v"(off), "v"(d), "s"(base), "i"(imm) : "memory");
}
__device__ __forceinline__ void st4_sys(uint64_t addr, unsigned val) {
    asm volatile("global_store_dword %0, %1, off sc0 sc1" :: "v"(addr), "v"(val) : "memory");
}
template <int N> __device__ __forceinline__ void wait_vm() {
    asm volatile("s_waitcnt vmcnt(%0)" :: "i"(N) : "memory");
}
#define GLDS_BITS " sc0 sc1"
__device__ __forceinline__ void glds16_sys(unsigned voff, const char* sbase, unsigned ldsaddr) {
    unsigned keep;
    asm volatile("s_mov_b32 %0, m0\n\ts_mov_b32 m0, %3\n\ts_nop 0\n\t"
                 "global_load_lds_dwordx4 %1, %2" GLDS_BITS "\n\ts_mov_b32 m0, %0"
                 : "=&s"(keep) : "v"(voff), "s"(sbase), "s"(ldsaddr) : "memory");
}
__device__ __forceinline__ void glds4_sys(unsigned voff, const char* sbase, unsigned ldsaddr) {
    unsigned keep;
    asm volatile("s_mov_b32 %0, m0\n\ts_mov_b32 m0, %3\n\ts_nop 0\n\t"
                 "global_load_lds_dword %1, %2 " POLL_BITS "\n\ts_mov_b32 m0, %0"
                 : "=&s"(keep) : "v"(voff), "s"(sbase), "s"(ldsaddr) : "memory");
}
#define WG_BARRIER_NOMEM() do { asm volatile("" ::: "memory"); __builtin_amdgcn_s_barrier(); asm volatile("" ::: "memory"); } while (0)
#define WG_BARRIER() do { asm volatile("" ::: "memory"); __builtin_amdgcn_s_waitcnt(0xC07F); __builtin_amdgcn_s_barrier(); asm volatile("" ::: "memory"); } while (0)

struct RawW { unsigned d[16]; };
struct RawX { u32x4 q[4]; };

__device__ __forceinline__ const char* uni(const char* p) {
    const uint64_t v = (uint64_t)p;
    const unsigned lo = __builtin_amdgcn_readfirstlane((unsigned)v), hi = __builtin_amdgcn_readfirstlane((unsigned)(v >> 32));
    return (const char*)(((uint64_t)hi << 32) | lo);
}
__device__ __forceinline__ void raw_issue(RawW& r, const char* src, int v, int s, int lane) {
    const unsigned off = (unsigned)(64 * s + 16 * (lane >> 4)) * NN + (unsigned)(v - 64) * 64u + 4u * (lane & 15);
#pragma unroll
    for (int j = 0; j < 16; ++j)
        asm volatile("global_load_dword %0, %1, %2" : "=v"(r.d[j]) : "v"(off + (unsigned)j * NN), "s"(src) : "memory");
}
__device__ __forceinline__ void raw_issue(RawX& r, const char* src, int v, int s, int lane) {
    const unsigned off = (unsigned)(64 * v + (lane & 15)) * KSH + 64u * s + 16u * (lane >> 4);
#pragma unroll
    for (int f = 0; f < 4; ++f)
        asm volatile("global_load_dwordx4 %0, %1, %2" : "=v"(r.q[f]) : "v"(off + (unsigned)f * 16u * KSH), "s"(src) : "memory");
}
__device__ __forceinline__ void raw_pin(RawW& r) {
#pragma unroll
    for (int j = 0; j < 16; ++j) asm volatile("" : "+v"(r.d[j]));
}
__device__ __forceinline__ void raw_pin(RawX& r) {
#pragma unroll
    for (int f = 0; f < 4; ++f) asm volatile("" : "+v"(r.q[f]));
}
__device__ __forceinline__ void raw_format(u32x4 (&F)[4], const RawW& r) {
#pragma unroll
    for (int q4 = 0; q4 < 4; ++q4) {
        const unsigned r0 = r.d[4 * q4], r1 = r.d[4 * q4 + 1], r2 = r.d[4 * q4 + 2], r3 = r.d[4 * q4 + 3];
        const unsigned t0 = __builtin_amdgcn_perm(r1, r0, 0x05010400u), t1 = __builtin_amdgcn_perm(r1, r0, 0x07030602u);
        const unsigned t2 = __builtin_amdgcn_perm(r3, r2, 0x05010400u), t3 = __builtin_amdgcn_perm(r3, r2, 0x07030602u);
        F[0][q4] = __builtin_amdgcn_perm(t2, t0, 0x05040100u);
        F[1][q4] = __builtin_amdgcn_perm(t2, t0, 0x07060302u);
        F[2][q4] = __builtin_amdgcn_perm(t3, t1, 0x05040100u);
        F[3][q4] = __builtin_amdgcn_perm(t3, t1, 0x07060302u);
    }
#pragma unroll
    for (int f = 0; f < 4; ++f) asm volatile("" : "+v"(F[f]));
}
__device__ __forceinline__ void raw_format(u32x4 (&F)[4], const RawX& r) {
#pragma unroll
    for (int f = 0; f < 4; ++f) { F[f] = r.q[f]; asm volatile("" : "+v"(F[f])); }
}
__device__ __forceinline__ unsigned unit_doff(int rank, int s, int v, unsigned seq) {
    return DATA_OFF + (seq & 1u) * SLOT_BYTES + (unsigned)(rank * 8 + s) * STEP_BYTES
         + (unsigned)(v >= 64 ? XFR + 4 * (v - 64) : 4 * v) * 1024u;
}
__device__ __forceinline__ unsigned unit_foff(int rank, int s, int v) {
    return FLAG_OFF + (unsigned)((rank * 8 + s) * 128 + v) * FLAG_STRIDE;
}

template <int J> struct IC { static constexpr int value = J; };
template <bool B> struct BC { static constexpr bool value = B; };
template <int J, int N, class Fn> __device__ __forceinline__ void static_for(Fn&& fn) {
    if constexpr (J < N) { fn(IC<J>{}); static_for<J + 1, N>(fn); }
}

typedef __attribute__((address_space(3))) int lds_int;

__device__ __forceinline__ unsigned next_seq(const Args& a, int b, int wave, int lane) {
    unsigned old = 0;
    if (lane == 0)
        old = __hip_atomic_fetch_add((unsigned*)(a.ws[7] + CNT_OFF) + b * 8 + wave, 1u, __ATOMIC_RELAXED, __HIP_MEMORY_SCOPE_AGENT);
    return __builtin_amdgcn_readfirstlane(old) + 1u;
}

__device__ __forceinline__ void comm_wave(const Args& a, int gw, int lane, int b, int wave, volatile lds_int* done) {
    const char* base[7];
#pragma unroll
    for (int p = 0; p < 7; ++p) base[p] = uni(a.ws[p]);
    const char* wsrc = uni(a.w);
    const char* xsrc = uni(a.x);
    const int rank = __builtin_amdgcn_readfirstlane(a.rank);
    RawW rw; RawX rx;
    raw_issue(rw, wsrc, unit_at(gw, rank), 0, lane);
    const unsigned seq = next_seq(a, b, wave, lane);
    wait_vm<0>();
    raw_pin(rw);
    for (int s = 0; s < 8; ++s) {
        static_for<0, 2 * UPH>([&](auto jc) {
            constexpr int j = decltype(jc)::value;
            constexpr bool cur_w = j < UPH, next_w = (j + 1 < UPH) || (j + 1 == 2 * UPH);
            const int v = unit_at(j * TW + gw, rank);
            u32x4 F[4];
            if constexpr (cur_w) raw_format(F, rw); else raw_format(F, rx);
            const unsigned doff = unit_doff(rank, s, v, seq) + lane * 16u;
            const int sn = (j + 1 == 2 * UPH) ? (s < 7 ? s + 1 : 7) : s;
            const int vn = unit_at(((j + 1) % (2 * UPH)) * TW + gw, rank);
            if constexpr (next_w) raw_issue(rw, wsrc, vn, sn, lane); else raw_issue(rx, xsrc, vn, sn, lane);
#pragma unroll
            for (int f = 0; f < 4; ++f) {
#pragma unroll
                for (int p = 0; p < 7; ++p) st16_sys(doff, F[f], base[p], f * 1024);
                if (f == LF - 1) {
                    wait_vm<7 * LF + 4>();
                    if (lane == 0) done[0] = s * 2 * UPH + j;
                } else if (f >= LF) wait_vm<7 * LF>();
            }
            wait_vm<28>();
            if constexpr (next_w) raw_pin(rw); else raw_pin(rx);
        });
    }
    wait_vm<0>();
    if (lane == 0) done[0] = 8 * 2 * UPH;
}

__device__ __forceinline__ void flag_wave(const Args& a, int gw0, int lane, unsigned seq, volatile lds_int* done) {
    uint64_t fl = (uint64_t)a.ws[lane < 7 ? lane : 0];
    asm volatile("" : "+v"(fl));
    constexpr int NU = 16 * UPH;
    int sent[CW];
#pragma unroll
    for (int w = 0; w < CW; ++w) sent[w] = 0;
    const unsigned long long t0 = __builtin_amdgcn_s_memrealtime();
    while (true) {
        bool all = true;
#pragma unroll
        for (int w = 0; w < CW; ++w) {
            const int d = __builtin_amdgcn_readfirstlane(done[w * 16]);
            while (sent[w] < d) {
                const int k = sent[w]++;
                const int s = k / (2 * UPH), j = k % (2 * UPH);
                const int v = unit_at(j * TW + gw0 + w, a.rank);
                if (lane < 7) st4_sys(fl + unit_foff(a.rank, s, v), seq);
            }
            all = all && sent[w] == NU;
        }
        if (all || __builtin_amdgcn_s_memrealtime() - t0 > DEADLINE_TICKS) break;
        __builtin_amdgcn_s_sleep(1);
    }
}

__device__ __forceinline__ void gemm_wg(const Args& a, char* smem, int gidx, int wave, int lane, int b) {
    unsigned seq;
    const int xcd = gidx & 7, idx = gidx >> 3;
    const int I = idx % NBI, J = xcd * 2 + idx / NBI;
    const int wr = wave >> 2, wc = wave & 3;
    char* myws = (char*)uni(a.ws[7]);

    volatile lds_int* cnt_lds = (volatile lds_int*)((__attribute__((address_space(3))) char*)smem + CTRL_LDS);
    volatile lds_int* pb_lds = (volatile lds_int*)((__attribute__((address_space(3))) char*)smem + PB_LDS);
    volatile lds_int* qt_lds = (volatile lds_int*)((__attribute__((address_space(3))) char*)smem + QT_LDS);
    if (threadIdx.x < 68) qt_lds[threadIdx.x] = 0;
    if (threadIdx.x == 0) cnt_lds[0] = 0;

    {
        const int u = gidx * 8 + wave;
        if (u >= 1024) seq = next_seq(a, b, wave, lane);
        else {
            u32x4 F[4];
            const int s = u >> 7, v = u & 127;
            if (v >= 64) { RawW r; raw_issue(r, uni(a.w), v, s, lane); seq = next_seq(a, b, wave, lane); wait_vm<0>(); raw_pin(r); raw_format(F, r); }
            else         { RawX r; raw_issue(r, uni(a.x), v, s, lane); seq = next_seq(a, b, wave, lane); wait_vm<0>(); raw_pin(r); raw_format(F, r); }
            const unsigned doff = unit_doff(a.rank, s, v, seq) + lane * 16u;
#pragma unroll
            for (int f = 0; f < 4; ++f) st16_sys(doff, F[f], myws, f * 1024);
            wait_vm<0>();
            if (lane == 0) __hip_atomic_store((gu32*)(myws + unit_foff(a.rank, s, v)), seq, __ATOMIC_RELAXED, __HIP_MEMORY_SCOPE_SYSTEM);
        }
    }

    seq = __builtin_amdgcn_readfirstlane(seq);
    const char* data = myws + DATA_OFF + (size_t)(seq & 1u) * SLOT_BYTES;
    const unsigned lds0 = (unsigned)(uintptr_t)(__attribute__((address_space(3))) char*)smem;
    const bool g1 = wave >= 4;
    const bool poller = wave == 3;
    const int dv = wave < 3 ? wave : wave - 1;
    const char* sbase[5];
    unsigned ldst[5];
#pragma unroll
    for (int i = 0; i < 5; ++i) {
        int f = 5 * dv + i; if (f > STAGE_FR - 1) f = STAGE_FR - 1;
        sbase[i] = data + (size_t)(f < MT ? MT * I + f : XFR + 16 * J + (f - MT)) * 1024u;
        ldst[i] = lds0 + f * 1024u;
    }
    const unsigned lane16 = lane * 16u;
    const int strm = lane / 9 < 7 ? lane / 9 : 6, el = lane - 9 * (lane / 9);
    unsigned pfoff;
    {
        const int u0 = (MT * I) >> 2;
        int u1 = (MT * I + MT - 1); if (u1 > 255) u1 = 255; u1 >>= 2;
        int fi;
        if (el < 5) { fi = u0 + el; if (fi > u1) fi = u1; }
        else fi = 64 + 4 * J + (el - 5);
        if (lane == 63) fi = u0;
        pfoff = FLAG_OFF + (unsigned)fi * FLAG_STRIDE;
    }
    const int psrc8 = (int)((a.order[gidx] >> (4 * (strm + 1))) & 7u) * 8;
    int pc = strm == 0 ? -2 : -1;
    auto stream_q = [&]() {
        const int own = a.rank * 8 + (pc == -1 && strm == 0 ? 7 : strm);
        return pc >= 0 ? psrc8 + (pc < 8 ? pc : 7) : own;
    };
    int npub = 0;
    auto poll_issue = [&]() {
        glds4_sys((unsigned)stream_q() * (128u * FLAG_STRIDE) + pfoff, myws, lds0 + PB_LDS);
    };
    auto poll_take = [&]() {
        wait_vm<0>();
        const unsigned long long ok = __ballot((unsigned)pb_lds[lane] >= seq);
        const bool rdy = ((unsigned)(ok >> (9 * strm)) & 0x1FFu) == 0x1FFu && pc < 8;
        const unsigned long long lead = __ballot(rdy && el == 0 && lane < 63);
        if (__builtin_amdgcn_readfirstlane((unsigned)lead | (unsigned)(lead >> 32))) {
            const int pos = npub + (int)__builtin_amdgcn_mbcnt_hi((unsigned)(lead >> 32), __builtin_amdgcn_mbcnt_lo((unsigned)lead, 0u));
            if (rdy && el == 0 && lane < 63) qt_lds[pos] = stream_q();
            npub = __builtin_amdgcn_readfirstlane(npub + __popcll(lead));
            if (lane == 0) cnt_lds[0] = npub;
            pc += rdy ? 1 : 0;
        }
    };
    auto voff_from = [&](unsigned q) { return q * STEP_BYTES + lane16; };
    auto issue = [&](int t, unsigned voff) {
        const unsigned so = (unsigned)(t & (RING - 1)) * STAGE_BYTES;
#pragma unroll
        for (int i = 0; i < 5; ++i) glds16_sys(voff, sbase[i], ldst[i] + so);
    };
    i32x4 acc[9][4];
#pragma unroll
    for (int m = 0; m < 9; ++m)
#pragma unroll
        for (int n = 0; n < 4; ++n) acc[m][n] = (i32x4){0, 0, 0, 0};
    i32x4 Af[9], Bf[4];
    int iss = 0;
#define SB() __builtin_amdgcn_sched_barrier(0)
#define LIKELY(x) __builtin_expect(!!(x), 1)
#define UNLIKELY(x) __builtin_expect(!!(x), 0)

    auto loader_try = [&](int lim) {
        int r = __builtin_amdgcn_readfirstlane(cnt_lds[0]);
        if (r > lim) r = lim;
        const bool any = iss < r;
        for (; iss < r; ++iss) issue(iss, voff_from((unsigned)qt_lds[iss]));
        return any;
    };
    auto loader_block = [&](int u, int lim) {
        while (true) {
            loader_try(lim);
            if (iss > u) break;
            __builtin_amdgcn_s_sleep(1);
        }
        const unsigned long long t0 = __builtin_amdgcn_s_memrealtime();
        while (iss < lim && iss < 64 && __builtin_amdgcn_s_memrealtime() - t0 < LINGER)
            if (!loader_try(lim)) __builtin_amdgcn_s_sleep(1);
    };
    bool pending = false;
    auto poller_block = [&](int u, int lim) {
        gu32* tmo = (gu32*)(myws + TMO_OFF);
        unsigned long long t0 = __builtin_amdgcn_s_memrealtime();
        if (pending) { poll_take(); pending = false; }
        for (unsigned spins = 1; npub <= u; ++spins) {
            poll_issue();
            poll_take();
            if (npub > u) break;
            __builtin_amdgcn_s_sleep(PSLEEP);
            if ((spins & 15u) == 0u) {
                const unsigned tv = __hip_atomic_load(tmo, __ATOMIC_RELAXED, __HIP_MEMORY_SCOPE_AGENT);
                if (__builtin_amdgcn_readfirstlane(tv) == seq || __builtin_amdgcn_s_memrealtime() - t0 > DEADLINE_TICKS) {
                    if (lane == 0) __hip_atomic_store(tmo, seq, __ATOMIC_RELAXED, __HIP_MEMORY_SCOPE_AGENT);
                    npub = 64;
                    if (lane == 0) cnt_lds[0] = 64;
                }
            }
        }
        t0 = __builtin_amdgcn_s_memrealtime();
        while (npub < lim && npub < 64 && __builtin_amdgcn_s_memrealtime() - t0 < LINGER) { poll_issue(); poll_take(); }
    };
    auto wait_landed = [&](int u) {
        const int ahead = iss - u - 1;
        wait_vm<10>();
        if (UNLIKELY(ahead < 2)) { wait_vm<5>(); if (ahead < 1) wait_vm<0>(); }
    };

    auto read_frags = [&](int t, auto dma, unsigned voff) {
        const char* st = smem + (t & (RING - 1)) * STAGE_BYTES + lane16;
        const char* sa = st + wr * 9 * 1024;
        const char* sb = st + MT * 1024 + wc * 4096;
        const unsigned so = (unsigned)(iss & (RING - 1)) * STAGE_BYTES;
        constexpr bool D = decltype(dma)::value;
        if constexpr (D) { glds16_sys(voff, sbase[0], ldst[0] + so); SB(); }
#pragma unroll
        for (int n = 0; n < 4; ++n) Bf[n] = *(const i32x4*)(sb + n * 1024);
        if constexpr (D) { SB(); glds16_sys(voff, sbase[1], ldst[1] + so); SB(); }
#pragma unroll
        for (int m = 0; m < 3; ++m) Af[m] = *(const i32x4*)(sa + m * 1024);
        if constexpr (D) { SB(); glds16_sys(voff, sbase[2], ldst[2] + so); SB(); }
#pragma unroll
        for (int m = 3; m < 6; ++m) Af[m] = *(const i32x4*)(sa + m * 1024);
        if constexpr (D) { SB(); glds16_sys(voff, sbase[3], ldst[3] + so); SB(); }
#pragma unroll
        for (int m = 6; m < 9; ++m) Af[m] = *(const i32x4*)(sa + m * 1024);
        if constexpr (D) { SB(); glds16_sys(voff, sbase[4], ldst[4] + so); }
        SB();
    };
    auto mma_rows = [&](int m0, int m1) {
#pragma unroll
        for (int m = m0; m < m1; ++m)
#pragma unroll
            for (int n = 0; n < 4; ++n) {
                acc[m][n] = __builtin_amdgcn_mfma_i32_16x16x64_i8(Af[m], Bf[n], acc[m][n], 0, 0, 0);
            }
    };
    auto mma_phase = [&](auto&& h1, auto&& h2, auto&& h3) {
        __builtin_amdgcn_s_setprio(1);
        mma_rows(0, 3);
        SB();
        h1();
        SB();
        mma_rows(3, 4);
        SB();
        h2();
        SB();
        mma_rows(4, 5);
        SB();
        h3();
        SB();
        mma_rows(5, 9);
        __builtin_amdgcn_s_setprio(0);
    };
#define STAMP0()
#define STAMP(i)
    typedef BC<true> True_;
    typedef BC<false> False_;
    WG_BARRIER();
    if (poller) {
        poller_block(0, RING);
        WG_BARRIER();
        STAMP0();
        for (int t = 0; t < 64; ++t) {
            read_frags(t, False_{}, 0u);
            if (pending) { poll_take(); pending = false; }
            if (npub < 64 && npub <= t + 1 + RING) { poll_issue(); pending = true; }
            STAMP(0);
            WG_BARRIER();
            STAMP(1);
            SB();
            mma_phase([]() {}, []() {}, []() {});
            SB();
            STAMP(2);
            if (UNLIKELY(npub <= t + 1 && t < 63)) poller_block(t + 1, t + RING);
            STAMP(3);
            WG_BARRIER();
            STAMP(4);
        }
    } else if (!g1) {
        loader_block(0, RING);
        wait_landed(0);
        bool more = iss < RING && iss < __builtin_amdgcn_readfirstlane(cnt_lds[0]);
        unsigned voff = voff_from((unsigned)qt_lds[iss]);
        WG_BARRIER();
        STAMP0();
        for (int t = 0; t < 64; ++t) {
            if (LIKELY(more)) { read_frags(t, True_{}, voff); ++iss; } else read_frags(t, False_{}, voff);
            STAMP(0);
            WG_BARRIER();
            STAMP(1);
            const int cv = cnt_lds[0];
            const unsigned qv = (unsigned)qt_lds[iss];
            SB();
            int cs;
            mma_phase([&]() { cs = __builtin_amdgcn_readfirstlane(cv); },
                      [&]() { more = (iss < t + 1 + RING) & (iss < cs); },
                      [&]() { voff = voff_from(qv); });
            SB();
            STAMP(2);
            if (LIKELY(t < 63)) {
                if (UNLIKELY(iss <= t + 1)) {
                    loader_block(t + 1, t + RING);
                    more = iss < t + 1 + RING && iss < __builtin_amdgcn_readfirstlane(cnt_lds[0]);
                    voff = voff_from((unsigned)qt_lds[iss]);
                }
                wait_landed(t + 1);
            }
            STAMP(3);
            WG_BARRIER();
            STAMP(4);
        }
    } else {
        loader_block(0, RING);
        wait_landed(0);
        WG_BARRIER();
        bool more = iss < RING && iss < __builtin_amdgcn_readfirstlane(cnt_lds[0]);
        unsigned voff = voff_from((unsigned)qt_lds[iss]);
        WG_BARRIER();
        STAMP0();
        for (int t = 0; t < 64; ++t) {
            if (LIKELY(more)) { read_frags(t, True_{}, voff); ++iss; } else read_frags(t, False_{}, voff);
            STAMP(0);
            if (LIKELY(t < 63)) {
                if (UNLIKELY(iss <= t + 1)) loader_block(t + 1, t + RING);
                wait_landed(t + 1);
            }
            STAMP(3);
            WG_BARRIER();
            STAMP(1);
            const int cv = cnt_lds[0];
            const unsigned qv = (unsigned)qt_lds[iss];
            SB();
            int cs;
            mma_phase([&]() { cs = __builtin_amdgcn_readfirstlane(cv); },
                      [&]() { more = (iss < t + 1 + RING) & (iss < cs); },
                      [&]() { voff = voff_from(qv); });
            SB();
            STAMP(2);
            if (LIKELY(t < 63)) WG_BARRIER();
            STAMP(4);
        }
    }

    const unsigned tv = __hip_atomic_load((gu32*)(myws + TMO_OFF), __ATOMIC_RELAXED, __HIP_MEMORY_SCOPE_AGENT);
    const bool bad = __builtin_amdgcn_readfirstlane(tv) == seq;
    const float scale = a.sx[0] * a.sw[0];
    float* outp = a.out + (size_t)(I * (MT * 16) + wr * 144 + (lane >> 4) * 4) * NN + J * 256 + wc * 64 + (lane & 15) * 4;
    auto put = [&](int m, int r, f32x4 o) {
        asm volatile("global_store_dwordx4 %0, %1, off " EPI_BITS "\n\ts_nop 1" :: "v"(outp + (size_t)(m * 16 + r) * NN), "v"(o) : "memory");
    };
    if (UNLIKELY(bad)) {
        const float qn = __builtin_nanf("");
        for (int m = 0; m < 9; ++m)
            if (MT * I + wr * 9 + m < 256)
                for (int r = 0; r < 4; ++r) put(m, r, (f32x4){qn, qn, qn, qn});
        return;
    }
#pragma unroll
    for (int m = 0; m < 9; ++m) {
        if (MT * I + wr * 9 + m < 256) {
#pragma unroll
            for (int r = 0; r < 4; ++r) {
                f32x4 o;
#pragma unroll
                for (int n = 0; n < 4; ++n) {
                    float y = (float)acc[m][n][r] * scale;
                    float c = __builtin_amdgcn_fmed3f(y, -60.0f, 60.0f);
                    float e = __builtin_amdgcn_exp2f(c * -1.4426950408889634f);
                    o[n] = y * __builtin_amdgcn_rcpf(1.0f + e);
                }
                put(m, r, o);
            }
        }
    }
}

__global__ void __launch_bounds__(THREADS, 2) fused_gemm_ag(Args a) {
    extern __shared__ __attribute__((aligned(16))) char smem[];
    const int b = blockIdx.x;
    const int wave = __builtin_amdgcn_readfirstlane(threadIdx.x >> 6);
    const int lane = threadIdx.x & 63;
    if (b < NCOMM && wave >= CW && wave != 7) return;
    if (b < NCOMM) {
        volatile lds_int* done = (volatile lds_int*)((__attribute__((address_space(3))) char*)smem);
        if (wave == 7) {
            if (lane < CW) done[lane * 16] = 0;
            WG_BARRIER();
            flag_wave(a, b * CW, lane, next_seq(a, b, wave, lane), done);
        } else {
            WG_BARRIER_NOMEM();
            comm_wave(a, b * CW + wave, lane, b, wave, done + wave * 16);
        }
    } else {
        gemm_wg(a, smem, b - NCOMM, wave, lane, b);
    }
}

}

extern "C" void kernel_launch(int rank, int nranks, ncclComm_t comm, hipStream_t stream,
                              void* const* d_in, const int* in_sizes, int n_in,
                              void* d_out, int out_size,
                              const int* in_dtypes, int out_dtype,
                              void* const* d_out_all,
                              void* const* d_ws_all, size_t ws_bytes) {
    (void)comm; (void)in_sizes; (void)n_in; (void)in_dtypes; (void)out_dtype; (void)d_out_all;
    if (nranks != NR || (unsigned)rank >= (unsigned)NR || ws_bytes < WS_NEEDED || out_size != MM * NN) {
        fprintf(stderr, "kernel_launch: unexpected configuration\n");
        return;
    }
    HIP_CHECK(hipFuncSetAttribute((const void*)fused_gemm_ag, hipFuncAttributeMaxDynamicSharedMemorySize, LDS_BYTES));
    Args a;
    memset(&a, 0, sizeof a);
    a.x = (const char*)d_in[0];
    a.w = (const char*)d_in[1];
    a.sx = (const float*)d_in[2];
    a.sw = (const float*)d_in[3];
    a.out = (float*)d_out;
    for (int j = 0; j < NR; ++j) a.ws[j] = (char*)d_ws_all[(rank + 1 + j) % NR];
    a.rank = rank;
    for (int g = 0; g < NGEMM; ++g) {
        const int xcd = g & 7, idx = g >> 3, I = idx % NBI, J = xcd * 2 + idx / NBI;
        const int u0 = (MT * I) >> 2;
        int u1 = MT * I + MT - 1; if (u1 > 255) u1 = 255; u1 >>= 2;
        int key[NR];
        for (int r = 0; r < NR; ++r) {
            int mx = 0;
            for (int u = u0; u <= u1; ++u) { int p = pos_of(u, r); if (p > mx) mx = p; }
            for (int u = 64 + 4 * J; u < 64 + 4 * J + 4; ++u) { int p = pos_of(u, r); if (p > mx) mx = p; }
            key[r] = mx * 8 + r;
        }
        unsigned ord = (unsigned)rank;
        int used = 1 << rank;
        for (int k = 1; k < 8; ++k) {
            int best = -1;
            for (int r = 0; r < NR; ++r) if (!((used >> r) & 1) && (best < 0 || key[r] < key[best])) best = r;
            used |= 1 << best;
            ord |= (unsigned)best << (4 * k);
        }
        a.order[g] = ord;
    }
    hipLaunchKernelGGL(fused_gemm_ag, dim3(GRID), dim3(THREADS), LDS_BYTES, stream, a);
    HIP_CHECK(hipGetLastError());
}
```

```cpp
#include <hip/hip_runtime.h>
#include <rccl/rccl.h>
#include <cstdio>
#include <cstdint>
#include <cstring>

#define HIP_CHECK(expr) do { hipError_t _e = (expr); if (_e != hipSuccess) { \
    fprintf(stderr, "HIP error %s at %s:%d\n", hipGetErrorString(_e), __FILE__, __LINE__); \
    return; } } while (0)

namespace {

typedef unsigned u32x4 __attribute__((ext_vector_type(4)));
typedef int i32x4 __attribute__((ext_vector_type(4)));
typedef float f32x4 __attribute__((ext_vector_type(4)));
typedef __attribute__((address_space(1))) unsigned gu32;

constexpr int NR = 8;
constexpr int MM = 4096, NN = 4096;
constexpr int KSH = 512;
constexpr int NCOMM = 16, NGEMM = 240, GRID = NCOMM + NGEMM, THREADS = 512;
constexpr int CW = 1;
constexpr int TW = NCOMM * CW;
#define NDEL_ 1
constexpr int NDEL = NDEL_;
static_assert(NDEL >= 1 && NDEL <= 3, "");
constexpr int NT = 64 + 7 * NDEL;
constexpr int MT = 18;
constexpr int NBI = 15;
constexpr int XFR = NBI * MT;
constexpr int WFR = 256;
constexpr unsigned STEP_BYTES = (XFR + WFR) * 1024u;
constexpr unsigned SLOT_BYTES = 64u * STEP_BYTES;
constexpr unsigned CNT_OFF = 0;
constexpr unsigned TMO_OFF = 16384;
#define FSTRIDE 128
constexpr unsigned FLAG_STRIDE = FSTRIDE;
constexpr unsigned FLAG_OFF = 1u << 20;
static_assert(64 * 128 * FLAG_STRIDE <= (1u << 20), "");
constexpr unsigned DATA_OFF = 2u << 20;
constexpr size_t WS_NEEDED = (size_t)DATA_OFF + 2ull * SLOT_BYTES;
constexpr int STAGE_FR = MT + 16;
constexpr int STAGE_BYTES = STAGE_FR * 1024;
constexpr int RING = 4;
constexpr int CTRL_LDS = RING * STAGE_BYTES;
constexpr int PB_LDS = CTRL_LDS + 64;
constexpr int ST_LDS = PB_LDS + 512;
constexpr int QT_LDS = ST_LDS + 7 * 64;
constexpr int QT_WORDS = 96;
constexpr int QT_BYTES = QT_WORDS * 4;
constexpr int LDS_BYTES = QT_LDS + QT_BYTES;
constexpr unsigned long long DEADLINE_TICKS = 50000000ull;
#define LINGER 60
#define POLL_BITS "sc0 sc1"
#define EPI_BITS "sc1"
#define EPI_WIN 4
#define PDEPTH 1
#define PSLEEP 4
#define LF_ 3
constexpr int LF = LF_;

struct Args {
    const char* x;
    const char* w;
    const float* sx;
    const float* sw;
    float* out;
    char* ws[NR];
    int rank;
    int mode;
    unsigned order[NGEMM];
};

__host__ __device__ inline int pos_of(int v, int r) { return v >= 64 ? ((v + 8 * r) & 63) : 64 + ((v + 8 * r) & 63); }
__host__ __device__ inline void item_of(int k, int gw, int r, int& s, int& v) {
    constexpr int N1 = 8 * (8 - NDEL), N2 = N1 + 6 * NDEL;
    if (k < N1) {
        s = k >> 3;
        const int j = k & 7, u = ((j & 3) * 16 + gw - 8 * r) & 63;
        v = j < 4 ? 64 + u : u;
    } else if (k < N2) {
        const int kk = k - N1, j = kk % 6;
        s = 8 - NDEL + kk / 6;
        if (j < 2) { const int idx = (j * 16 + gw - 4 * r) & 31; v = 64 + 4 * (idx >> 1) + (idx & 1); }
        else v = ((j - 2) * 16 + gw - 8 * r) & 63;
    } else {
        const int kk = (k < 63 ? k : 63) - N2, idx = ((kk & 1) * 16 + gw - 4 * r) & 31;
        s = 8 - NDEL + (kk >> 1);
        v = 64 + 4 * (idx >> 1) + 2 + (idx & 1);
    }
}

__device__ __forceinline__ void st16_sys(unsigned off, u32x4 d, const char* base, int imm) {
    asm volatile("global_store_dwordx4 %0, %1, %2 offset:%3 sc0 sc1\n\ts_nop 1"
                 :: "v"(off), "v"(d), "s"(base), "i"(imm) : "memory");
}
__device__ __forceinline__ void st4_sys(uint64_t addr, unsigned val) {
    asm volatile("global_store_dword %0, %1, off sc0 sc1" :: "v"(addr), "v"(val) : "memory");
}
__device__ __forceinline__ int smin3(int a, int b, int c) {
    int r;
    asm volatile("s_min_i32 %0, %1, %2\n\ts_min_i32 %0, %0, %3" : "=&s"(r) : "s"(a), "s"(b), "s"(c) : "scc");
    return r;
}
template <int N> __device__ __forceinline__ void wait_vm() {
    asm volatile("s_waitcnt vmcnt(%0)" :: "i"(N) : "memory");
}
#define GLDS_BITS " sc0 sc1"
__device__ __forceinline__ void glds16_sys(unsigned voff, const char* sbase, unsigned ldsaddr) {
    unsigned keep;
    asm volatile("s_mov_b32 %0, m0\n\ts_mov_b32 m0, %3\n\ts_nop 0\n\t"
                 "global_load_lds_dwordx4 %1, %2" GLDS_BITS "\n\ts_mov_b32 m0, %0"
                 : "=&s"(keep) : "v"(voff), "s"(sbase), "s"(ldsaddr) : "memory");
}
__device__ __forceinline__ void glds4_sys(unsigned voff, const char* sbase, unsigned ldsaddr) {
    unsigned keep;
    asm volatile("s_mov_b32 %0, m0\n\ts_mov_b32 m0, %3\n\ts_nop 0\n\t"
                 "global_load_lds_dword %1, %2 " POLL_BITS "\n\ts_mov_b32 m0, %0"
                 : "=&s"(keep) : "v"(voff), "s"(sbase), "s"(ldsaddr) : "memory");
}
#define LIKELY(x) __builtin_expect(!!(x), 1)
#define UNLIKELY(x) __builtin_expect(!!(x), 0)
#define WG_BARRIER_NOMEM() do { asm volatile("" ::: "memory"); __builtin_amdgcn_s_barrier(); asm volatile("" ::: "memory"); } while (0)
#define WG_BARRIER() do { asm volatile("" ::: "memory"); __builtin_amdgcn_s_waitcnt(0xC07F); __builtin_amdgcn_s_barrier(); asm volatile("" ::: "memory"); } while (0)

struct RawW { unsigned d[16]; };
struct RawX { u32x4 q[4]; };

__device__ __forceinline__ const char* uni(const char* p) {
    const uint64_t v = (uint64_t)p;
    const unsigned lo = __builtin_amdgcn_readfirstlane((unsigned)v), hi = __builtin_amdgcn_readfirstlane((unsigned)(v >> 32));
    return (const char*)(((uint64_t)hi << 32) | lo);
}
__device__ __forceinline__ void raw_issue(RawW& r, const char* src, int v, int s, int lane) {
    const unsigned off = (unsigned)(64 * s + 16 * (lane >> 4)) * NN + (unsigned)(v - 64) * 64u + 4u * (lane & 15);
#pragma unroll
    for (int j = 0; j < 16; ++j)
        asm volatile("global_load_dword %0, %1, %2" : "=v"(r.d[j]) : "v"(off + (unsigned)j * NN), "s"(src) : "memory");
}
__device__ __forceinline__ void raw_issue(RawX& r, const char* src, int v, int s, int lane) {
    const unsigned off = (unsigned)(64 * v + (lane & 15)) * KSH + 64u * s + 16u * (lane >> 4);
#pragma unroll
    for (int f = 0; f < 4; ++f)
        asm volatile("global_load_dwordx4 %0, %1, %2" : "=v"(r.q[f]) : "v"(off + (unsigned)f * 16u * KSH), "s"(src) : "memory");
}
__device__ __forceinline__ void raw_pin(RawW& r) {
#pragma unroll
    for (int j = 0; j < 16; ++j) asm volatile("" : "+v"(r.d[j]));
}
__device__ __forceinline__ void raw_pin(RawX& r) {
#pragma unroll
    for (int f = 0; f < 4; ++f) asm volatile("" : "+v"(r.q[f]));
}
__device__ __forceinline__ void raw_format(u32x4 (&F)[4], const RawW& r) {
#pragma unroll
    for (int q4 = 0; q4 < 4; ++q4) {
        const unsigned r0 = r.d[4 * q4], r1 = r.d[4 * q4 + 1], r2 = r.d[4 * q4 + 2], r3 = r.d[4 * q4 + 3];
        const unsigned t0 = __builtin_amdgcn_perm(r1, r0, 0x05010400u), t1 = __builtin_amdgcn_perm(r1, r0, 0x07030602u);
        const unsigned t2 = __builtin_amdgcn_perm(r3, r2, 0x05010400u), t3 = __builtin_amdgcn_perm(r3, r2, 0x07030602u);
        F[0][q4] = __builtin_amdgcn_perm(t2, t0, 0x05040100u);
        F[1][q4] = __builtin_amdgcn_perm(t2, t0, 0x07060302u);
        F[2][q4] = __builtin_amdgcn_perm(t3, t1, 0x05040100u);
        F[3][q4] = __builtin_amdgcn_perm(t3, t1, 0x07060302u);
    }
#pragma unroll
    for (int f = 0; f < 4; ++f) asm volatile("" : "+v"(F[f]));
}
__device__ __forceinline__ void raw_format(u32x4 (&F)[4], const RawX& r) {
#pragma unroll
    for (int f = 0; f < 4; ++f) { F[f] = r.q[f]; asm volatile("" : "+v"(F[f])); }
}
__device__ __forceinline__ unsigned unit_doff(int rank, int s, int v, unsigned seq) {
    return DATA_OFF + (seq & 1u) * SLOT_BYTES + (unsigned)(rank * 8 + s) * STEP_BYTES
         + (unsigned)(v >= 64 ? XFR + 4 * (v - 64) : 4 * v) * 1024u;
}
__device__ __forceinline__ unsigned unit_foff(int rank, int s, int v) {
    return FLAG_OFF + (unsigned)((rank * 8 + s) * 128 + v) * FLAG_STRIDE;
}

template <int J> struct IC { static constexpr int value = J; };
template <bool B> struct BC { static constexpr bool value = B; };
template <int J, int N, class Fn> __device__ __forceinline__ void static_for(Fn&& fn) {
    if constexpr (J < N) { fn(IC<J>{}); static_for<J + 1, N>(fn); }
}

typedef __attribute__((address_space(3))) int lds_int;

__device__ __forceinline__ unsigned next_seq(const Args& a, int b, int wave, int lane) {
    unsigned old = 0;
    if (lane == 0)
        old = __hip_atomic_fetch_add((unsigned*)(a.ws[7] + CNT_OFF) + b * 8 + wave, 1u, __ATOMIC_RELAXED, __HIP_MEMORY_SCOPE_AGENT);
    return __builtin_amdgcn_readfirstlane(old) + 1u;
}

__device__ __forceinline__ void comm_wave(const Args& a, int gw, int lane, int b, int wave, volatile lds_int* done) {
    const char* base[7];
#pragma unroll
    for (int p = 0; p < 7; ++p) base[p] = uni(a.ws[p]);
    const char* wsrc = uni(a.w);
    const char* xsrc = uni(a.x);
    const int rank = __builtin_amdgcn_readfirstlane(a.rank);
    RawW rw; RawX rx;
    int s, v;
    item_of(0, gw, rank, s, v);
    raw_issue(rw, wsrc, v, s, lane);
    const unsigned seq = next_seq(a, b, wave, lane);
    wait_vm<0>();
    raw_pin(rw);
    int k = 0;
    auto unit = [&](auto cw, auto nw) {
        constexpr bool cur_w = decltype(cw)::value, next_w = decltype(nw)::value;
        u32x4 F[4];
        if constexpr (cur_w) raw_format(F, rw); else raw_format(F, rx);
        const unsigned doff = unit_doff(rank, s, v, seq) + lane * 16u;
        item_of(k + 1, gw, rank, s, v);
        if constexpr (next_w) raw_issue(rw, wsrc, v, s, lane); else raw_issue(rx, xsrc, v, s, lane);
#pragma unroll
        for (int f = 0; f < 4; ++f) {
#pragma unroll
            for (int p = 0; p < 7; ++p) st16_sys(doff, F[f], base[p], f * 1024);
            if (f == LF - 1) {
                wait_vm<7 * LF + 4>();
                if (lane == 0) done[0] = k;
            } else if (f >= LF) wait_vm<7 * LF>();
        }
        wait_vm<28>();
        if constexpr (next_w) raw_pin(rw); else raw_pin(rx);
        ++k;
    };
    typedef BC<true> W_;
    typedef BC<false> X_;
    for (int c = 0; c < 8 - NDEL; ++c) {
        unit(W_{}, W_{}); unit(W_{}, W_{}); unit(W_{}, W_{}); unit(W_{}, X_{});
        unit(X_{}, X_{}); unit(X_{}, X_{}); unit(X_{}, X_{}); unit(X_{}, W_{});
    }
    for (int c = 0; c < NDEL; ++c) {
        unit(W_{}, W_{}); unit(W_{}, X_{});
        unit(X_{}, X_{}); unit(X_{}, X_{}); unit(X_{}, X_{}); unit(X_{}, W_{});
    }
    for (int c = 0; c < 2 * NDEL; ++c) unit(W_{}, W_{});
    wait_vm<0>();
    if (lane == 0) done[0] = 64;
}

__device__ __forceinline__ void flag_wave(const Args& a, int gw0, int lane, unsigned seq, volatile lds_int* done) {
    uint64_t fl = (uint64_t)a.ws[lane < 7 ? lane : 0];
    asm volatile("" : "+v"(fl));
    constexpr int NU = 64;
    int sent[CW];
#pragma unroll
    for (int w = 0; w < CW; ++w) sent[w] = 0;
    const unsigned long long t0 = __builtin_amdgcn_s_memrealtime();
    while (true) {
        bool all = true;
#pragma unroll
        for (int w = 0; w < CW; ++w) {
            const int d = __builtin_amdgcn_readfirstlane(done[w * 16]);
            while (sent[w] < d) {
                const int k = sent[w]++;
                int s, v;
                item_of(k, gw0 + w, a.rank, s, v);
                if (lane < 7) st4_sys(fl + unit_foff(a.rank, s, v), seq);
            }
            all = all && sent[w] == NU;
        }
        if (all || __builtin_amdgcn_s_memrealtime() - t0 > DEADLINE_TICKS) break;
        __builtin_amdgcn_s_sleep(1);
    }
}

__device__ __forceinline__ void gemm_wg(const Args& a, char* smem, int gidx, int wave, int lane, int b) {
    unsigned seq;
    const int xcd = gidx & 7, idx = gidx >> 3;
    const int I = idx % NBI, J = xcd * 2 + idx / NBI;
    const int wr = (wave >> 1) & 1, wc = (wave & 1) + ((wave >> 2) << 1);
    char* myws = (char*)uni(a.ws[7]);
    const int rank = __builtin_amdgcn_readfirstlane(a.rank);

    volatile lds_int* cnt_lds = (volatile lds_int*)((__attribute__((address_space(3))) char*)smem + CTRL_LDS);
    volatile lds_int* pb_lds = (volatile lds_int*)((__attribute__((address_space(3))) char*)smem + PB_LDS);
    volatile lds_int* qt_lds = (volatile lds_int*)((__attribute__((address_space(3))) char*)smem + QT_LDS);
    if (threadIdx.x < QT_WORDS) {
        int e = 0;
        qt_lds[threadIdx.x] = e;
    }
    if (threadIdx.x == 0) {
        cnt_lds[0] = 64 << 8;
        for (int i = 8; i < 16; ++i) cnt_lds[i] = 0;
    }

    {
        const int u = gidx * 8 + wave;
        if (u >= 1024) seq = next_seq(a, b, wave, lane);
        else {
            u32x4 F[4];
            const int s = u >> 7, v = u & 127;
            if (v >= 64) { RawW r; raw_issue(r, uni(a.w), v, s, lane); seq = next_seq(a, b, wave, lane); wait_vm<0>(); raw_pin(r); raw_format(F, r); }
            else         { RawX r; raw_issue(r, uni(a.x), v, s, lane); seq = next_seq(a, b, wave, lane); wait_vm<0>(); raw_pin(r); raw_format(F, r); }
            const unsigned doff = unit_doff(rank, s, v, seq) + lane * 16u;
#pragma unroll
            for (int f = 0; f < 4; ++f) st16_sys(doff, F[f], myws, f * 1024);
            wait_vm<0>();
            if (lane == 0) __hip_atomic_store((gu32*)(myws + unit_foff(rank, s, v)), seq, __ATOMIC_RELAXED, __HIP_MEMORY_SCOPE_SYSTEM);
        }
    }

    seq = __builtin_amdgcn_readfirstlane(seq);
    const char* data = myws + DATA_OFF + (size_t)(seq & 1u) * SLOT_BYTES;
    const unsigned lds0 = (unsigned)(uintptr_t)(__attribute__((address_space(3))) char*)smem;
    const bool g1 = wave >= 4;
    const bool poller = wave == 3;
    const int dv = wave < 3 ? wave : wave - 1;
    auto frag_src = [&](int f) { return data + (size_t)(f < MT ? MT * I + f : XFR + 16 * J + (f - MT)) * 1024u; };
    const char* sbase[5];
    const char* salt[5];
    unsigned ldst[5];
#pragma unroll
    for (int i = 0; i < 5; ++i) {
        int f = 5 * dv + i; if (f > STAGE_FR - 1) f = STAGE_FR - 1;
        sbase[i] = frag_src(f);
        salt[i] = frag_src(f >= MT + 8 ? f - 8 : f);
        ldst[i] = lds0 + f * 1024u;
    }
    const unsigned lane16 = lane * 16u;
    const int strm = lane / 9 < 7 ? lane / 9 : 6, el = lane - 9 * (lane / 9);
    unsigned pfoff;
    {
        const int u0 = (MT * I) >> 2;
        int u1 = (MT * I + MT - 1); if (u1 > 255) u1 = 255; u1 >>= 2;
        int fi;
        if (el < 5) { fi = u0 + el; if (fi > u1) fi = u1; }
        else fi = 64 + 4 * J + (el - 5);
        if (lane == 63) fi = u0;
        pfoff = FLAG_OFF + (unsigned)fi * FLAG_STRIDE;
    }
    volatile lds_int* st_lds = (volatile lds_int*)((__attribute__((address_space(3))) char*)smem + ST_LDS);
    if (poller) {
        for (int x = lane; x < 7 * 16; x += 64) {
            const int i = x >> 4, c = (x & 15) - (i == 0 ? 2 : 1);
            const int src8 = (int)((a.order[gidx] >> (4 * (i + 1))) & 7u) * 8;
            int e = 0;
            if (c < 0) e = (rank * 8 + (c == -1 && i == 0 ? 7 : i)) | (0x1FF << 8);
            else if (c < 8 - NDEL) e = (src8 + c) | (0x1FF << 8);
            else if (c < 8) e = (src8 + c) | (1 << 6) | (0x7F << 8);
            else if (c < 8 + NDEL) e = (src8 + c - NDEL) | (2 << 6) | (0x180 << 8);
            st_lds[x] = e;
        }
        __builtin_amdgcn_s_waitcnt(0xC07F);
    }
    int npub = 0;
    int fspec = 64;
    int pc = 16 * strm;
    unsigned pent, pnext, paddr, needm;
    unsigned long long leadm;
    auto stream_state = [&]() {
        paddr = (pent & 63u) * (128u * FLAG_STRIDE) + pfoff;
        needm = pent >> 8;
        leadm = __ballot(el == 0 && lane < 63 && pent != 0u && ((pent & 128u) == 0u || npub >= 64));
        asm volatile("" : "+v"(paddr), "+v"(needm));
    };
    if (poller) { pent = (unsigned)st_lds[pc]; pnext = (unsigned)st_lds[pc + 1]; stream_state(); }
    auto poll_issue = [&](int buf, int& snap) {
        snap = pc;
        glds4_sys(paddr, myws, lds0 + PB_LDS + (unsigned)buf * 256u);
    };
    auto poll_eval = [&](unsigned fv, int snap) {
        const unsigned long long ok = __ballot(fv >= seq && snap == pc);
        const bool rdy = ((unsigned)(ok >> (9 * strm)) & needm) == needm;
        const unsigned long long lead = __ballot(rdy) & leadm;
        if (UNLIKELY(lead != 0)) {
            const int pos = npub + (int)__builtin_amdgcn_mbcnt_hi((unsigned)(lead >> 32), __builtin_amdgcn_mbcnt_lo((unsigned)lead, 0u));
            const bool me = (lead >> lane) & 1ull;
            if (me) qt_lds[pos] = (int)(pent & 255u);
            if (fspec == 64 && __ballot(me && (pent & 192u) != 0u)) fspec = npub;
            npub = __builtin_amdgcn_readfirstlane(npub + __popcll(lead));
            fspec = __builtin_amdgcn_readfirstlane(fspec);
            if (lane == 0) cnt_lds[0] = npub | (fspec << 8);
            const bool adv = (lead >> (9 * strm)) & 1ull;
            pc += adv ? 1 : 0;
            pent = adv ? pnext : pent;
            stream_state();
            pnext = (unsigned)st_lds[pc + 1];
        }
    };
    int snap0 = 99, snap1 = 99;
    int act = 0;
    auto poll_now = [&]() {
        if (act) wait_vm<0>();
        act = 0;
        poll_issue(0, snap0);
        wait_vm<0>();
        poll_eval((unsigned)pb_lds[lane], snap0);
    };
    auto voff_from = [&](unsigned e) { return (e & 63u) * STEP_BYTES + lane16; };
    auto left_only = [&](unsigned e) { return (__builtin_amdgcn_readfirstlane(e) >> 6) == 1u; };
    auto issue = [&](int t, unsigned e) {
        const unsigned voff = voff_from(e), so = (unsigned)(t & (RING - 1)) * STAGE_BYTES;
        const bool lo = left_only(e);
#pragma unroll
        for (int i = 0; i < 5; ++i) glds16_sys(voff, lo ? salt[i] : sbase[i], ldst[i] + so);
    };
    i32x4 acc[9][4];
#pragma unroll
    for (int m = 0; m < 9; ++m)
#pragma unroll
        for (int n = 0; n < 4; ++n) acc[m][n] = (i32x4){0, 0, 0, 0};
    i32x4 Af[9], Bf[4];
    int iss = 0;
#define SB() __builtin_amdgcn_sched_barrier(0)

    auto loader_try = [&](int lim) {
        int r = __builtin_amdgcn_readfirstlane(cnt_lds[0]) & 255;
        if (r > lim) r = lim;
        const bool any = iss < r;
        for (; iss < r; ++iss) issue(iss, (unsigned)qt_lds[iss]);
        return any;
    };
    auto loader_block = [&](int u, int lim) {
        if (lim > 64) lim = 64;
        while (true) {
            loader_try(lim);
            if (iss > u) break;
            __builtin_amdgcn_s_sleep(1);
        }
        const unsigned long long t0 = __builtin_amdgcn_s_memrealtime();
        while (iss < lim && __builtin_amdgcn_s_memrealtime() - t0 < LINGER)
            if (!loader_try(lim)) __builtin_amdgcn_s_sleep(1);
    };
    auto poller_block = [&](int u, int lim) {
        gu32* tmo = (gu32*)(myws + TMO_OFF);
        unsigned long long t0 = __builtin_amdgcn_s_memrealtime();
        for (unsigned spins = 1; npub <= u; ++spins) {
            poll_now();
            if (npub > u) break;
            __builtin_amdgcn_s_sleep(PSLEEP);
            if ((spins & 15u) == 0u) {
                const unsigned tv = __hip_atomic_load(tmo, __ATOMIC_RELAXED, __HIP_MEMORY_SCOPE_AGENT);
                if (__builtin_amdgcn_readfirstlane(tv) == seq || __builtin_amdgcn_s_memrealtime() - t0 > DEADLINE_TICKS) {
                    if (lane == 0) __hip_atomic_store(tmo, seq, __ATOMIC_RELAXED, __HIP_MEMORY_SCOPE_AGENT);
                    npub = NT;
                    if (lane == 0) cnt_lds[0] = NT | (fspec << 8);
                }
            }
        }
        t0 = __builtin_amdgcn_s_memrealtime();
        while (npub < lim && npub < NT && __builtin_amdgcn_s_memrealtime() - t0 < LINGER) poll_now();
    };
    auto wait_landed = [&](int u) {
        const int ahead = iss - u - 1;
        wait_vm<10>();
        if (UNLIKELY(ahead < 2)) { wait_vm<5>(); if (ahead < 1) wait_vm<0>(); }
    };
    auto read_frags = [&](int t, auto dma, unsigned voff, bool lo) {
        const char* st = smem + (t & (RING - 1)) * STAGE_BYTES + lane16;
        const char* sa = st + wr * 9 * 1024;
        const char* sb = st + MT * 1024 + wc * 4096;
        const unsigned so = (unsigned)(iss & (RING - 1)) * STAGE_BYTES;
        constexpr bool D = decltype(dma)::value;
        auto req = [&](int i) { glds16_sys(voff, lo ? salt[i] : sbase[i], ldst[i] + so); };
        if constexpr (D) { req(0); SB(); }
#pragma unroll
        for (int n = 0; n < 4; ++n) Bf[n] = *(const i32x4*)(sb + n * 1024);
        if constexpr (D) { SB(); req(1); SB(); }
#pragma unroll
        for (int m = 0; m < 3; ++m) Af[m] = *(const i32x4*)(sa + m * 1024);
        if constexpr (D) { SB(); req(2); SB(); }
#pragma unroll
        for (int m = 3; m < 6; ++m) Af[m] = *(const i32x4*)(sa + m * 1024);
        if constexpr (D) { SB(); req(3); SB(); }
#pragma unroll
        for (int m = 6; m < 9; ++m) Af[m] = *(const i32x4*)(sa + m * 1024);
        if constexpr (D) { SB(); req(4); }
        SB();
    };
    auto mma_rows = [&](int m0, int m1) {
#pragma unroll
        for (int m = m0; m < m1; ++m)
#pragma unroll
            for (int n = 0; n < 4; ++n) {
                acc[m][n] = __builtin_amdgcn_mfma_i32_16x16x64_i8(Af[m], Bf[n], acc[m][n], 0, 0, 0);
            }
    };
    auto mma_phase = [&](auto&& h1, auto&& h2, auto&& h3) {
        __builtin_amdgcn_s_setprio(1);
        mma_rows(0, 3);
        SB();
        h1();
        SB();
        mma_rows(3, 4);
        SB();
        h2();
        SB();
        mma_rows(4, 5);
        SB();
        h3();
        SB();
        mma_rows(5, 9);
        __builtin_amdgcn_s_setprio(0);
    };
#define STAMP0()
#define STAMP(i)
#define DSTAMP(i)
    typedef BC<true> True_;
    typedef BC<false> False_;
    WG_BARRIER();
    if (poller) {
        const char* xsrc2 = frag_src(0);
        const char* wsrc2 = frag_src(MT + 8);
        int iss2 = 64, posted = 64;
        auto issue2 = [&]() {
            const unsigned voff2 = voff_from((unsigned)qt_lds[iss2]);
            const unsigned l2 = __builtin_amdgcn_readfirstlane(lds0 + (unsigned)(iss2 & (RING - 1)) * STAGE_BYTES);
#pragma unroll
            for (int i = 0; i < 8; ++i) glds16_sys(voff2 + i * 1024u, wsrc2, l2 + (MT + 8 + i) * 1024u);
#pragma unroll
            for (int i = 0; i < MT; ++i) glds16_sys(voff2 + i * 1024u, xsrc2, l2 + i * 1024u);
            ++iss2;
        };
        poller_block(0, RING);
        WG_BARRIER();
        STAMP0();
        bool inflight = false;
#pragma nounroll
        for (int t = 0; t < 64; ++t) {
            DSTAMP(t);
            npub = __builtin_amdgcn_readfirstlane(npub); fspec = __builtin_amdgcn_readfirstlane(fspec);
            iss2 = __builtin_amdgcn_readfirstlane(iss2);
            unsigned fv = 0;
            if (LIKELY(inflight)) { wait_vm<0>(); fv = (unsigned)pb_lds[lane]; }
            read_frags(t, False_{}, 0u, false);
            __builtin_amdgcn_s_setprio(2);
            bool adv = false;
            if (LIKELY(inflight)) {
                const unsigned long long ok = __ballot(fv >= seq && snap0 == pc);
                const bool rdy = ((unsigned)(ok >> (9 * strm)) & needm) == needm;
                const unsigned long long lead = __ballot(rdy) & leadm;
                if (lead != 0) {
                    const int pos = npub + (int)__builtin_amdgcn_mbcnt_hi((unsigned)(lead >> 32), __builtin_amdgcn_mbcnt_lo((unsigned)lead, 0u));
                    const bool me = (lead >> lane) & 1ull;
                    if (me) qt_lds[pos] = (int)(pent & 255u);
                    if (fspec == 64 && __ballot(me && (pent & 192u) != 0u)) fspec = npub;
                    npub = __builtin_amdgcn_readfirstlane(npub + __popcll(lead));
                    fspec = __builtin_amdgcn_readfirstlane(fspec);
                    if (lane == 0) cnt_lds[0] = npub | (fspec << 8);
                    adv = (lead >> (9 * strm)) & 1ull;
                }
            }
            __builtin_amdgcn_s_setprio(0);
            if (UNLIKELY(t + RING > 64)) { while (iss2 < npub && iss2 < t + RING) issue2(); }
            STAMP(0);
            WG_BARRIER();
            STAMP(1);
            SB();
            mma_phase([&]() {
                          pc += adv ? 1 : 0;
                          pent = adv ? pnext : pent;
                          paddr = (pent & 63u) * (128u * FLAG_STRIDE) + pfoff;
                          needm = pent >> 8;
                          asm volatile("" : "+v"(paddr), "+v"(needm));
                      },
                      [&]() {
                          leadm = __ballot(el == 0 && lane < 63 && pent != 0u && ((pent & 128u) == 0u || npub >= 64));
                          pnext = (unsigned)st_lds[pc + 1];
                      },
                      [&]() {
                          inflight = npub <= t + 2 + RING && npub < NT && iss2 == 64;
                          if (inflight) poll_issue(0, snap0);
                      });
            SB();
            STAMP(2);
            if (UNLIKELY(npub <= t + 1 && t < 63)) { poller_block(t + 1, t + RING); inflight = false; }
            STAMP(3);
            WG_BARRIER();
            STAMP(4);
        }
        act = inflight ? 1 : 0;
        DSTAMP(64);
        while (posted < NT) {
            if (iss2 == npub && npub < NT) {
                if (posted == iss2) poller_block(npub, npub + 1);
                else poll_now();
            }
            const int cons = cnt_lds[12 + (lane & 3)];
            while (iss2 < npub && __all(iss2 - RING - 63 <= cons)) issue2();
            if (posted < iss2) {
                const int younger = iss2 - posted - 1;
                if (younger >= 2) wait_vm<52>(); else if (younger == 1) wait_vm<26>(); else wait_vm<0>();
                ++posted;
                if (lane == 0) cnt_lds[8] = posted - 64;
            } else __builtin_amdgcn_s_sleep(1);
        }
    } else if (!g1) {
        loader_block(0, RING);
        wait_landed(0);
        int lim = smin3(RING, __builtin_amdgcn_readfirstlane(cnt_lds[0]) & 255, 64);
        unsigned voff = voff_from((unsigned)qt_lds[iss]);
        WG_BARRIER();
        STAMP0();
        for (int t = 0; t < 64; ++t) {
            if (LIKELY(iss < lim)) { read_frags(t, True_{}, voff, false); ++iss; } else read_frags(t, False_{}, voff, false);
            STAMP(0);
            WG_BARRIER();
            STAMP(1);
            const int cv = cnt_lds[0];
            const unsigned qe = (unsigned)qt_lds[iss];
            SB();
            int cs;
            mma_phase([&]() { cs = __builtin_amdgcn_readfirstlane(cv); },
                      [&]() { lim = smin3(t + 1 + RING, cs & 255, 64); },
                      [&]() { voff = voff_from(qe); asm volatile("" : "+v"(voff)); });
            SB();
            STAMP(2);
            if (LIKELY(t < 63)) {
                if (UNLIKELY(iss <= t + 1)) {
                    loader_block(t + 1, t + RING);
                    lim = smin3(t + 1 + RING, __builtin_amdgcn_readfirstlane(cnt_lds[0]) & 255, 64);
                    voff = voff_from((unsigned)qt_lds[iss]);
                }
                wait_landed(t + 1);
            }
            STAMP(3);
            WG_BARRIER();
            STAMP(4);
        }
    } else {
        loader_block(0, RING);
        wait_landed(0);
        WG_BARRIER();
        int cw = __builtin_amdgcn_readfirstlane(cnt_lds[0]);
        int lim = smin3(RING, cw & 255, 64);
        unsigned voff = voff_from((unsigned)qt_lds[iss]);
        WG_BARRIER();
        STAMP0();
        int t = 0;
        for (; t + 1 + RING <= (cw >> 8) && t + RING < 64; ++t) {
            if (LIKELY(iss < lim)) { read_frags(t, True_{}, voff, false); ++iss; } else read_frags(t, False_{}, voff, false);
            STAMP(0);
            if (UNLIKELY(iss <= t + 1)) loader_block(t + 1, t + RING);
            wait_landed(t + 1);
            STAMP(3);
            WG_BARRIER();
            STAMP(1);
            const int cv = cnt_lds[0];
            const unsigned qe = (unsigned)qt_lds[iss];
            SB();
            mma_phase([&]() { cw = __builtin_amdgcn_readfirstlane(cv); },
                      [&]() { lim = smin3(t + 1 + RING, cw & 255, 64); },
                      [&]() { voff = voff_from(qe); asm volatile("" : "+v"(voff)); });
            SB();
            STAMP(2);
            WG_BARRIER();
            STAMP(4);
        }
        int cty = __builtin_amdgcn_readfirstlane(qt_lds[t]) >> 6;
        lim = smin3(t + RING, __builtin_amdgcn_readfirstlane(cnt_lds[0]) & 255, 64);
        unsigned qe = (unsigned)qt_lds[iss];
        voff = voff_from(qe);
        int ity = __builtin_amdgcn_readfirstlane(qe) >> 6;
        for (; t < 63; ++t) {
            if (LIKELY(iss < lim)) { read_frags(t, True_{}, voff, ity == 1); ++iss; } else read_frags(t, False_{}, voff, false);
            STAMP(0);
            if (UNLIKELY(iss <= t + 1)) loader_block(t + 1, t + RING);
            wait_landed(t + 1);
            STAMP(3);
            WG_BARRIER();
            STAMP(1);
            const int cv = cnt_lds[0];
            qe = (unsigned)qt_lds[iss];
            const int te = qt_lds[t + 1];
            SB();
            int cs, nty;
            auto h1 = [&]() { cs = __builtin_amdgcn_readfirstlane(cv) & 255; };
            auto h2 = [&]() { lim = smin3(t + 1 + RING, cs, 64); ity = __builtin_amdgcn_readfirstlane(qe) >> 6; };
            auto h3 = [&]() { voff = voff_from(qe); asm volatile("" : "+v"(voff)); nty = __builtin_amdgcn_readfirstlane(te) >> 6; };
            if (LIKELY(cty == 0)) mma_phase(h1, h2, h3); else { h1(); h2(); h3(); }
            cty = nty;
            SB();
            STAMP(2);
            WG_BARRIER();
            STAMP(4);
        }
        read_frags(63, False_{}, 0u, false);
        WG_BARRIER();
        SB();
        if (cty == 0) mma_phase([]() {}, []() {}, []() {});
        SB();
        auto await = [&](int k, int seen) {
            while (seen < k - 63) { __builtin_amdgcn_s_sleep(0); seen = __builtin_amdgcn_readfirstlane(cnt_lds[8]); }
            asm volatile("" ::: "memory");
        };
        await(64, 0);
        read_frags(64, False_{}, 0u, false);
#pragma nounroll
        for (int u = 64; u < NT; ++u) {
            if (wave == 4) DSTAMP(65 + (u - 64));
            __builtin_amdgcn_s_waitcnt(0xC07F);
            asm volatile("" ::: "memory");
            if (lane == 0) cnt_lds[8 + wave] = u - 63;
            const int av = cnt_lds[8];
            SB();
            __builtin_amdgcn_s_setprio(1);
            mma_rows(0, 2);
            SB();
            if (u + 1 < NT) await(u + 1, __builtin_amdgcn_readfirstlane(av));
            const char* st = smem + ((u + 1) & (RING - 1)) * STAGE_BYTES + lane16;
            const char* sa = st + wr * 9 * 1024;
            const char* sb = st + MT * 1024 + wc * 4096;
            i32x4 Bn[4];
#pragma unroll
            for (int n = 0; n < 4; ++n) Bn[n] = *(const i32x4*)(sb + n * 1024);
            Af[0] = *(const i32x4*)(sa);
            Af[1] = *(const i32x4*)(sa + 1024);
            SB();
#pragma unroll
            for (int m = 2; m < 9; ++m) {
                mma_rows(m, m + 1);
                SB();
                Af[m] = *(const i32x4*)(sa + m * 1024);
                SB();
            }
            __builtin_amdgcn_s_setprio(0);
#pragma unroll
            for (int n = 0; n < 4; ++n) Bf[n] = Bn[n];
            SB();
        }
    }

    const unsigned tv = __hip_atomic_load((gu32*)(myws + TMO_OFF), __ATOMIC_RELAXED, __HIP_MEMORY_SCOPE_AGENT);
    const bool bad = __builtin_amdgcn_readfirstlane(tv) == seq;
    const float scale = a.sx[0] * a.sw[0];
    float* outp = a.out + (size_t)(I * (MT * 16) + wr * 144 + (lane >> 4) * 4) * NN + J * 256 + wc * 64 + (lane & 15) * 4;
    auto put = [&](int m, int r, f32x4 o) {
        float* p = outp + (size_t)(m * 16 + r) * NN;
        asm volatile("global_store_dwordx4 %0, %1, off " EPI_BITS "\n\ts_nop 1" :: "v"(p), "v"(o) : "memory");
    };
    if (UNLIKELY(bad)) {
        const float qn = __builtin_nanf("");
        for (int m = 0; m < 9; ++m)
            if (MT * I + wr * 9 + m < 256)
                for (int r = 0; r < 4; ++r) put(m, r, (f32x4){qn, qn, qn, qn});
        return;
    }
    typedef float f32x2 __attribute__((ext_vector_type(2)));
    const float nl2e = -1.4426950408889634f, tscale = scale * nl2e, tmax = 60.0f * 1.4426950408889634f;
#pragma unroll
    for (int m = 0; m < 9; ++m) {
        if (MT * I + wr * 9 + m < 256) {
            f32x2 y[4][2], d[4][2];
#pragma unroll
            for (int r = 0; r < 4; ++r)
#pragma unroll
                for (int h = 0; h < 2; ++h) {
                    const f32x2 v = {(float)acc[m][2 * h][r], (float)acc[m][2 * h + 1][r]};
                    y[r][h] = v * scale;
                    d[r][h] = v * tscale;
                }
#pragma unroll
            for (int r = 0; r < 4; ++r)
#pragma unroll
                for (int h = 0; h < 2; ++h) {
                    d[r][h].x = __builtin_amdgcn_exp2f(__builtin_amdgcn_fmed3f(d[r][h].x, -tmax, tmax));
                    d[r][h].y = __builtin_amdgcn_exp2f(__builtin_amdgcn_fmed3f(d[r][h].y, -tmax, tmax));
                }
#pragma unroll
            for (int r = 0; r < 4; ++r)
#pragma unroll
                for (int h = 0; h < 2; ++h) {
                    d[r][h] = d[r][h] + 1.0f;
                    d[r][h].x = __builtin_amdgcn_rcpf(d[r][h].x);
                    d[r][h].y = __builtin_amdgcn_rcpf(d[r][h].y);
                }
#pragma unroll
            for (int r = 0; r < 4; ++r) {
                const f32x2 o0 = y[r][0] * d[r][0], o1 = y[r][1] * d[r][1];
                put(m, r, (f32x4){o0.x, o0.y, o1.x, o1.y});
            }
        }
    }
}

__global__ void __launch_bounds__(THREADS, 2) fused_gemm_ag(Args a) {
    extern __shared__ __attribute__((aligned(16))) char smem[];
    const int b = blockIdx.x;
    const int wave = __builtin_amdgcn_readfirstlane(threadIdx.x >> 6);
    const int lane = threadIdx.x & 63;
    if (b < NCOMM && wave >= CW && wave != 7) return;
    if (b < NCOMM) {
        volatile lds_int* done = (volatile lds_int*)((__attribute__((address_space(3))) char*)smem);
        if (wave == 7) {
            if (lane < CW) done[lane * 16] = 0;
            WG_BARRIER();
            flag_wave(a, b * CW, lane, next_seq(a, b, wave, lane), done);
        } else {
            WG_BARRIER_NOMEM();
            comm_wave(a, b * CW + wave, lane, b, wave, done + wave * 16);
        }
    } else {
        gemm_wg(a, smem, b - NCOMM, wave, lane, b);
    }
}

}

extern "C" void kernel_launch(int rank, int nranks, ncclComm_t comm, hipStream_t stream,
                              void* const* d_in, const int* in_sizes, int n_in,
                              void* d_out, int out_size,
                              const int* in_dtypes, int out_dtype,
                              void* const* d_out_all,
                              void* const* d_ws_all, size_t ws_bytes) {
    (void)comm; (void)in_sizes; (void)n_in; (void)in_dtypes; (void)out_dtype; (void)d_out_all;
    if (nranks != NR || (unsigned)rank >= (unsigned)NR || ws_bytes < WS_NEEDED || out_size != MM * NN) {
        fprintf(stderr, "kernel_launch: unexpected configuration\n");
        return;
    }
    HIP_CHECK(hipFuncSetAttribute((const void*)fused_gemm_ag, hipFuncAttributeMaxDynamicSharedMemorySize, LDS_BYTES));
    Args a;
    memset(&a, 0, sizeof a);
    a.x = (const char*)d_in[0];
    a.w = (const char*)d_in[1];
    a.sx = (const float*)d_in[2];
    a.sw = (const float*)d_in[3];
    a.out = (float*)d_out;
    for (int j = 0; j < NR; ++j) a.ws[j] = (char*)d_ws_all[(rank + 1 + j) % NR];
    a.rank = rank;
    for (int g = 0; g < NGEMM; ++g) {
        const int xcd = g & 7, idx = g >> 3, I = idx % NBI, J = xcd * 2 + idx / NBI;
        const int u0 = (MT * I) >> 2;
        int u1 = MT * I + MT - 1; if (u1 > 255) u1 = 255; u1 >>= 2;
        int key[NR];
        for (int r = 0; r < NR; ++r) {
            int mx = 0;
            for (int u = u0; u <= u1; ++u) { int p = pos_of(u, r); if (p > mx) mx = p; }
            for (int u = 64 + 4 * J; u < 64 + 4 * J + 4; ++u) { int p = pos_of(u, r); if (p > mx) mx = p; }
            key[r] = mx * 8 + r;
        }
        unsigned ord = (unsigned)rank;
        int used = 1 << rank;
        for (int k = 1; k < 8; ++k) {
            int best = -1;
            for (int r = 0; r < NR; ++r) if (!((used >> r) & 1) && (best < 0 || key[r] < key[best])) best = r;
            used |= 1 << best;
            ord |= (unsigned)best << (4 * k);
        }
        a.order[g] = ord;
    }
    hipLaunchKernelGGL(fused_gemm_ag, dim3(GRID), dim3(THREADS), LDS_BYTES, stream, a);
    HIP_CHECK(hipGetLastError());
}
```

```cpp
#include <hip/hip_runtime.h>
#include <rccl/rccl.h>
#include <cstdio>
#include <cstdint>
#include <cstring>

#define HIP_CHECK(expr) do { hipError_t _e = (expr); if (_e != hipSuccess) { \
    fprintf(stderr, "HIP error %s at %s:%d\n", hipGetErrorString(_e), __FILE__, __LINE__); \
    return; } } while (0)

namespace {

typedef unsigned u32x4 __attribute__((ext_vector_type(4)));
typedef int i32x4 __attribute__((ext_vector_type(4)));
typedef float f32x4 __attribute__((ext_vector_type(4)));
typedef __attribute__((address_space(1))) unsigned gu32;

constexpr int NR = 8;
constexpr int MM = 4096, NN = 4096;
constexpr int KSH = 512;
constexpr int NCOMM = 16, NGEMM = 240, GRID = NCOMM + NGEMM, THREADS = 512;
constexpr int CW = 1;
constexpr int TW = NCOMM * CW;
#define NDEL_ 0
constexpr int NDEL = NDEL_;
#define LATEJ_ 0
constexpr int LATEJ = NDEL ? 0 : LATEJ_;
constexpr int NSEND = NDEL + LATEJ;
static_assert(NDEL >= 0 && NDEL <= 3, "");
constexpr int NT = 64 + 7 * NDEL;
constexpr int MT = 18;
constexpr int NBI = 15;
constexpr int XFR = NBI * MT;
constexpr int WFR = 256;
constexpr unsigned STEP_BYTES = (XFR + WFR) * 1024u;
constexpr unsigned SLOT_BYTES = 64u * STEP_BYTES;
constexpr unsigned CNT_OFF = 0;
constexpr unsigned TMO_OFF = 16384;
#define FSTRIDE 128
constexpr unsigned FLAG_STRIDE = FSTRIDE;
constexpr unsigned FLAG_OFF = 1u << 20;
static_assert(64 * 128 * FLAG_STRIDE <= (1u << 20), "");
constexpr unsigned DATA_OFF = 2u << 20;
constexpr size_t WS_NEEDED = (size_t)DATA_OFF + 2ull * SLOT_BYTES;
constexpr int STAGE_FR = MT + 16;
constexpr int STAGE_BYTES = STAGE_FR * 1024;
constexpr int RING = 4;
constexpr int CTRL_LDS = RING * STAGE_BYTES;
constexpr int PB_LDS = CTRL_LDS + 64;
constexpr int ST_LDS = PB_LDS + 512;
constexpr int QT_LDS = ST_LDS + 7 * 64;
constexpr int QT_WORDS = 96;
constexpr int QT_BYTES = QT_WORDS * 4;
constexpr int LDS_BYTES = QT_LDS + QT_BYTES;
constexpr unsigned long long DEADLINE_TICKS = 50000000ull;
#define LINGER 60
#define POLL_BITS "sc0 sc1"
#define EPI_BITS "sc1"
#define EPI_WIN 4
#define PDEPTH 1
#define PSLEEP 4
#define LF_ 3
constexpr int LF = LF_;

struct Args {
    const char* x;
    const char* w;
    const float* sx;
    const float* sw;
    float* out;
    char* ws[NR];
    int rank;
    int mode;
    unsigned order[NGEMM];
};

__host__ __device__ inline void tile_of(int g, int& I, int& J) {
    const int xcd = g & 7, idx = g >> 3;
    const int l = (xcd & 1) * 30 + idx;
    I = l >> 2; J = (xcd >> 1) * 4 + (l & 3);
}
__host__ __device__ inline int pos_of(int v, int r) { return v >= 64 ? ((v + 8 * r) & 63) : 64 + ((v + 8 * r) & 63); }
__host__ __device__ inline void item_of(int k, int gw, int r, int& s, int& v) {
    constexpr int N1 = 8 * (8 - NSEND), N2 = N1 + 6 * NSEND;
    if (k < N1) {
        s = k >> 3;
        const int j = k & 7, u = ((j & 3) * 16 + gw - 8 * r) & 63;
        v = j < 4 ? 64 + u : u;
    } else if (k < N2) {
        const int kk = k - N1, j = kk % 6;
        s = 8 - NSEND + kk / 6;
        if (j < 2) { const int idx = (j * 16 + gw - 4 * r) & 31; v = NDEL ? 64 + 4 * (idx >> 1) + (idx & 1) : 64 + idx; }
        else v = ((j - 2) * 16 + gw - 8 * r) & 63;
    } else {
        const int kk = (k < 63 ? k : 63) - N2, idx = ((kk & 1) * 16 + gw - 4 * r) & 31;
        s = 8 - NSEND + (kk >> 1);
        v = NDEL ? 64 + 4 * (idx >> 1) + 2 + (idx & 1) : 96 + idx;
    }
}

__device__ __forceinline__ void st16_sys(unsigned off, u32x4 d, const char* base, int imm) {
    asm volatile("global_store_dwordx4 %0, %1, %2 offset:%3 sc0 sc1\n\ts_nop 1"
                 :: "v"(off), "v"(d), "s"(base), "i"(imm) : "memory");
}
__device__ __forceinline__ void st4_sys(uint64_t addr, unsigned val) {
    asm volatile("global_store_dword %0, %1, off sc0 sc1" :: "v"(addr), "v"(val) : "memory");
}
__device__ __forceinline__ int smin3(int a, int b, int c) {
    int r;
    asm volatile("s_min_i32 %0, %1, %2\n\ts_min_i32 %0, %0, %3" : "=&s"(r) : "s"(a), "s"(b), "s"(c) : "scc");
    return r;
}
template <int N> __device__ __forceinline__ void wait_vm() {
    asm volatile("s_waitcnt vmcnt(%0)" :: "i"(N) : "memory");
}
#define GLDS_BITS " sc0 sc1"
__device__ __forceinline__ void glds16_sys(unsigned voff, const char* sbase, unsigned ldsaddr) {
    unsigned keep;
    asm volatile("s_mov_b32 %0, m0\n\ts_mov_b32 m0, %3\n\ts_nop 0\n\t"
                 "global_load_lds_dwordx4 %1, %2" GLDS_BITS "\n\ts_mov_b32 m0, %0"
                 : "=&s"(keep) : "v"(voff), "s"(sbase), "s"(ldsaddr) : "memory");
}
__device__ __forceinline__ void glds4_sys(unsigned voff, const char* sbase, unsigned ldsaddr) {
    unsigned keep;
    asm volatile("s_mov_b32 %0, m0\n\ts_mov_b32 m0, %3\n\ts_nop 0\n\t"
                 "global_load_lds_dword %1, %2 " POLL_BITS "\n\ts_mov_b32 m0, %0"
                 : "=&s"(keep) : "v"(voff), "s"(sbase), "s"(ldsaddr) : "memory");
}
#define LIKELY(x) __builtin_expect(!!(x), 1)
#define UNLIKELY(x) __builtin_expect(!!(x), 0)
#define WG_BARRIER_NOMEM() do { asm volatile("" ::: "memory"); __builtin_amdgcn_s_barrier(); asm volatile("" ::: "memory"); } while (0)
#define WG_BARRIER() do { asm volatile("" ::: "memory"); __builtin_amdgcn_s_waitcnt(0xC07F); __builtin_amdgcn_s_barrier(); asm volatile("" ::: "memory"); } while (0)

struct RawW { unsigned d[16]; };
struct RawX { u32x4 q[4]; };

__device__ __forceinline__ const char* uni(const char* p) {
    const uint64_t v = (uint64_t)p;
    const unsigned lo = __builtin_amdgcn_readfirstlane((unsigned)v), hi = __builtin_amdgcn_readfirstlane((unsigned)(v >> 32));
    return (const char*)(((uint64_t)hi << 32) | lo);
}
__device__ __forceinline__ void raw_issue(RawW& r, const char* src, int v, int s, int lane) {
    const unsigned off = (unsigned)(64 * s + 16 * (lane >> 4)) * NN + (unsigned)(v - 64) * 64u + 4u * (lane & 15);
#pragma unroll
    for (int j = 0; j < 16; ++j)
        asm volatile("global_load_dword %0, %1, %2" : "=v"(r.d[j]) : "v"(off + (unsigned)j * NN), "s"(src) : "memory");
}
__device__ __forceinline__ void raw_issue(RawX& r, const char* src, int v, int s, int lane) {
    const unsigned off = (unsigned)(64 * v + (lane & 15)) * KSH + 64u * s + 16u * (lane >> 4);
#pragma unroll
    for (int f = 0; f < 4; ++f)
        asm volatile("global_load_dwordx4 %0, %1, %2" : "=v"(r.q[f]) : "v"(off + (unsigned)f * 16u * KSH), "s"(src) : "memory");
}
__device__ __forceinline__ void raw_pin(RawW& r) {
#pragma unroll
    for (int j = 0; j < 16; ++j) asm volatile("" : "+v"(r.d[j]));
}
__device__ __forceinline__ void raw_pin(RawX& r) {
#pragma unroll
    for (int f = 0; f < 4; ++f) asm volatile("" : "+v"(r.q[f]));
}
__device__ __forceinline__ void raw_format(u32x4 (&F)[4], const RawW& r) {
#pragma unroll
    for (int q4 = 0; q4 < 4; ++q4) {
        const unsigned r0 = r.d[4 * q4], r1 = r.d[4 * q4 + 1], r2 = r.d[4 * q4 + 2], r3 = r.d[4 * q4 + 3];
        const unsigned t0 = __builtin_amdgcn_perm(r1, r0, 0x05010400u), t1 = __builtin_amdgcn_perm(r1, r0, 0x07030602u);
        const unsigned t2 = __builtin_amdgcn_perm(r3, r2, 0x05010400u), t3 = __builtin_amdgcn_perm(r3, r2, 0x07030602u);
        F[0][q4] = __builtin_amdgcn_perm(t2, t0, 0x05040100u);
        F[1][q4] = __builtin_amdgcn_perm(t2, t0, 0x07060302u);
        F[2][q4] = __builtin_amdgcn_perm(t3, t1, 0x05040100u);
        F[3][q4] = __builtin_amdgcn_perm(t3, t1, 0x07060302u);
    }
#pragma unroll
    for (int f = 0; f < 4; ++f) asm volatile("" : "+v"(F[f]));
}
__device__ __forceinline__ void raw_format(u32x4 (&F)[4], const RawX& r) {
#pragma unroll
    for (int f = 0; f < 4; ++f) { F[f] = r.q[f]; asm volatile("" : "+v"(F[f])); }
}
__device__ __forceinline__ unsigned unit_doff(int rank, int s, int v, unsigned seq) {
    return DATA_OFF + (seq & 1u) * SLOT_BYTES + (unsigned)(rank * 8 + s) * STEP_BYTES
         + (unsigned)(v >= 64 ? XFR + 4 * (v - 64) : 4 * v) * 1024u;
}
__device__ __forceinline__ unsigned unit_foff(int rank, int s, int v) {
    return FLAG_OFF + (unsigned)((rank * 8 + s) * 128 + v) * FLAG_STRIDE;
}

template <int J> struct IC { static constexpr int value = J; };
template <bool B> struct BC { static constexpr bool value = B; };
template <int J, int N, class Fn> __device__ __forceinline__ void static_for(Fn&& fn) {
    if constexpr (J < N) { fn(IC<J>{}); static_for<J + 1, N>(fn); }
}

typedef __attribute__((address_space(3))) int lds_int;

__device__ __forceinline__ unsigned next_seq(const Args& a, int b, int wave, int lane) {
    unsigned old = 0;
    if (lane == 0)
        old = __hip_atomic_fetch_add((unsigned*)(a.ws[7] + CNT_OFF) + b * 8 + wave, 1u, __ATOMIC_RELAXED, __HIP_MEMORY_SCOPE_AGENT);
    return __builtin_amdgcn_readfirstlane(old) + 1u;
}

__device__ __forceinline__ void comm_wave(const Args& a, int gw, int lane, int b, int wave, volatile lds_int* done) {
    const char* base[7];
#pragma unroll
    for (int p = 0; p < 7; ++p) base[p] = uni(a.ws[p]);
    const char* wsrc = uni(a.w);
    const char* xsrc = uni(a.x);
    const int rank = __builtin_amdgcn_readfirstlane(a.rank);
    RawW rw; RawX rx;
    int s, v;
    item_of(0, gw, rank, s, v);
    raw_issue(rw, wsrc, v, s, lane);
    const unsigned seq = next_seq(a, b, wave, lane);
    wait_vm<0>();
    raw_pin(rw);
    int k = 0;
    auto unit = [&](auto cw, auto nw) {
        constexpr bool cur_w = decltype(cw)::value, next_w = decltype(nw)::value;
        u32x4 F[4];
        if constexpr (cur_w) raw_format(F, rw); else raw_format(F, rx);
        const unsigned doff = unit_doff(rank, s, v, seq) + lane * 16u;
        item_of(k + 1, gw, rank, s, v);
        if constexpr (next_w) raw_issue(rw, wsrc, v, s, lane); else raw_issue(rx, xsrc, v, s, lane);
#pragma unroll
        for (int f = 0; f < 4; ++f) {
#pragma unroll
            for (int p = 0; p < 7; ++p) st16_sys(doff, F[f], base[p], f * 1024);
            if (f == LF - 1) {
                wait_vm<7 * LF + 4>();
                if (lane == 0) done[0] = k;
            } else if (f >= LF) wait_vm<7 * LF>();
        }
        wait_vm<28>();
        if constexpr (next_w) raw_pin(rw); else raw_pin(rx);
        ++k;
    };
    typedef BC<true> W_;
    typedef BC<false> X_;
    for (int c = 0; c < 8 - NSEND; ++c) {
        unit(W_{}, W_{}); unit(W_{}, W_{}); unit(W_{}, W_{}); unit(W_{}, X_{});
        unit(X_{}, X_{}); unit(X_{}, X_{}); unit(X_{}, X_{}); unit(X_{}, W_{});
    }
    for (int c = 0; c < NSEND; ++c) {
        unit(W_{}, W_{}); unit(W_{}, X_{});
        unit(X_{}, X_{}); unit(X_{}, X_{}); unit(X_{}, X_{}); unit(X_{}, W_{});
    }
    for (int c = 0; c < 2 * NSEND; ++c) unit(W_{}, W_{});
    wait_vm<0>();
    if (lane == 0) done[0] = 64;
}

__device__ __forceinline__ void flag_wave(const Args& a, int gw0, int lane, unsigned seq, volatile lds_int* done) {
    uint64_t fl = (uint64_t)a.ws[lane < 7 ? lane : 0];
    asm volatile("" : "+v"(fl));
    constexpr int NU = 64;
    int sent[CW];
#pragma unroll
    for (int w = 0; w < CW; ++w) sent[w] = 0;
    const unsigned long long t0 = __builtin_amdgcn_s_memrealtime();
    while (true) {
        bool all = true;
#pragma unroll
        for (int w = 0; w < CW; ++w) {
            const int d = __builtin_amdgcn_readfirstlane(done[w * 16]);
            while (sent[w] < d) {
                const int k = sent[w]++;
                int s, v;
                item_of(k, gw0 + w, a.rank, s, v);
                if (lane < 7) st4_sys(fl + unit_foff(a.rank, s, v), seq);
            }
            all = all && sent[w] == NU;
        }
        if (all || __builtin_amdgcn_s_memrealtime() - t0 > DEADLINE_TICKS) break;
        __builtin_amdgcn_s_sleep(1);
    }
}

__device__ __forceinline__ void gemm_wg(const Args& a, char* smem, int gidx, int wave, int lane, int b) {
    unsigned seq;
    int I, J;
    tile_of(gidx, I, J);
    const int wr = (wave >> 1) & 1, wc = (wave & 1) + ((wave >> 2) << 1);
    char* myws = (char*)uni(a.ws[7]);
    const int rank = __builtin_amdgcn_readfirstlane(a.rank);

    volatile lds_int* cnt_lds = (volatile lds_int*)((__attribute__((address_space(3))) char*)smem + CTRL_LDS);
    volatile lds_int* pb_lds = (volatile lds_int*)((__attribute__((address_space(3))) char*)smem + PB_LDS);
    volatile lds_int* qt_lds = (volatile lds_int*)((__attribute__((address_space(3))) char*)smem + QT_LDS);
    if (threadIdx.x < QT_WORDS) {
        int e = 0;
        qt_lds[threadIdx.x] = e;
    }
    if (threadIdx.x == 0) {
        cnt_lds[0] = 64 << 8;
        for (int i = 8; i < 16; ++i) cnt_lds[i] = 0;
    }

    {
        const int u = gidx * 8 + wave;
        if (u >= 1024) seq = next_seq(a, b, wave, lane);
        else {
            u32x4 F[4];
            const int s = u >> 7, v = u & 127;
            if (v >= 64) { RawW r; raw_issue(r, uni(a.w), v, s, lane); seq = next_seq(a, b, wave, lane); wait_vm<0>(); raw_pin(r); raw_format(F, r); }
            else         { RawX r; raw_issue(r, uni(a.x), v, s, lane); seq = next_seq(a, b, wave, lane); wait_vm<0>(); raw_pin(r); raw_format(F, r); }
            const unsigned doff = unit_doff(rank, s, v, seq) + lane * 16u;
#pragma unroll
            for (int f = 0; f < 4; ++f) st16_sys(doff, F[f], myws, f * 1024);
            wait_vm<0>();
            if (lane == 0) __hip_atomic_store((gu32*)(myws + unit_foff(rank, s, v)), seq, __ATOMIC_RELAXED, __HIP_MEMORY_SCOPE_SYSTEM);
        }
    }

    seq = __builtin_amdgcn_readfirstlane(seq);
    const char* data = myws + DATA_OFF + (size_t)(seq & 1u) * SLOT_BYTES;
    const unsigned lds0 = (unsigned)(uintptr_t)(__attribute__((address_space(3))) char*)smem;
    const bool g1 = wave >= 4;
    const bool poller = wave == 3;
    const int dv = wave < 3 ? wave : wave - 1;
    auto frag_src = [&](int f) { return data + (size_t)(f < MT ? MT * I + f : XFR + 16 * J + (f - MT)) * 1024u; };
    const char* sbase[5];
    const char* salt[5];
    unsigned ldst[5];
#pragma unroll
    for (int i = 0; i < 5; ++i) {
        int f = 5 * dv + i; if (f > STAGE_FR - 1) f = STAGE_FR - 1;
        sbase[i] = frag_src(f);
        salt[i] = frag_src(f >= MT + 8 ? f - 8 : f);
        ldst[i] = lds0 + f * 1024u;
    }
    const unsigned lane16 = lane * 16u;
    const int strm = lane / 9 < 7 ? lane / 9 : 6, el = lane - 9 * (lane / 9);
    unsigned pfoff;
    {
        const int u0 = (MT * I) >> 2;
        int u1 = (MT * I + MT - 1); if (u1 > 255) u1 = 255; u1 >>= 2;
        int fi;
        if (el < 5) { fi = u0 + el; if (fi > u1) fi = u1; }
        else fi = 64 + 4 * J + (el - 5);
        if (lane == 63) fi = u0;
        pfoff = FLAG_OFF + (unsigned)fi * FLAG_STRIDE;
    }
    volatile lds_int* st_lds = (volatile lds_int*)((__attribute__((address_space(3))) char*)smem + ST_LDS);
    if (poller) {
        for (int x = lane; x < 7 * 16; x += 64) {
            const int i = x >> 4, c = (x & 15) - (i == 0 ? 2 : 1);
            const int src8 = (int)((a.order[gidx] >> (4 * (i + 1))) & 7u) * 8;
            int e = 0;
            if (c < 0) e = (rank * 8 + (c == -1 && i == 0 ? 7 : i)) | (0x1FF << 8);
            else if (c < 8 - NDEL) e = (src8 + c) | (0x1FF << 8);
            else if (c < 8) e = (src8 + c) | (1 << 6) | (0x7F << 8);
            else if (c < 8 + NDEL) e = (src8 + c - NDEL) | (2 << 6) | (0x180 << 8);
            st_lds[x] = e;
        }
        __builtin_amdgcn_s_waitcnt(0xC07F);
    }
    int npub = 0;
    int fspec = 64;
    int pc = 16 * strm;
    unsigned pent, pnext, paddr, needm;
    unsigned long long leadm;
    auto stream_state = [&]() {
        paddr = (pent & 63u) * (128u * FLAG_STRIDE) + pfoff;
        needm = pent >> 8;
        leadm = __ballot(el == 0 && lane < 63 && pent != 0u && ((pent & 128u) == 0u || npub >= 64));
        asm volatile("" : "+v"(paddr), "+v"(needm));
    };
    if (poller) { pent = (unsigned)st_lds[pc]; pnext = (unsigned)st_lds[pc + 1]; stream_state(); }
    auto poll_issue = [&](int buf, int& snap) {
        snap = pc;
        glds4_sys(paddr, myws, lds0 + PB_LDS + (unsigned)buf * 256u);
    };
    auto poll_eval = [&](unsigned fv, int snap) {
        const unsigned long long ok = __ballot(fv >= seq && snap == pc);
        const bool rdy = ((unsigned)(ok >> (9 * strm)) & needm) == needm;
        const unsigned long long lead = __ballot(rdy) & leadm;
        if (UNLIKELY(lead != 0)) {
            const int pos = npub + (int)__builtin_amdgcn_mbcnt_hi((unsigned)(lead >> 32), __builtin_amdgcn_mbcnt_lo((unsigned)lead, 0u));
            const bool me = (lead >> lane) & 1ull;
            if (me) qt_lds[pos] = (int)(pent & 255u);
            if (fspec == 64 && __ballot(me && (pent & 192u) != 0u)) fspec = npub;
            npub = __builtin_amdgcn_readfirstlane(npub + __popcll(lead));
            fspec = __builtin_amdgcn_readfirstlane(fspec);
            if (lane == 0) cnt_lds[0] = npub | (fspec << 8);
            const bool adv = (lead >> (9 * strm)) & 1ull;
            pc += adv ? 1 : 0;
            pent = adv ? pnext : pent;
            stream_state();
            pnext = (unsigned)st_lds[pc + 1];
        }
    };
    int snap0 = 99, snap1 = 99;
    int act = 0;
    auto poll_now = [&]() {
        if (act) wait_vm<0>();
        act = 0;
        poll_issue(0, snap0);
        wait_vm<0>();
        poll_eval((unsigned)pb_lds[lane], snap0);
    };
    auto voff_from = [&](unsigned e) { return (e & 63u) * STEP_BYTES + lane16; };
    auto left_only = [&](unsigned e) { return (__builtin_amdgcn_readfirstlane(e) >> 6) == 1u; };
    auto issue = [&](int t, unsigned e) {
        const unsigned voff = voff_from(e), so = (unsigned)(t & (RING - 1)) * STAGE_BYTES;
        const bool lo = left_only(e);
#pragma unroll
        for (int i = 0; i < 5; ++i) glds16_sys(voff, lo ? salt[i] : sbase[i], ldst[i] + so);
    };
    i32x4 acc[9][4];
#pragma unroll
    for (int m = 0; m < 9; ++m)
#pragma unroll
        for (int n = 0; n < 4; ++n) acc[m][n] = (i32x4){0, 0, 0, 0};
    i32x4 Af[9], Bf[4];
    int iss = 0;
#define SB() __builtin_amdgcn_sched_barrier(0)

    auto loader_try = [&](int lim) {
        int r = __builtin_amdgcn_readfirstlane(cnt_lds[0]) & 255;
        if (r > lim) r = lim;
        const bool any = iss < r;
        for (; iss < r; ++iss) issue(iss, (unsigned)qt_lds[iss]);
        return any;
    };
    auto loader_block = [&](int u, int lim) {
        if (lim > 64) lim = 64;
        while (true) {
            loader_try(lim);
            if (iss > u) break;
            __builtin_amdgcn_s_sleep(1);
        }
        const unsigned long long t0 = __builtin_amdgcn_s_memrealtime();
        while (iss < lim && __builtin_amdgcn_s_memrealtime() - t0 < LINGER)
            if (!loader_try(lim)) __builtin_amdgcn_s_sleep(1);
    };
    auto poller_block = [&](int u, int lim) {
        gu32* tmo = (gu32*)(myws + TMO_OFF);
        unsigned long long t0 = __builtin_amdgcn_s_memrealtime();
        for (unsigned spins = 1; npub <= u; ++spins) {
            poll_now();
            if (npub > u) break;
            __builtin_amdgcn_s_sleep(PSLEEP);
            if ((spins & 15u) == 0u) {
                const unsigned tv = __hip_atomic_load(tmo, __ATOMIC_RELAXED, __HIP_MEMORY_SCOPE_AGENT);
                if (__builtin_amdgcn_readfirstlane(tv) == seq || __builtin_amdgcn_s_memrealtime() - t0 > DEADLINE_TICKS) {
                    if (lane == 0) __hip_atomic_store(tmo, seq, __ATOMIC_RELAXED, __HIP_MEMORY_SCOPE_AGENT);
                    npub = NT;
                    if (lane == 0) cnt_lds[0] = NT | (fspec << 8);
                }
            }
        }
        t0 = __builtin_amdgcn_s_memrealtime();
        while (npub < lim && npub < NT && __builtin_amdgcn_s_memrealtime() - t0 < LINGER) poll_now();
    };
    auto wait_landed = [&](int u) {
        const int ahead = iss - u - 1;
        wait_vm<10>();
        if (UNLIKELY(ahead < 2)) { wait_vm<5>(); if (ahead < 1) wait_vm<0>(); }
    };
    auto read_frags = [&](int t, auto dma, unsigned voff, bool lo) {
        const char* st = smem + (t & (RING - 1)) * STAGE_BYTES + lane16;
        const char* sa = st + wr * 9 * 1024;
        const char* sb = st + MT * 1024 + wc * 4096;
        const unsigned so = (unsigned)(iss & (RING - 1)) * STAGE_BYTES;
        constexpr bool D = decltype(dma)::value;
        auto req = [&](int i) { glds16_sys(voff, lo ? salt[i] : sbase[i], ldst[i] + so); };
        if constexpr (D) { req(0); SB(); }
#pragma unroll
        for (int n = 0; n < 4; ++n) Bf[n] = *(const i32x4*)(sb + n * 1024);
        if constexpr (D) { SB(); req(1); SB(); }
#pragma unroll
        for (int m = 0; m < 3; ++m) Af[m] = *(const i32x4*)(sa + m * 1024);
        if constexpr (D) { SB(); req(2); SB(); }
#pragma unroll
        for (int m = 3; m < 6; ++m) Af[m] = *(const i32x4*)(sa + m * 1024);
        if constexpr (D) { SB(); req(3); SB(); }
#pragma unroll
        for (int m = 6; m < 9; ++m) Af[m] = *(const i32x4*)(sa + m * 1024);
        if constexpr (D) { SB(); req(4); }
        SB();
    };
    auto mma_rows = [&](int m0, int m1) {
#pragma unroll
        for (int m = m0; m < m1; ++m)
#pragma unroll
            for (int n = 0; n < 4; ++n) {
                acc[m][n] = __builtin_amdgcn_mfma_i32_16x16x64_i8(Af[m], Bf[n], acc[m][n], 0, 0, 0);
            }
    };
    auto mma_phase = [&](auto&& h1, auto&& h2, auto&& h3) {
        __builtin_amdgcn_s_setprio(1);
        mma_rows(0, 3);
        SB();
        h1();
        SB();
        mma_rows(3, 4);
        SB();
        h2();
        SB();
        mma_rows(4, 5);
        SB();
        h3();
        SB();
        mma_rows(5, 9);
        __builtin_amdgcn_s_setprio(0);
    };
#define STAMP0()
#define STAMP(i)
#define DSTAMP(i)
    typedef BC<true> True_;
    typedef BC<false> False_;
    WG_BARRIER();
    if (poller) {
        const char* xsrc2 = frag_src(0);
        const char* wsrc2 = frag_src(MT + 8);
        int iss2 = 64, posted = 64;
        auto issue2 = [&]() {
            const unsigned voff2 = voff_from((unsigned)qt_lds[iss2]);
            const unsigned l2 = __builtin_amdgcn_readfirstlane(lds0 + (unsigned)(iss2 & (RING - 1)) * STAGE_BYTES);
#pragma unroll
            for (int i = 0; i < 8; ++i) glds16_sys(voff2 + i * 1024u, wsrc2, l2 + (MT + 8 + i) * 1024u);
#pragma unroll
            for (int i = 0; i < MT; ++i) glds16_sys(voff2 + i * 1024u, xsrc2, l2 + i * 1024u);
            ++iss2;
        };
        poller_block(0, RING);
        WG_BARRIER();
        STAMP0();
        bool inflight = false;
#pragma nounroll
        for (int t = 0; t < 64; ++t) {
            DSTAMP(t);
            npub = __builtin_amdgcn_readfirstlane(npub); fspec = __builtin_amdgcn_readfirstlane(fspec);
            iss2 = __builtin_amdgcn_readfirstlane(iss2);
            unsigned fv = 0;
            if (LIKELY(inflight)) { wait_vm<0>(); fv = (unsigned)pb_lds[lane]; }
            read_frags(t, False_{}, 0u, false);
            __builtin_amdgcn_s_setprio(2);
            bool adv = false;
            if (LIKELY(inflight)) {
                const unsigned long long ok = __ballot(fv >= seq && snap0 == pc);
                const bool rdy = ((unsigned)(ok >> (9 * strm)) & needm) == needm;
                const unsigned long long lead = __ballot(rdy) & leadm;
                if (lead != 0) {
                    const int pos = npub + (int)__builtin_amdgcn_mbcnt_hi((unsigned)(lead >> 32), __builtin_amdgcn_mbcnt_lo((unsigned)lead, 0u));
                    const bool me = (lead >> lane) & 1ull;
                    if (me) qt_lds[pos] = (int)(pent & 255u);
                    if (fspec == 64 && __ballot(me && (pent & 192u) != 0u)) fspec = npub;
                    npub = __builtin_amdgcn_readfirstlane(npub + __popcll(lead));
                    fspec = __builtin_amdgcn_readfirstlane(fspec);
                    if (lane == 0) cnt_lds[0] = npub | (fspec << 8);
                    adv = (lead >> (9 * strm)) & 1ull;
                }
            }
            __builtin_amdgcn_s_setprio(0);
            if (UNLIKELY(t + RING > 64)) { while (iss2 < npub && iss2 < t + RING) issue2(); }
            STAMP(0);
            WG_BARRIER();
            STAMP(1);
            SB();
            mma_phase([&]() {
                          pc += adv ? 1 : 0;
                          pent = adv ? pnext : pent;
                          paddr = (pent & 63u) * (128u * FLAG_STRIDE) + pfoff;
                          needm = pent >> 8;
                          asm volatile("" : "+v"(paddr), "+v"(needm));
                      },
                      [&]() {
                          leadm = __ballot(el == 0 && lane < 63 && pent != 0u && ((pent & 128u) == 0u || npub >= 64));
                          pnext = (unsigned)st_lds[pc + 1];
                      },
                      [&]() {
                          inflight = npub <= t + 2 + RING && npub < NT && iss2 == 64;
                          if (inflight) poll_issue(0, snap0);
                      });
            SB();
            STAMP(2);
            if (UNLIKELY(npub <= t + 1 && t < 63)) { poller_block(t + 1, t + RING); inflight = false; }
            STAMP(3);
            WG_BARRIER();
            STAMP(4);
        }
        act = inflight ? 1 : 0;
        DSTAMP(64);
        while (posted < NT) {
            if (iss2 == npub && npub < NT) {
                if (posted == iss2) poller_block(npub, npub + 1);
                else poll_now();
            }
            const int cons = cnt_lds[12 + (lane & 3)];
            while (iss2 < npub && __all(iss2 - RING - 63 <= cons)) issue2();
            if (posted < iss2) {
                const int younger = iss2 - posted - 1;
                if (younger >= 2) wait_vm<52>(); else if (younger == 1) wait_vm<26>(); else wait_vm<0>();
                ++posted;
                if (lane == 0) cnt_lds[8] = posted - 64;
            } else __builtin_amdgcn_s_sleep(1);
        }
    } else if (!g1) {
        loader_block(0, RING);
        wait_landed(0);
        int lim = smin3(RING, __builtin_amdgcn_readfirstlane(cnt_lds[0]) & 255, 64);
        unsigned voff = voff_from((unsigned)qt_lds[iss]);
        WG_BARRIER();
        STAMP0();
        for (int t = 0; t < 64; ++t) {
            if (LIKELY(iss < lim)) { read_frags(t, True_{}, voff, false); ++iss; } else read_frags(t, False_{}, voff, false);
            STAMP(0);
            WG_BARRIER();
            STAMP(1);
            const int cv = cnt_lds[0];
            const unsigned qe = (unsigned)qt_lds[iss];
            SB();
            int cs;
            mma_phase([&]() { cs = __builtin_amdgcn_readfirstlane(cv); },
                      [&]() { lim = smin3(t + 1 + RING, cs & 255, 64); },
                      [&]() { voff = voff_from(qe); asm volatile("" : "+v"(voff)); });
            SB();
            STAMP(2);
            if (LIKELY(t < 63)) {
                if (UNLIKELY(iss <= t + 1)) {
                    loader_block(t + 1, t + RING);
                    lim = smin3(t + 1 + RING, __builtin_amdgcn_readfirstlane(cnt_lds[0]) & 255, 64);
                    voff = voff_from((unsigned)qt_lds[iss]);
                }
                wait_landed(t + 1);
            }
            STAMP(3);
            WG_BARRIER();
            STAMP(4);
        }
    } else {
        loader_block(0, RING);
        wait_landed(0);
        WG_BARRIER();
        int cw = __builtin_amdgcn_readfirstlane(cnt_lds[0]);
        int lim = smin3(RING, cw & 255, 64);
        unsigned voff = voff_from((unsigned)qt_lds[iss]);
        WG_BARRIER();
        STAMP0();
        int t = 0;
        for (; t + 1 + RING <= (cw >> 8) && t + RING < 64; ++t) {
            if (LIKELY(iss < lim)) { read_frags(t, True_{}, voff, false); ++iss; } else read_frags(t, False_{}, voff, false);
            STAMP(0);
            if (UNLIKELY(iss <= t + 1)) loader_block(t + 1, t + RING);
            wait_landed(t + 1);
            STAMP(3);
            WG_BARRIER();
            STAMP(1);
            const int cv = cnt_lds[0];
            const unsigned qe = (unsigned)qt_lds[iss];
            SB();
            mma_phase([&]() { cw = __builtin_amdgcn_readfirstlane(cv); },
                      [&]() { lim = smin3(t + 1 + RING, cw & 255, 64); },
                      [&]() { voff = voff_from(qe); asm volatile("" : "+v"(voff)); });
            SB();
            STAMP(2);
            WG_BARRIER();
            STAMP(4);
        }
        int cty = __builtin_amdgcn_readfirstlane(qt_lds[t]) >> 6;
        lim = smin3(t + RING, __builtin_amdgcn_readfirstlane(cnt_lds[0]) & 255, 64);
        unsigned qe = (unsigned)qt_lds[iss];
        voff = voff_from(qe);
        int ity = __builtin_amdgcn_readfirstlane(qe) >> 6;
        for (; t < 63; ++t) {
            if (LIKELY(iss < lim)) { read_frags(t, True_{}, voff, ity == 1); ++iss; } else read_frags(t, False_{}, voff, false);
            STAMP(0);
            if (UNLIKELY(iss <= t + 1)) loader_block(t + 1, t + RING);
            wait_landed(t + 1);
            STAMP(3);
            WG_BARRIER();
            STAMP(1);
            const int cv = cnt_lds[0];
            qe = (unsigned)qt_lds[iss];
            const int te = qt_lds[t + 1];
            SB();
            int cs, nty;
            auto h1 = [&]() { cs = __builtin_amdgcn_readfirstlane(cv) & 255; };
            auto h2 = [&]() { lim = smin3(t + 1 + RING, cs, 64); ity = __builtin_amdgcn_readfirstlane(qe) >> 6; };
            auto h3 = [&]() { voff = voff_from(qe); asm volatile("" : "+v"(voff)); nty = __builtin_amdgcn_readfirstlane(te) >> 6; };
            if (LIKELY(cty == 0)) mma_phase(h1, h2, h3); else { h1(); h2(); h3(); }
            cty = nty;
            SB();
            STAMP(2);
            WG_BARRIER();
            STAMP(4);
        }
        read_frags(63, False_{}, 0u, false);
        WG_BARRIER();
        SB();
        if (cty == 0) mma_phase([]() {}, []() {}, []() {});
        SB();
        if constexpr (NT > 64) {
        auto await = [&](int k, int seen) {
            while (seen < k - 63) { __builtin_amdgcn_s_sleep(0); seen = __builtin_amdgcn_readfirstlane(cnt_lds[8]); }
            asm volatile("" ::: "memory");
        };
        await(64, 0);
        read_frags(64, False_{}, 0u, false);
#pragma nounroll
        for (int u = 64; u < NT; ++u) {
            if (wave == 4) DSTAMP(65 + (u - 64));
            __builtin_amdgcn_s_waitcnt(0xC07F);
            asm volatile("" ::: "memory");
            if (lane == 0) cnt_lds[8 + wave] = u - 63;
            const int av = cnt_lds[8];
            SB();
            __builtin_amdgcn_s_setprio(1);
            mma_rows(0, 2);
            SB();
            if (u + 1 < NT) await(u + 1, __builtin_amdgcn_readfirstlane(av));
            const char* st = smem + ((u + 1) & (RING - 1)) * STAGE_BYTES + lane16;
            const char* sa = st + wr * 9 * 1024;
            const char* sb = st + MT * 1024 + wc * 4096;
            i32x4 Bn[4];
#pragma unroll
            for (int n = 0; n < 4; ++n) Bn[n] = *(const i32x4*)(sb + n * 1024);
            Af[0] = *(const i32x4*)(sa);
            Af[1] = *(const i32x4*)(sa + 1024);
            SB();
#pragma unroll
            for (int m = 2; m < 9; ++m) {
                mma_rows(m, m + 1);
                SB();
                Af[m] = *(const i32x4*)(sa + m * 1024);
                SB();
            }
            __builtin_amdgcn_s_setprio(0);
#pragma unroll
            for (int n = 0; n < 4; ++n) Bf[n] = Bn[n];
            SB();
        }
            }
    }

    const unsigned tv = __hip_atomic_load((gu32*)(myws + TMO_OFF), __ATOMIC_RELAXED, __HIP_MEMORY_SCOPE_AGENT);
    const bool bad = __builtin_amdgcn_readfirstlane(tv) == seq;
    const float scale = a.sx[0] * a.sw[0];
    float* outp = a.out + (size_t)(I * (MT * 16) + wr * 144 + (lane >> 4) * 4) * NN + J * 256 + wc * 64 + (lane & 15) * 4;
    auto put = [&](int m, int r, f32x4 o) {
        float* p = outp + (size_t)(m * 16 + r) * NN;
        asm volatile("global_store_dwordx4 %0, %1, off " EPI_BITS "\n\ts_nop 1" :: "v"(p), "v"(o) : "memory");
    };
    if (UNLIKELY(bad)) {
        const float qn = __builtin_nanf("");
        for (int m = 0; m < 9; ++m)
            if (MT * I + wr * 9 + m < 256)
                for (int r = 0; r < 4; ++r) put(m, r, (f32x4){qn, qn, qn, qn});
        return;
    }
    typedef float f32x2 __attribute__((ext_vector_type(2)));
    const float nl2e = -1.4426950408889634f, tscale = scale * nl2e, tmax = 60.0f * 1.4426950408889634f;
#pragma unroll
    for (int m = 0; m < 9; ++m) {
        if (MT * I + wr * 9 + m < 256) {
            f32x2 y[4][2], d[4][2];
#pragma unroll
            for (int r = 0; r < 4; ++r)
#pragma unroll
                for (int h = 0; h < 2; ++h) {
                    const f32x2 v = {(float)acc[m][2 * h][r], (float)acc[m][2 * h + 1][r]};
                    y[r][h] = v * scale;
                    d[r][h] = v * tscale;
                }
#pragma unroll
            for (int r = 0; r < 4; ++r)
#pragma unroll
                for (int h = 0; h < 2; ++h) {
                    d[r][h].x = __builtin_amdgcn_exp2f(__builtin_amdgcn_fmed3f(d[r][h].x, -tmax, tmax));
                    d[r][h].y = __builtin_amdgcn_exp2f(__builtin_amdgcn_fmed3f(d[r][h].y, -tmax, tmax));
                }
#pragma unroll
            for (int r = 0; r < 4; ++r)
#pragma unroll
                for (int h = 0; h < 2; ++h) {
                    d[r][h] = d[r][h] + 1.0f;
                    d[r][h].x = __builtin_amdgcn_rcpf(d[r][h].x);
                    d[r][h].y = __builtin_amdgcn_rcpf(d[r][h].y);
                }
#pragma unroll
            for (int r = 0; r < 4; ++r) {
                const f32x2 o0 = y[r][0] * d[r][0], o1 = y[r][1] * d[r][1];
                put(m, r, (f32x4){o0.x, o0.y, o1.x, o1.y});
            }
        }
    }
}

__global__ void __launch_bounds__(THREADS, 2) fused_gemm_ag(Args a) {
    extern __shared__ __attribute__((aligned(16))) char smem[];
    const int b = blockIdx.x;
    const int wave = __builtin_amdgcn_readfirstlane(threadIdx.x >> 6);
    const int lane = threadIdx.x & 63;
    if (b < NCOMM && wave >= CW && wave != 7) return;
    if (b < NCOMM) {
        volatile lds_int* done = (volatile lds_int*)((__attribute__((address_space(3))) char*)smem);
        if (wave == 7) {
            if (lane < CW) done[lane * 16] = 0;
            WG_BARRIER();
            flag_wave(a, b * CW, lane, next_seq(a, b, wave, lane), done);
        } else {
            WG_BARRIER_NOMEM();
            comm_wave(a, b * CW + wave, lane, b, wave, done + wave * 16);
        }
    } else {
        gemm_wg(a, smem, b - NCOMM, wave, lane, b);
    }
}

}

extern "C" void kernel_launch(int rank, int nranks, ncclComm_t comm, hipStream_t stream,
                              void* const* d_in, const int* in_sizes, int n_in,
                              void* d_out, int out_size,
                              const int* in_dtypes, int out_dtype,
                              void* const* d_out_all,
                              void* const* d_ws_all, size_t ws_bytes) {
    (void)comm; (void)in_sizes; (void)n_in; (void)in_dtypes; (void)out_dtype; (void)d_out_all;
    if (nranks != NR || (unsigned)rank >= (unsigned)NR || ws_bytes < WS_NEEDED || out_size != MM * NN) {
        fprintf(stderr, "kernel_launch: unexpected configuration\n");
        return;
    }
    HIP_CHECK(hipFuncSetAttribute((const void*)fused_gemm_ag, hipFuncAttributeMaxDynamicSharedMemorySize, LDS_BYTES));
    Args a;
    memset(&a, 0, sizeof a);
    a.x = (const char*)d_in[0];
    a.w = (const char*)d_in[1];
    a.sx = (const float*)d_in[2];
    a.sw = (const float*)d_in[3];
    a.out = (float*)d_out;
    for (int j = 0; j < NR; ++j) a.ws[j] = (char*)d_ws_all[(rank + 1 + j) % NR];
    a.rank = rank;
    for (int g = 0; g < NGEMM; ++g) {
        int I, J;
        tile_of(g, I, J);
        const int u0 = (MT * I) >> 2;
        int u1 = MT * I + MT - 1; if (u1 > 255) u1 = 255; u1 >>= 2;
        int key[NR];
        for (int r = 0; r < NR; ++r) {
            int mx = 0;
            for (int u = u0; u <= u1; ++u) { int p = pos_of(u, r); if (p > mx) mx = p; }
            for (int u = 64 + 4 * J; u < 64 + 4 * J + 4; ++u) { int p = pos_of(u, r); if (p > mx) mx = p; }
            key[r] = mx * 8 + r;
        }
        unsigned ord = (unsigned)rank;
        int used = 1 << rank;
        for (int k = 1; k < 8; ++k) {
            int best = -1;
            for (int r = 0; r < NR; ++r) if (!((used >> r) & 1) && (best < 0 || key[r] < key[best])) best = r;
            used |= 1 << best;
            ord |= (unsigned)best << (4 * k);
        }
        a.order[g] = ord;
    }
    hipLaunchKernelGGL(fused_gemm_ag, dim3(GRID), dim3(THREADS), LDS_BYTES, stream, a);
    HIP_CHECK(hipGetLastError());
}
```

```cpp
#include <hip/hip_runtime.h>
#include <rccl/rccl.h>
#include <cstdio>
#include <cstdint>
#include <cstring>

#define HIP_CHECK(expr) do { hipError_t _e = (expr); if (_e != hipSuccess) { \
    fprintf(stderr, "HIP error %s at %s:%d\n", hipGetErrorString(_e), __FILE__, __LINE__); \
    return; } } while (0)

namespace {

typedef unsigned u32x4 __attribute__((ext_vector_type(4)));
typedef int i32x4 __attribute__((ext_vector_type(4)));
typedef float f32x4 __attribute__((ext_vector_type(4)));
typedef float f32x2 __attribute__((ext_vector_type(2)));
typedef __attribute__((address_space(1))) unsigned gu32;
typedef __attribute__((address_space(3))) int lds_int;
typedef __attribute__((address_space(3))) char lds_char;

constexpr int NR = 8;
constexpr int MM = 4096, NN = 4096;
constexpr int KSH = 512;
constexpr int NCOMM = 16, NGEMM = 240, GRID = NCOMM + NGEMM, THREADS = 512;
constexpr int TW = NCOMM;
constexpr int MT = 18;
constexpr int NBI = 15;
constexpr int XFR = NBI * MT;
constexpr int WFR = 256;
constexpr unsigned STEP_BYTES = (XFR + WFR) * 1024u;
constexpr unsigned SLOT_BYTES = 64u * STEP_BYTES;
constexpr unsigned CNT_OFF = 0;
constexpr unsigned TMO_OFF = 16384;
constexpr unsigned FLAG_STRIDE = 128;
constexpr unsigned FLAG_OFF = 1u << 20;
constexpr unsigned DATA_OFF = 2u << 20;
static_assert(64 * 128 * FLAG_STRIDE <= DATA_OFF - FLAG_OFF, "");
constexpr size_t WS_NEEDED = (size_t)DATA_OFF + 2ull * SLOT_BYTES;
constexpr int STAGE_FR = MT + 16;
constexpr int STAGE_BYTES = STAGE_FR * 1024;
constexpr int RING = 4;
constexpr int CTRL_LDS = RING * STAGE_BYTES;
constexpr int PB_LDS = CTRL_LDS + 64;
constexpr int ST_LDS = PB_LDS + 256;
constexpr int QT_LDS = ST_LDS + 7 * 64;
constexpr int QT_WORDS = 72;
constexpr int LDS_BYTES = QT_LDS + QT_WORDS * 4;
constexpr unsigned long long DEADLINE_TICKS = 50000000ull;
#define LINGER 60
#define PSLEEP 4
#define PAHEAD 5
constexpr int LF = 3;

struct Args {
    const char* x;
    const char* w;
    const float* sx;
    const float* sw;
    float* out;
    char* ws[NR];
    int rank;
    int mode;
    unsigned order[NGEMM];
};

__host__ __device__ inline void tile_of(int g, int& I, int& J) {
    const int xcd = g & 7, l = (xcd & 1) * 30 + (g >> 3);
    I = l >> 2; J = (xcd >> 1) * 4 + (l & 3);
}
__host__ __device__ inline int pos_of(int v, int r) { return v >= 64 ? ((v + 8 * r) & 63) : 64 + ((v + 8 * r) & 63); }
__host__ __device__ inline void item_of(int k, int gw, int r, int& s, int& v) {
    if (k > 63) k = 63;
    s = k >> 3;
    const int j = k & 7, u = ((j & 3) * 16 + gw - 8 * r) & 63;
    v = j < 4 ? 64 + u : u;
}
__device__ __forceinline__ unsigned unit_doff(int rank, int s, int v, unsigned seq) {
    return DATA_OFF + (seq & 1u) * SLOT_BYTES + (unsigned)(rank * 8 + s) * STEP_BYTES
         + (unsigned)(v >= 64 ? XFR + 4 * (v - 64) : 4 * v) * 1024u;
}
__device__ __forceinline__ unsigned unit_foff(int rank, int s, int v) {
    return FLAG_OFF + (unsigned)((rank * 8 + s) * 128 + v) * FLAG_STRIDE;
}

__device__ __forceinline__ void st16_sys(unsigned off, u32x4 d, const char* base, int imm) {
    asm volatile("global_store_dwordx4 %0, %1, %2 offset:%3 sc0 sc1\n\ts_nop 1"
                 :: "v"(off), "v"(d), "s"(base), "i"(imm) : "memory");
}
__device__ __forceinline__ void st4_sys(uint64_t addr, unsigned val) {
    asm volatile("global_store_dword %0, %1, off sc0 sc1" :: "v"(addr), "v"(val) : "memory");
}
__device__ __forceinline__ void glds16_sys(unsigned voff, const char* sbase, unsigned ldsaddr) {
    unsigned keep;
    asm volatile("s_mov_b32 %0, m0\n\ts_mov_b32 m0, %3\n\ts_nop 0\n\t"
                 "global_load_lds_dwordx4 %1, %2 sc0 sc1\n\ts_mov_b32 m0, %0"
                 : "=&s"(keep) : "v"(voff), "s"(sbase), "s"(ldsaddr) : "memory");
}
__device__ __forceinline__ void glds4_sys(unsigned voff, const char* sbase, unsigned ldsaddr) {
    unsigned keep;
    asm volatile("s_mov_b32 %0, m0\n\ts_mov_b32 m0, %3\n\ts_nop 0\n\t"
                 "global_load_lds_dword %1, %2 sc0 sc1\n\ts_mov_b32 m0, %0"
                 : "=&s"(keep) : "v"(voff), "s"(sbase), "s"(ldsaddr) : "memory");
}
template <int N> __device__ __forceinline__ void wait_vm() {
    asm volatile("s_waitcnt vmcnt(%0)" :: "i"(N) : "memory");
}
__device__ __forceinline__ int smin3(int a, int b, int c) {
    int r;
    asm volatile("s_min_i32 %0, %1, %2\n\ts_min_i32 %0, %0, %3" : "=&s"(r) : "s"(a), "s"(b), "s"(c) : "scc");
    return r;
}
__device__ __forceinline__ const char* uni(const char* p) {
    const uint64_t v = (uint64_t)p;
    const unsigned lo = __builtin_amdgcn_readfirstlane((unsigned)v), hi = __builtin_amdgcn_readfirstlane((unsigned)(v >> 32));
    return (const char*)(((uint64_t)hi << 32) | lo);
}
#define LIKELY(x) __builtin_expect(!!(x), 1)
#define UNLIKELY(x) __builtin_expect(!!(x), 0)
#define SB() __builtin_amdgcn_sched_barrier(0)
#define WG_BARRIER_NOMEM() do { asm volatile("" ::: "memory"); __builtin_amdgcn_s_barrier(); asm volatile("" ::: "memory"); } while (0)
#define WG_BARRIER() do { asm volatile("" ::: "memory"); __builtin_amdgcn_s_waitcnt(0xC07F); __builtin_amdgcn_s_barrier(); asm volatile("" ::: "memory"); } while (0)
template <bool B> struct BC { static constexpr bool value = B; };

struct RawW { unsigned d[16]; };
struct RawX { u32x4 q[4]; };
__device__ __forceinline__ void raw_issue(RawW& r, const char* src, int v, int s, int lane) {
    const unsigned off = (unsigned)(64 * s + 16 * (lane >> 4)) * NN + (unsigned)(v - 64) * 64u + 4u * (lane & 15);
#pragma unroll
    for (int j = 0; j < 16; ++j)
        asm volatile("global_load_dword %0, %1, %2" : "=v"(r.d[j]) : "v"(off + (unsigned)j * NN), "s"(src) : "memory");
}
__device__ __forceinline__ void raw_issue(RawX& r, const char* src, int v, int s, int lane) {
    const unsigned off = (unsigned)(64 * v + (lane & 15)) * KSH + 64u * s + 16u * (lane >> 4);
#pragma unroll
    for (int f = 0; f < 4; ++f)
        asm volatile("global_load_dwordx4 %0, %1, %2" : "=v"(r.q[f]) : "v"(off + (unsigned)f * 16u * KSH), "s"(src) : "memory");
}
__device__ __forceinline__ void raw_pin(RawW& r) {
#pragma unroll
    for (int j = 0; j < 16; ++j) asm volatile("" : "+v"(r.d[j]));
}
__device__ __forceinline__ void raw_pin(RawX& r) {
#pragma unroll
    for (int f = 0; f < 4; ++f) asm volatile("" : "+v"(r.q[f]));
}
__device__ __forceinline__ void raw_format(u32x4 (&F)[4], const RawW& r) {
#pragma unroll
    for (int q4 = 0; q4 < 4; ++q4) {
        const unsigned r0 = r.d[4 * q4], r1 = r.d[4 * q4 + 1], r2 = r.d[4 * q4 + 2], r3 = r.d[4 * q4 + 3];
        const unsigned t0 = __builtin_amdgcn_perm(r1, r0, 0x05010400u), t1 = __builtin_amdgcn_perm(r1, r0, 0x07030602u);
        const unsigned t2 = __builtin_amdgcn_perm(r3, r2, 0x05010400u), t3 = __builtin_amdgcn_perm(r3, r2, 0x07030602u);
        F[0][q4] = __builtin_amdgcn_perm(t2, t0, 0x05040100u);
        F[1][q4] = __builtin_amdgcn_perm(t2, t0, 0x07060302u);
        F[2][q4] = __builtin_amdgcn_perm(t3, t1, 0x05040100u);
        F[3][q4] = __builtin_amdgcn_perm(t3, t1, 0x07060302u);
    }
#pragma unroll
    for (int f = 0; f < 4; ++f) asm volatile("" : "+v"(F[f]));
}
__device__ __forceinline__ void raw_format(u32x4 (&F)[4], const RawX& r) {
#pragma unroll
    for (int f = 0; f < 4; ++f) { F[f] = r.q[f]; asm volatile("" : "+v"(F[f])); }
}

__device__ __forceinline__ unsigned next_seq(const Args& a, int b, int wave, int lane) {
    unsigned old = 0;
    if (lane == 0)
        old = __hip_atomic_fetch_add((unsigned*)(a.ws[7] + CNT_OFF) + b * 8 + wave, 1u, __ATOMIC_RELAXED, __HIP_MEMORY_SCOPE_AGENT);
    return __builtin_amdgcn_readfirstlane(old) + 1u;
}

__device__ __forceinline__ void comm_wave(const Args& a, int gw, int lane, volatile lds_int* done) {
    const char* base[7];
#pragma unroll
    for (int p = 0; p < 7; ++p) base[p] = uni(a.ws[p]);
    const char* wsrc = uni(a.w);
    const char* xsrc = uni(a.x);
    const int rank = __builtin_amdgcn_readfirstlane(a.rank);
    RawW rw; RawX rx;
    int s, v;
    item_of(0, gw, rank, s, v);
    raw_issue(rw, wsrc, v, s, lane);
    const unsigned seq = next_seq(a, gw, 0, lane);
    wait_vm<0>();
    raw_pin(rw);
    int k = 0;
    auto unit = [&](auto cw, auto nw) {
        constexpr bool cur_w = decltype(cw)::value, next_w = decltype(nw)::value;
        u32x4 F[4];
        if constexpr (cur_w) raw_format(F, rw); else raw_format(F, rx);
        const unsigned doff = unit_doff(rank, s, v, seq) + lane * 16u;
        item_of(k + 1, gw, rank, s, v);
        if constexpr (next_w) { if (v < 64) v = 64; raw_issue(rw, wsrc, v, s, lane); } else raw_issue(rx, xsrc, v, s, lane);
#pragma unroll
        for (int f = 0; f < 4; ++f) {
#pragma unroll
            for (int p = 0; p < 7; ++p) st16_sys(doff, F[f], base[p], f * 1024);
            if (f == LF - 1) {
                wait_vm<7 * LF + 4>();
                if (lane == 0) done[0] = k;
            } else if (f >= LF) wait_vm<7 * LF>();
        }
        wait_vm<28>();
        if constexpr (next_w) raw_pin(rw); else raw_pin(rx);
        ++k;
    };
    typedef BC<true> W_;
    typedef BC<false> X_;
    for (int c = 0; c < 8; ++c) {
        unit(W_{}, W_{}); unit(W_{}, W_{}); unit(W_{}, W_{}); unit(W_{}, X_{});
        unit(X_{}, X_{}); unit(X_{}, X_{}); unit(X_{}, X_{}); unit(X_{}, W_{});
    }
    wait_vm<0>();
    if (lane == 0) done[0] = 64;
}

__device__ __forceinline__ void flag_wave(const Args& a, int gw, int lane, unsigned seq, volatile lds_int* done) {
    uint64_t fl = (uint64_t)a.ws[lane < 7 ? lane : 0];
    asm volatile("" : "+v"(fl));
    int sent = 0;
    const unsigned long long t0 = __builtin_amdgcn_s_memrealtime();
    while (sent < 64) {
        const int d = __builtin_amdgcn_readfirstlane(done[0]);
        while (sent < d) {
            int s, v;
            item_of(sent, gw, a.rank, s, v);
            if (lane < 7) st4_sys(fl + unit_foff(a.rank, s, v), seq);
            ++sent;
        }
        if (__builtin_amdgcn_s_memrealtime() - t0 > DEADLINE_TICKS) break;
        __builtin_amdgcn_s_sleep(1);
    }
}

__device__ __forceinline__ void gemm_wg(const Args& a, char* smem, int gidx, int wave, int lane, int b) {
    unsigned seq;
    int I, J;
    tile_of(gidx, I, J);
    const int wr = (wave >> 1) & 1, wc = (wave & 1) + ((wave >> 2) << 1);
    char* myws = (char*)uni(a.ws[7]);
    const int rank = __builtin_amdgcn_readfirstlane(a.rank);

    volatile lds_int* cnt_lds = (volatile lds_int*)((lds_char*)smem + CTRL_LDS);
    volatile lds_int* pb_lds = (volatile lds_int*)((lds_char*)smem + PB_LDS);
    volatile lds_int* st_lds = (volatile lds_int*)((lds_char*)smem + ST_LDS);
    volatile lds_int* qt_lds = (volatile lds_int*)((lds_char*)smem + QT_LDS);
    if (threadIdx.x < QT_WORDS) {
        int e = 0;
        qt_lds[threadIdx.x] = e;
    }
    if (threadIdx.x == 0) {
        cnt_lds[0] = 0;
    }

    {
        const int u = gidx * 8 + wave;
        if (u >= 1024) seq = next_seq(a, b, wave, lane);
        else {
            u32x4 F[4];
            const int s = u >> 7, v = u & 127;
            if (v >= 64) { RawW r; raw_issue(r, uni(a.w), v, s, lane); seq = next_seq(a, b, wave, lane); wait_vm<0>(); raw_pin(r); raw_format(F, r); }
            else         { RawX r; raw_issue(r, uni(a.x), v, s, lane); seq = next_seq(a, b, wave, lane); wait_vm<0>(); raw_pin(r); raw_format(F, r); }
            const unsigned doff = unit_doff(rank, s, v, seq) + lane * 16u;
#pragma unroll
            for (int f = 0; f < 4; ++f) st16_sys(doff, F[f], myws, f * 1024);
            wait_vm<0>();
            if (lane == 0) __hip_atomic_store((gu32*)(myws + unit_foff(rank, s, v)), seq, __ATOMIC_RELAXED, __HIP_MEMORY_SCOPE_SYSTEM);
        }
    }

    seq = __builtin_amdgcn_readfirstlane(seq);
    const char* data = myws + DATA_OFF + (size_t)(seq & 1u) * SLOT_BYTES;
    const unsigned lds0 = (unsigned)(uintptr_t)(lds_char*)smem;
    const bool g1 = wave >= 4;
    const bool poller = wave == 3;
    const int dv = wave < 3 ? wave : wave - 1;
    const char* sbase[5];
    unsigned ldst[5];
#pragma unroll
    for (int i = 0; i < 5; ++i) {
        int f = 5 * dv + i; if (f > STAGE_FR - 1) f = STAGE_FR - 1;
        sbase[i] = data + (size_t)(f < MT ? MT * I + f : XFR + 16 * J + (f - MT)) * 1024u;
        ldst[i] = lds0 + f * 1024u;
    }
    const unsigned lane16 = lane * 16u;

    const int strm = lane / 9 < 7 ? lane / 9 : 6, el = lane - 9 * (lane / 9);
    unsigned pfoff;
    {
        const int u0 = (MT * I) >> 2;
        int u1 = (MT * I + MT - 1); if (u1 > 255) u1 = 255; u1 >>= 2;
        int fi;
        if (el < 5) { fi = u0 + el; if (fi > u1) fi = u1; }
        else fi = 64 + 4 * J + (el - 5);
        if (lane == 63) fi = u0;
        pfoff = FLAG_OFF + (unsigned)fi * FLAG_STRIDE;
    }
    if (poller) {
        for (int x = lane; x < 7 * 16; x += 64) {
            const int i = x >> 4, c = (x & 15) - (i == 0 ? 2 : 1);
            const int src8 = (int)((a.order[gidx] >> (4 * (i + 1))) & 7u) * 8;
            int e = 0;
            if (c < 0) e = (rank * 8 + (c == -1 && i == 0 ? 7 : i)) | 64;
            else if (c < 8) e = (src8 + c) | 64;
            st_lds[x] = e;
        }
        __builtin_amdgcn_s_waitcnt(0xC07F);
    }
    int npub = 0;
    int pc = 16 * strm;
    unsigned pent = 0, pnext = 0, paddr = 0;
    unsigned long long leadm = 0;
    auto stream_state = [&]() {
        paddr = (pent & 63u) * (128u * FLAG_STRIDE) + pfoff;
        leadm = __ballot(el == 0 && lane < 63 && pent != 0u);
        asm volatile("" : "+v"(paddr));
    };
    if (poller) { pent = (unsigned)st_lds[pc]; pnext = (unsigned)st_lds[pc + 1]; stream_state(); }
    int snap = -1;
    auto poll_issue = [&]() {
        snap = pc;
        glds4_sys(paddr, myws, lds0 + PB_LDS);
    };
    auto poll_eval = [&](unsigned fv) {
        const unsigned long long ok = __ballot(fv >= seq && snap == pc);
        const bool rdy = ((unsigned)(ok >> (9 * strm)) & 0x1FFu) == 0x1FFu;
        const unsigned long long lead = __ballot(rdy) & leadm;
        bool adv = false;
        if (lead != 0) {
            const int pos = npub + (int)__builtin_amdgcn_mbcnt_hi((unsigned)(lead >> 32), __builtin_amdgcn_mbcnt_lo((unsigned)lead, 0u));
            if ((lead >> lane) & 1ull) qt_lds[pos] = (int)(pent & 63u);
            npub = __builtin_amdgcn_readfirstlane(npub + __popcll(lead));
            if (lane == 0) cnt_lds[0] = npub;
            adv = (lead >> (9 * strm)) & 1ull;
        }
        return adv;
    };
    auto stream_advance = [&](bool adv) {
        pc += adv ? 1 : 0;
        pent = adv ? pnext : pent;
    };
    auto poll_now = [&]() {
        poll_issue();
        wait_vm<0>();
        stream_advance(poll_eval((unsigned)pb_lds[lane]));
        stream_state();
        pnext = (unsigned)st_lds[pc + 1];
    };
    auto voff_from = [&](unsigned q) { return q * STEP_BYTES + lane16; };
    auto issue = [&](int t, unsigned voff) {
        const unsigned so = (unsigned)(t & (RING - 1)) * STAGE_BYTES;
#pragma unroll
        for (int i = 0; i < 5; ++i) glds16_sys(voff, sbase[i], ldst[i] + so);
    };
#define DSTAMP(i)
    i32x4 acc[9][4];
#pragma unroll
    for (int m = 0; m < 9; ++m)
#pragma unroll
        for (int n = 0; n < 4; ++n) acc[m][n] = (i32x4){0, 0, 0, 0};
    i32x4 Af[9], Bf[4];
    int iss = 0;

    auto loader_try = [&](int lim) {
        int r = __builtin_amdgcn_readfirstlane(cnt_lds[0]);
        if (r > lim) r = lim;
        const bool any = iss < r;
        for (; iss < r; ++iss) issue(iss, voff_from((unsigned)qt_lds[iss]));
        return any;
    };
    auto loader_block = [&](int u, int lim) {
        if (lim > 64) lim = 64;
        while (true) {
            loader_try(lim);
            if (iss > u) break;
            __builtin_amdgcn_s_sleep(1);
        }
        const unsigned long long t0 = __builtin_amdgcn_s_memrealtime();
        while (iss < lim && __builtin_amdgcn_s_memrealtime() - t0 < LINGER)
            if (!loader_try(lim)) __builtin_amdgcn_s_sleep(1);
    };
    auto poller_block = [&](int u, int lim) {
        gu32* tmo = (gu32*)(myws + TMO_OFF);
        unsigned long long t0 = __builtin_amdgcn_s_memrealtime();
        wait_vm<0>();
        for (unsigned spins = 1; npub <= u; ++spins) {
            poll_now();
            if (npub > u) break;
            __builtin_amdgcn_s_sleep(PSLEEP);
            if ((spins & 15u) == 0u) {
                const unsigned tv = __hip_atomic_load(tmo, __ATOMIC_RELAXED, __HIP_MEMORY_SCOPE_AGENT);
                if (__builtin_amdgcn_readfirstlane(tv) == seq || __builtin_amdgcn_s_memrealtime() - t0 > DEADLINE_TICKS) {
                    if (lane == 0) __hip_atomic_store(tmo, seq, __ATOMIC_RELAXED, __HIP_MEMORY_SCOPE_AGENT);
                    npub = 64;
                    if (lane == 0) cnt_lds[0] = 64;
                }
            }
        }
        t0 = __builtin_amdgcn_s_memrealtime();
        while (npub < lim && npub < 64 && __builtin_amdgcn_s_memrealtime() - t0 < LINGER) poll_now();
    };
    auto wait_landed = [&](int u) {
        const int ahead = iss - u - 1;
        wait_vm<10>();
        if (UNLIKELY(ahead < 2)) { wait_vm<5>(); if (ahead < 1) wait_vm<0>(); }
    };
    auto read_frags = [&](int t, auto dma, unsigned voff) {
        const char* st = smem + (t & (RING - 1)) * STAGE_BYTES + lane16;
        const char* sa = st + wr * 9 * 1024;
        const char* sb = st + MT * 1024 + wc * 4096;
        const unsigned so = (unsigned)(iss & (RING - 1)) * STAGE_BYTES;
        constexpr bool D = decltype(dma)::value;
        if constexpr (D) { glds16_sys(voff, sbase[0], ldst[0] + so); SB(); }
#pragma unroll
        for (int n = 0; n < 4; ++n) Bf[n] = *(const i32x4*)(sb + n * 1024);
        if constexpr (D) { SB(); glds16_sys(voff, sbase[1], ldst[1] + so); SB(); }
#pragma unroll
        for (int m = 0; m < 3; ++m) Af[m] = *(const i32x4*)(sa + m * 1024);
        if constexpr (D) { SB(); glds16_sys(voff, sbase[2], ldst[2] + so); SB(); }
#pragma unroll
        for (int m = 3; m < 6; ++m) Af[m] = *(const i32x4*)(sa + m * 1024);
        if constexpr (D) { SB(); glds16_sys(voff, sbase[3], ldst[3] + so); SB(); }
#pragma unroll
        for (int m = 6; m < 9; ++m) Af[m] = *(const i32x4*)(sa + m * 1024);
        if constexpr (D) { SB(); glds16_sys(voff, sbase[4], ldst[4] + so); }
        SB();
    };
    auto mma_rows = [&](int m0, int m1) {
#pragma unroll
        for (int m = m0; m < m1; ++m)
#pragma unroll
            for (int n = 0; n < 4; ++n) acc[m][n] = __builtin_amdgcn_mfma_i32_16x16x64_i8(Af[m], Bf[n], acc[m][n], 0, 0, 0);
    };
    auto mma_phase = [&](auto&& h1, auto&& h2, auto&& h3) {
        __builtin_amdgcn_s_setprio(1);
        mma_rows(0, 3);
        SB();
        h1();
        SB();
        mma_rows(3, 4);
        SB();
        h2();
        SB();
        mma_rows(4, 5);
        SB();
        h3();
        SB();
        mma_rows(5, 9);
        __builtin_amdgcn_s_setprio(0);
    };
    typedef BC<true> True_;
    typedef BC<false> False_;
    WG_BARRIER();
    if (poller) {
        poller_block(0, RING);
        WG_BARRIER();
        bool inflight = false;
#pragma nounroll
        for (int t = 0; t < 64; ++t) {
            DSTAMP(t);
            npub = __builtin_amdgcn_readfirstlane(npub);
            unsigned fv = 0;
            if (LIKELY(inflight)) { wait_vm<0>(); fv = (unsigned)pb_lds[lane]; }
            read_frags(t, False_{}, 0u);
            bool adv = false;
            __builtin_amdgcn_s_setprio(2);
            if (LIKELY(inflight)) adv = poll_eval(fv);
            __builtin_amdgcn_s_setprio(0);
            WG_BARRIER();
            SB();
            mma_phase([&]() { stream_advance(adv); stream_state(); },
                      [&]() { pnext = (unsigned)st_lds[pc + 1]; },
                      [&]() {
                          inflight = npub <= t + PAHEAD && npub < 64;
                          if (inflight) poll_issue();
                      });
            SB();
            if (UNLIKELY(npub <= t + 1 && t < 63)) { poller_block(t + 1, t + RING); inflight = false; }
            WG_BARRIER();
        }
        DSTAMP(64);
    } else if (!g1) {
        loader_block(0, RING);
        wait_landed(0);
        int lim = smin3(RING, __builtin_amdgcn_readfirstlane(cnt_lds[0]), 64);
        unsigned voff = voff_from((unsigned)qt_lds[iss]);
        WG_BARRIER();
        for (int t = 0; t < 64; ++t) {
            if (LIKELY(iss < lim)) { read_frags(t, True_{}, voff); ++iss; } else read_frags(t, False_{}, voff);
            WG_BARRIER();
            const int cv = cnt_lds[0];
            const unsigned qe = (unsigned)qt_lds[iss];
            SB();
            int cs;
            mma_phase([&]() { cs = __builtin_amdgcn_readfirstlane(cv); },
                      [&]() { lim = smin3(t + 1 + RING, cs, 64); },
                      [&]() { voff = voff_from(qe); asm volatile("" : "+v"(voff)); });
            SB();
            if (LIKELY(t < 63)) {
                if (UNLIKELY(iss <= t + 1)) {
                    loader_block(t + 1, t + RING);
                    lim = smin3(t + 1 + RING, __builtin_amdgcn_readfirstlane(cnt_lds[0]), 64);
                    voff = voff_from((unsigned)qt_lds[iss]);
                }
                wait_landed(t + 1);
            }
            WG_BARRIER();
        }
    } else {
        loader_block(0, RING);
        wait_landed(0);
        WG_BARRIER();
        int lim = smin3(RING, __builtin_amdgcn_readfirstlane(cnt_lds[0]), 64);
        unsigned voff = voff_from((unsigned)qt_lds[iss]);
        WG_BARRIER();
        for (int t = 0; t < 63; ++t) {
            if (LIKELY(iss < lim)) { read_frags(t, True_{}, voff); ++iss; } else read_frags(t, False_{}, voff);
            if (UNLIKELY(iss <= t + 1)) loader_block(t + 1, t + RING);
            wait_landed(t + 1);
            WG_BARRIER();
            const int cv = cnt_lds[0];
            const unsigned qe = (unsigned)qt_lds[iss];
            SB();
            int cs;
            mma_phase([&]() { cs = __builtin_amdgcn_readfirstlane(cv); },
                      [&]() { lim = smin3(t + 1 + RING, cs, 64); },
                      [&]() { voff = voff_from(qe); asm volatile("" : "+v"(voff)); });
            SB();
            WG_BARRIER();
        }
        read_frags(63, False_{}, 0u);
        WG_BARRIER();
        SB();
        mma_phase([]() {}, []() {}, []() {});
        SB();
    }

    const unsigned tv = __hip_atomic_load((gu32*)(myws + TMO_OFF), __ATOMIC_RELAXED, __HIP_MEMORY_SCOPE_AGENT);
    const bool bad = __builtin_amdgcn_readfirstlane(tv) == seq;
    const float scale = a.sx[0] * a.sw[0];
    float* outp = a.out + (size_t)(I * (MT * 16) + wr * 144 + (lane >> 4) * 4) * NN + J * 256 + wc * 64 + (lane & 15) * 4;
    auto put = [&](int m, int r, f32x4 o) {
        asm volatile("global_store_dwordx4 %0, %1, off sc1\n\ts_nop 1" :: "v"(outp + (size_t)(m * 16 + r) * NN), "v"(o) : "memory");
    };
    if (UNLIKELY(bad)) {
        const float qn = __builtin_nanf("");
        for (int m = 0; m < 9; ++m)
            if (MT * I + wr * 9 + m < 256)
                for (int r = 0; r < 4; ++r) put(m, r, (f32x4){qn, qn, qn, qn});
        return;
    }
    const float tscale = scale * -1.4426950408889634f, tmax = 60.0f * 1.4426950408889634f;
#pragma unroll
    for (int m = 0; m < 9; ++m) {
        if (MT * I + wr * 9 + m < 256) {
            f32x2 y[4][2], d[4][2];
#pragma unroll
            for (int r = 0; r < 4; ++r)
#pragma unroll
                for (int h = 0; h < 2; ++h) {
                    const f32x2 v = {(float)acc[m][2 * h][r], (float)acc[m][2 * h + 1][r]};
                    y[r][h] = v * scale;
                    d[r][h] = v * tscale;
                }
#pragma unroll
            for (int r = 0; r < 4; ++r)
#pragma unroll
                for (int h = 0; h < 2; ++h) {
                    d[r][h].x = __builtin_amdgcn_exp2f(__builtin_amdgcn_fmed3f(d[r][h].x, -tmax, tmax));
                    d[r][h].y = __builtin_amdgcn_exp2f(__builtin_amdgcn_fmed3f(d[r][h].y, -tmax, tmax));
                }
#pragma unroll
            for (int r = 0; r < 4; ++r)
#pragma unroll
                for (int h = 0; h < 2; ++h) {
                    d[r][h] = d[r][h] + 1.0f;
                    d[r][h].x = __builtin_amdgcn_rcpf(d[r][h].x);
                    d[r][h].y = __builtin_amdgcn_rcpf(d[r][h].y);
                }
#pragma unroll
            for (int r = 0; r < 4; ++r) {
                const f32x2 o0 = y[r][0] * d[r][0], o1 = y[r][1] * d[r][1];
                put(m, r, (f32x4){o0.x, o0.y, o1.x, o1.y});
            }
        }
    }
}

__global__ void __launch_bounds__(THREADS, 2) fused_gemm_ag(Args a) {
    extern __shared__ __attribute__((aligned(16))) char smem[];
    const int b = blockIdx.x;
    const int wave = __builtin_amdgcn_readfirstlane(threadIdx.x >> 6);
    const int lane = threadIdx.x & 63;
    if (b < NCOMM && wave != 0 && wave != 7) return;
    if (b < NCOMM) {
        volatile lds_int* done = (volatile lds_int*)((lds_char*)smem);
        if (wave == 7) {
            if (lane == 0) done[0] = 0;
            WG_BARRIER();
            flag_wave(a, b, lane, next_seq(a, b, wave, lane), done);
        } else {
            WG_BARRIER_NOMEM();
            comm_wave(a, b, lane, done);
        }
    } else {
        gemm_wg(a, smem, b - NCOMM, wave, lane, b);
    }
}

}

extern "C" void kernel_launch(int rank, int nranks, ncclComm_t comm, hipStream_t stream,
                              void* const* d_in, const int* in_sizes, int n_in,
                              void* d_out, int out_size,
                              const int* in_dtypes, int out_dtype,
                              void* const* d_out_all,
                              void* const* d_ws_all, size_t ws_bytes) {
    (void)comm; (void)in_sizes; (void)n_in; (void)in_dtypes; (void)out_dtype; (void)d_out_all;
    if (nranks != NR || (unsigned)rank >= (unsigned)NR || ws_bytes < WS_NEEDED || out_size != MM * NN) {
        fprintf(stderr, "kernel_launch: unexpected configuration\n");
        return;
    }
    HIP_CHECK(hipFuncSetAttribute((const void*)fused_gemm_ag, hipFuncAttributeMaxDynamicSharedMemorySize, LDS_BYTES));
    Args a;
    memset(&a, 0, sizeof a);
    a.x = (const char*)d_in[0];
    a.w = (const char*)d_in[1];
    a.sx = (const float*)d_in[2];
    a.sw = (const float*)d_in[3];
    a.out = (float*)d_out;
    for (int j = 0; j < NR; ++j) a.ws[j] = (char*)d_ws_all[(rank + 1 + j) % NR];
    a.rank = rank;
    for (int g = 0; g < NGEMM; ++g) {
        int I, J;
        tile_of(g, I, J);
        const int u0 = (MT * I) >> 2;
        int u1 = MT * I + MT - 1; if (u1 > 255) u1 = 255; u1 >>= 2;
        int key[NR];
        for (int r = 0; r < NR; ++r) {
            int mx = 0;
            for (int u = u0; u <= u1; ++u) { int p = pos_of(u, r); if (p > mx) mx = p; }
            for (int u = 64 + 4 * J; u < 64 + 4 * J + 4; ++u) { int p = pos_of(u, r); if (p > mx) mx = p; }
            key[r] = mx * 8 + r;
        }
        unsigned ord = (unsigned)rank;
        int used = 1 << rank;
        for (int k = 1; k < 8; ++k) {
            int best = -1;
            for (int r = 0; r < NR; ++r) if (!((used >> r) & 1) && (best < 0 || key[r] < key[best])) best = r;
            used |= 1 << best;
            ord |= (unsigned)best << (4 * k);
        }
        a.order[g] = ord;
    }
    hipLaunchKernelGGL(fused_gemm_ag, dim3(GRID), dim3(THREADS), LDS_BYTES, stream, a);
    HIP_CHECK(hipGetLastError());
}
```

```cpp
#include <hip/hip_runtime.h>
#include <rccl/rccl.h>
#include <cstdio>
#include <cstdint>
#include <cstring>

#define HIP_CHECK(expr) do { hipError_t _e = (expr); if (_e != hipSuccess) { \
    fprintf(stderr, "HIP error %s at %s:%d\n", hipGetErrorString(_e), __FILE__, __LINE__); \
    return; } } while (0)

namespace {

typedef unsigned u32x4 __attribute__((ext_vector_type(4)));
typedef int i32x4 __attribute__((ext_vector_type(4)));
typedef float f32x4 __attribute__((ext_vector_type(4)));
typedef float f32x2 __attribute__((ext_vector_type(2)));
typedef __attribute__((address_space(1))) unsigned gu32;
typedef __attribute__((address_space(3))) int lds_int;
typedef __attribute__((address_space(3))) char lds_char;

constexpr int NR = 8;
constexpr int MM = 4096, NN = 4096;
constexpr int KSH = 512;
constexpr int NCOMM = 16, NGEMM = 240, GRID = NCOMM + NGEMM, THREADS = 512;
constexpr int TW = NCOMM;
constexpr int MT = 18;
constexpr int NBI = 15;
constexpr int XFR = NBI * MT;
constexpr int WFR = 256;
constexpr unsigned STEP_BYTES = (XFR + WFR) * 1024u;
constexpr unsigned SLOT_BYTES = 64u * STEP_BYTES;
constexpr unsigned CNT_OFF = 0;
constexpr unsigned TMO_OFF = 16384;
constexpr unsigned FLAG_STRIDE = 128;
constexpr unsigned FLAG_OFF = 1u << 20;
constexpr unsigned DATA_OFF = 2u << 20;
static_assert(64 * 128 * FLAG_STRIDE <= DATA_OFF - FLAG_OFF, "");
constexpr size_t WS_NEEDED = (size_t)DATA_OFF + 2ull * SLOT_BYTES;
constexpr int STAGE_FR = MT + 16;
constexpr int STAGE_BYTES = STAGE_FR * 1024;
constexpr int RING = 4;
constexpr int CTRL_LDS = RING * STAGE_BYTES;
constexpr int PB_LDS = CTRL_LDS + 64;
constexpr int ST_LDS = PB_LDS + 256;
constexpr int QT_LDS = ST_LDS + 7 * 64;
constexpr int QT_WORDS = 72;
constexpr int LDS_BYTES = QT_LDS + QT_WORDS * 4;
constexpr unsigned long long DEADLINE_TICKS = 50000000ull;
#define LINGER 60
#define PSLEEP 4
#define PAHEAD 5
constexpr int LF = 3;

struct Args {
    const char* x;
    const char* w;
    const float* sx;
    const float* sw;
    float* out;
    char* ws[NR];
    int rank;
    int mode;
    unsigned order[NGEMM];
};

__host__ __device__ inline void tile_of(int g, int& I, int& J) {
    const int xcd = g & 7, l = (xcd & 1) * 30 + (g >> 3);
    I = l >> 2; J = (xcd >> 1) * 4 + (l & 3);
}
__host__ __device__ inline int pos_of(int v, int r) { return v >= 64 ? ((v + 8 * r) & 63) : 64 + ((v + 8 * r) & 63); }
#define LATEJ 1
__host__ __device__ inline void item_of(int k, int gw, int r, int& s, int& v) {
    constexpr int N1 = 8 * (8 - LATEJ), N2 = N1 + 6 * LATEJ;
    if (k > 63) k = 63;
    if (k < N1) {
        s = k >> 3;
        const int j = k & 7, u = ((j & 3) * 16 + gw - 8 * r) & 63;
        v = j < 4 ? 64 + u : u;
    } else if (k < N2) {
        const int kk = k - N1, j = kk % 6;
        s = 8 - LATEJ + kk / 6;
        if (j < 2) { const int idx = (j * 16 + gw - 4 * r) & 31; v = 64 + ((idx & 24) << 1) + (idx & 7); }
        else v = ((j - 2) * 16 + gw - 8 * r) & 63;
    } else {
        const int kk = k - N2, idx = ((kk & 1) * 16 + gw - 4 * r) & 31;
        s = 8 - LATEJ + (kk >> 1);
        v = 64 + ((idx & 24) << 1) + 8 + (idx & 7);
    }
}
__device__ __forceinline__ unsigned unit_doff(int rank, int s, int v, unsigned seq) {
    return DATA_OFF + (seq & 1u) * SLOT_BYTES + (unsigned)(rank * 8 + s) * STEP_BYTES
         + (unsigned)(v >= 64 ? XFR + 4 * (v - 64) : 4 * v) * 1024u;
}
__device__ __forceinline__ unsigned unit_foff(int rank, int s, int v) {
    return FLAG_OFF + (unsigned)((rank * 8 + s) * 128 + v) * FLAG_STRIDE;
}

__device__ __forceinline__ void st16_sys(unsigned off, u32x4 d, const char* base, int imm) {
    asm volatile("global_store_dwordx4 %0, %1, %2 offset:%3 sc0 sc1\n\ts_nop 1"
                 :: "v"(off), "v"(d), "s"(base), "i"(imm) : "memory");
}
__device__ __forceinline__ void st4_sys(uint64_t addr, unsigned val) {
    asm volatile("global_store_dword %0, %1, off sc0 sc1" :: "v"(addr), "v"(val) : "memory");
}
__device__ __forceinline__ void glds16_sys(unsigned voff, const char* sbase, unsigned ldsaddr) {
    unsigned keep;
    asm volatile("s_mov_b32 %0, m0\n\ts_mov_b32 m0, %3\n\ts_nop 0\n\t"
                 "global_load_lds_dwordx4 %1, %2 sc0 sc1\n\ts_mov_b32 m0, %0"
                 : "=&s"(keep) : "v"(voff), "s"(sbase), "s"(ldsaddr) : "memory");
}
__device__ __forceinline__ void glds4_sys(unsigned voff, const char* sbase, unsigned ldsaddr) {
    unsigned keep;
    asm volatile("s_mov_b32 %0, m0\n\ts_mov_b32 m0, %3\n\ts_nop 0\n\t"
                 "global_load_lds_dword %1, %2 sc0 sc1\n\ts_mov_b32 m0, %0"
                 : "=&s"(keep) : "v"(voff), "s"(sbase), "s"(ldsaddr) : "memory");
}
template <int N> __device__ __forceinline__ void wait_vm() {
    asm volatile("s_waitcnt vmcnt(%0)" :: "i"(N) : "memory");
}
__device__ __forceinline__ int smin3(int a, int b, int c) {
    int r;
    asm volatile("s_min_i32 %0, %1, %2\n\ts_min_i32 %0, %0, %3" : "=&s"(r) : "s"(a), "s"(b), "s"(c) : "scc");
    return r;
}
__device__ __forceinline__ const char* uni(const char* p) {
    const uint64_t v = (uint64_t)p;
    const unsigned lo = __builtin_amdgcn_readfirstlane((unsigned)v), hi = __builtin_amdgcn_readfirstlane((unsigned)(v >> 32));
    return (const char*)(((uint64_t)hi << 32) | lo);
}
#define LIKELY(x) __builtin_expect(!!(x), 1)
#define UNLIKELY(x) __builtin_expect(!!(x), 0)
#define SB() __builtin_amdgcn_sched_barrier(0)
#define WG_BARRIER_NOMEM() do { asm volatile("" ::: "memory"); __builtin_amdgcn_s_barrier(); asm volatile("" ::: "memory"); } while (0)
#define WG_BARRIER() do { asm volatile("" ::: "memory"); __builtin_amdgcn_s_waitcnt(0xC07F); __builtin_amdgcn_s_barrier(); asm volatile("" ::: "memory"); } while (0)
template <bool B> struct BC { static constexpr bool value = B; };

struct RawW { unsigned d[16]; };
struct RawX { u32x4 q[4]; };
__device__ __forceinline__ void raw_issue(RawW& r, const char* src, int v, int s, int lane) {
    const unsigned off = (unsigned)(64 * s + 16 * (lane >> 4)) * NN + (unsigned)(v - 64) * 64u + 4u * (lane & 15);
#pragma unroll
    for (int j = 0; j < 16; ++j)
        asm volatile("global_load_dword %0, %1, %2" : "=v"(r.d[j]) : "v"(off + (unsigned)j * NN), "s"(src) : "memory");
}
__device__ __forceinline__ void raw_issue(RawX& r, const char* src, int v, int s, int lane) {
    const unsigned off = (unsigned)(64 * v + (lane & 15)) * KSH + 64u * s + 16u * (lane >> 4);
#pragma unroll
    for (int f = 0; f < 4; ++f)
        asm volatile("global_load_dwordx4 %0, %1, %2" : "=v"(r.q[f]) : "v"(off + (unsigned)f * 16u * KSH), "s"(src) : "memory");
}
__device__ __forceinline__ void raw_pin(RawW& r) {
#pragma unroll
    for (int j = 0; j < 16; ++j) asm volatile("" : "+v"(r.d[j]));
}
__device__ __forceinline__ void raw_pin(RawX& r) {
#pragma unroll
    for (int f = 0; f < 4; ++f) asm volatile("" : "+v"(r.q[f]));
}
__device__ __forceinline__ void raw_format(u32x4 (&F)[4], const RawW& r) {
#pragma unroll
    for (int q4 = 0; q4 < 4; ++q4) {
        const unsigned r0 = r.d[4 * q4], r1 = r.d[4 * q4 + 1], r2 = r.d[4 * q4 + 2], r3 = r.d[4 * q4 + 3];
        const unsigned t0 = __builtin_amdgcn_perm(r1, r0, 0x05010400u), t1 = __builtin_amdgcn_perm(r1, r0, 0x07030602u);
        const unsigned t2 = __builtin_amdgcn_perm(r3, r2, 0x05010400u), t3 = __builtin_amdgcn_perm(r3, r2, 0x07030602u);
        F[0][q4] = __builtin_amdgcn_perm(t2, t0, 0x05040100u);
        F[1][q4] = __builtin_amdgcn_perm(t2, t0, 0x07060302u);
        F[2][q4] = __builtin_amdgcn_perm(t3, t1, 0x05040100u);
        F[3][q4] = __builtin_amdgcn_perm(t3, t1, 0x07060302u);
    }
#pragma unroll
    for (int f = 0; f < 4; ++f) asm volatile("" : "+v"(F[f]));
}
__device__ __forceinline__ void raw_format(u32x4 (&F)[4], const RawX& r) {
#pragma unroll
    for (int f = 0; f < 4; ++f) { F[f] = r.q[f]; asm volatile("" : "+v"(F[f])); }
}

__device__ __forceinline__ unsigned next_seq(const Args& a, int b, int wave, int lane) {
    unsigned old = 0;
    if (lane == 0)
        old = __hip_atomic_fetch_add((unsigned*)(a.ws[7] + CNT_OFF) + b * 8 + wave, 1u, __ATOMIC_RELAXED, __HIP_MEMORY_SCOPE_AGENT);
    return __builtin_amdgcn_readfirstlane(old) + 1u;
}

__device__ __forceinline__ void comm_wave(const Args& a, int gw, int lane, volatile lds_int* done) {
    const char* base[7];
#pragma unroll
    for (int p = 0; p < 7; ++p) base[p] = uni(a.ws[p]);
    const char* wsrc = uni(a.w);
    const char* xsrc = uni(a.x);
    const int rank = __builtin_amdgcn_readfirstlane(a.rank);
    RawW rw; RawX rx;
    int s, v;
    item_of(0, gw, rank, s, v);
    raw_issue(rw, wsrc, v, s, lane);
    const unsigned seq = next_seq(a, gw, 0, lane);
    WG_BARRIER_NOMEM();
    wait_vm<0>();
    raw_pin(rw);
    int k = 0;
    auto unit = [&](auto cw, auto nw) {
        constexpr bool cur_w = decltype(cw)::value, next_w = decltype(nw)::value;
        u32x4 F[4];
        if constexpr (cur_w) raw_format(F, rw); else raw_format(F, rx);
        const unsigned doff = unit_doff(rank, s, v, seq) + lane * 16u;
        item_of(k + 1, gw, rank, s, v);
        if constexpr (next_w) { if (v < 64) v = 64; raw_issue(rw, wsrc, v, s, lane); } else raw_issue(rx, xsrc, v, s, lane);
#pragma unroll
        for (int f = 0; f < 4; ++f) {
#pragma unroll
            for (int p = 0; p < 7; ++p) st16_sys(doff, F[f], base[p], f * 1024);
            if (f == LF - 1) {
                wait_vm<7 * LF + 4>();
                if (lane == 0) done[0] = k;
            } else if (f >= LF) wait_vm<7 * LF>();
        }
        wait_vm<28>();
        if constexpr (next_w) raw_pin(rw); else raw_pin(rx);
        ++k;
    };
    typedef BC<true> W_;
    typedef BC<false> X_;
    for (int c = 0; c < 8 - LATEJ; ++c) {
        unit(W_{}, W_{}); unit(W_{}, W_{}); unit(W_{}, W_{}); unit(W_{}, X_{});
        unit(X_{}, X_{}); unit(X_{}, X_{}); unit(X_{}, X_{}); unit(X_{}, W_{});
    }
    for (int c = 0; c < LATEJ; ++c) {
        unit(W_{}, W_{}); unit(W_{}, X_{});
        unit(X_{}, X_{}); unit(X_{}, X_{}); unit(X_{}, X_{}); unit(X_{}, W_{});
    }
    for (int c = 0; c < 2 * LATEJ; ++c) unit(W_{}, W_{});
    wait_vm<0>();
    if (lane == 0) done[0] = 64;
}

__device__ __forceinline__ void flag_wave(const Args& a, int gw, int lane, unsigned seq, volatile lds_int* done) {
    uint64_t fl = (uint64_t)a.ws[lane < 7 ? lane : 0];
    asm volatile("" : "+v"(fl));
    int sent = 0;
    const unsigned long long t0 = __builtin_amdgcn_s_memrealtime();
    while (sent < 64) {
        const int d = __builtin_amdgcn_readfirstlane(done[0]);
        while (sent < d) {
            int s, v;
            item_of(sent, gw, a.rank, s, v);
            if (lane < 7) st4_sys(fl + unit_foff(a.rank, s, v), seq);
            ++sent;
        }
        if (__builtin_amdgcn_s_memrealtime() - t0 > DEADLINE_TICKS) break;
        __builtin_amdgcn_s_sleep(1);
    }
}

__device__ __forceinline__ void gemm_wg(const Args& a, char* smem, int gidx, int wave, int lane, int b) {
    unsigned seq;
    int I, J;
    tile_of(gidx, I, J);
    const int wr = (wave >> 1) & 1, wc = (wave & 1) + ((wave >> 2) << 1);
    char* myws = (char*)uni(a.ws[7]);
    const int rank = __builtin_amdgcn_readfirstlane(a.rank);

    volatile lds_int* cnt_lds = (volatile lds_int*)((lds_char*)smem + CTRL_LDS);
    volatile lds_int* pb_lds = (volatile lds_int*)((lds_char*)smem + PB_LDS);
    volatile lds_int* st_lds = (volatile lds_int*)((lds_char*)smem + ST_LDS);
    volatile lds_int* qt_lds = (volatile lds_int*)((lds_char*)smem + QT_LDS);
    if (threadIdx.x < QT_WORDS) {
        int e = 0;
        qt_lds[threadIdx.x] = e;
    }
    if (threadIdx.x == 0) {
        cnt_lds[0] = 0;
    }

    {
        const int u = gidx * 8 + wave;
        if (u >= 1024) seq = next_seq(a, b, wave, lane);
        else {
            u32x4 F[4];
            const int s = u >> 7, v = u & 127;
            if (v >= 64) { RawW r; raw_issue(r, uni(a.w), v, s, lane); seq = next_seq(a, b, wave, lane); wait_vm<0>(); raw_pin(r); raw_format(F, r); }
            else         { RawX r; raw_issue(r, uni(a.x), v, s, lane); seq = next_seq(a, b, wave, lane); wait_vm<0>(); raw_pin(r); raw_format(F, r); }
            const unsigned doff = unit_doff(rank, s, v, seq) + lane * 16u;
#pragma unroll
            for (int f = 0; f < 4; ++f) st16_sys(doff, F[f], myws, f * 1024);
            wait_vm<0>();
            if (lane == 0) __hip_atomic_store((gu32*)(myws + unit_foff(rank, s, v)), seq, __ATOMIC_RELAXED, __HIP_MEMORY_SCOPE_SYSTEM);
        }
    }

    seq = __builtin_amdgcn_readfirstlane(seq);
    const char* data = myws + DATA_OFF + (size_t)(seq & 1u) * SLOT_BYTES;
    const unsigned lds0 = (unsigned)(uintptr_t)(lds_char*)smem;
    const bool g1 = wave >= 4;
    const bool poller = wave == 3;
    const int dv = wave < 3 ? wave : wave - 1;
    const char* sbase[5];
    unsigned ldst[5];
#pragma unroll
    for (int i = 0; i < 5; ++i) {
        int f = 5 * dv + i; if (f > STAGE_FR - 1) f = STAGE_FR - 1;
        sbase[i] = data + (size_t)(f < MT ? MT * I + f : XFR + 16 * J + (f - MT)) * 1024u;
        ldst[i] = lds0 + f * 1024u;
    }
    const unsigned lane16 = lane * 16u;

    const int strm = lane / 9 < 7 ? lane / 9 : 6, el = lane - 9 * (lane / 9);
    unsigned pfoff;
    {
        const int u0 = (MT * I) >> 2;
        int u1 = (MT * I + MT - 1); if (u1 > 255) u1 = 255; u1 >>= 2;
        int fi;
        if (el < 5) { fi = u0 + el; if (fi > u1) fi = u1; }
        else fi = 64 + 4 * J + (el - 5);
        if (lane == 63) fi = u0;
        pfoff = FLAG_OFF + (unsigned)fi * FLAG_STRIDE;
    }
    if (poller) {
        for (int x = lane; x < 7 * 16; x += 64) {
            const int i = x >> 4, c = (x & 15) - (i == 0 ? 2 : 1);
            const int src8 = (int)((a.order[gidx] >> (4 * (i + 1))) & 7u) * 8;
            int e = 0;
            if (c < 0) e = (rank * 8 + (c == -1 && i == 0 ? 7 : i)) | 64;
            else if (c < 8) e = (src8 + c) | 64;
            st_lds[x] = e;
        }
        __builtin_amdgcn_s_waitcnt(0xC07F);
    }
    int npub = 0;
    int pc = 16 * strm;
    unsigned pent = 0, pnext = 0, paddr = 0;
    unsigned long long leadm = 0;
    auto stream_state = [&]() {
        paddr = (pent & 63u) * (128u * FLAG_STRIDE) + pfoff;
        leadm = __ballot(el == 0 && lane < 63 && pent != 0u);
        asm volatile("" : "+v"(paddr));
    };
    if (poller) { pent = (unsigned)st_lds[pc]; pnext = (unsigned)st_lds[pc + 1]; stream_state(); }
    int snap = -1;
    auto poll_issue = [&]() {
        snap = pc;
        glds4_sys(paddr, myws, lds0 + PB_LDS);
    };
    auto poll_eval = [&](unsigned fv) {
        const unsigned long long ok = __ballot(fv >= seq && snap == pc);
        const bool rdy = ((unsigned)(ok >> (9 * strm)) & 0x1FFu) == 0x1FFu;
        const unsigned long long lead = __ballot(rdy) & leadm;
        bool adv = false;
        if (lead != 0) {
            const int pos = npub + (int)__builtin_amdgcn_mbcnt_hi((unsigned)(lead >> 32), __builtin_amdgcn_mbcnt_lo((unsigned)lead, 0u));
            if ((lead >> lane) & 1ull) qt_lds[pos] = (int)(pent & 63u);
            npub = __builtin_amdgcn_readfirstlane(npub + __popcll(lead));
            if (lane == 0) cnt_lds[0] = npub;
            adv = (lead >> (9 * strm)) & 1ull;
        }
        return adv;
    };
    auto stream_advance = [&](bool adv) {
        pc += adv ? 1 : 0;
        pent = adv ? pnext : pent;
    };
    auto poll_finish = [&]() {
        wait_vm<0>();
        stream_advance(poll_eval((unsigned)pb_lds[lane]));
        stream_state();
        pnext = (unsigned)st_lds[pc + 1];
    };
    auto poll_now = [&]() { poll_issue(); poll_finish(); };
    auto voff_from = [&](unsigned q) { return q * STEP_BYTES + lane16; };
    auto issue = [&](int t, unsigned voff) {
        const unsigned so = (unsigned)(t & (RING - 1)) * STAGE_BYTES;
#pragma unroll
        for (int i = 0; i < 5; ++i) glds16_sys(voff, sbase[i], ldst[i] + so);
    };
#define DSTAMP(i)
    i32x4 acc[9][4];
#pragma unroll
    for (int m = 0; m < 9; ++m)
#pragma unroll
        for (int n = 0; n < 4; ++n) acc[m][n] = (i32x4){0, 0, 0, 0};
    i32x4 Af[9], Bf[4];
    int iss = 0;

    auto loader_try = [&](int lim) {
        int r = __builtin_amdgcn_readfirstlane(cnt_lds[0]);
        if (r > lim) r = lim;
        const bool any = iss < r;
        for (; iss < r; ++iss) issue(iss, voff_from((unsigned)qt_lds[iss]));
        return any;
    };
    auto loader_block = [&](int u, int lim) {
        if (lim > 64) lim = 64;
        while (true) {
            loader_try(lim);
            if (iss > u) break;
            __builtin_amdgcn_s_sleep(1);
        }
        const unsigned long long t0 = __builtin_amdgcn_s_memrealtime();
        while (iss < lim && __builtin_amdgcn_s_memrealtime() - t0 < LINGER)
            if (!loader_try(lim)) __builtin_amdgcn_s_sleep(1);
    };
    auto poller_block = [&](int u, int lim, bool inflight) {
        gu32* tmo = (gu32*)(myws + TMO_OFF);
        unsigned long long t0 = __builtin_amdgcn_s_memrealtime();
        if (inflight) poll_finish();
        for (unsigned spins = 1; npub <= u; ++spins) {
            poll_now();
            if (npub > u) break;
            __builtin_amdgcn_s_sleep(PSLEEP);
            if ((spins & 15u) == 0u) {
                const unsigned tv = __hip_atomic_load(tmo, __ATOMIC_RELAXED, __HIP_MEMORY_SCOPE_AGENT);
                if (__builtin_amdgcn_readfirstlane(tv) == seq || __builtin_amdgcn_s_memrealtime() - t0 > DEADLINE_TICKS) {
                    if (lane == 0) __hip_atomic_store(tmo, seq, __ATOMIC_RELAXED, __HIP_MEMORY_SCOPE_AGENT);
                    npub = 64;
                    if (lane == 0) cnt_lds[0] = 64;
                }
            }
        }
        t0 = __builtin_amdgcn_s_memrealtime();
        while (npub < lim && npub < 64 && __builtin_amdgcn_s_memrealtime() - t0 < LINGER) poll_now();
    };
    auto wait_landed = [&](int u) {
        const int ahead = iss - u - 1;
        wait_vm<10>();
        if (UNLIKELY(ahead < 2)) { wait_vm<5>(); if (ahead < 1) wait_vm<0>(); }
    };
    auto read_frags = [&](int t, auto dma, unsigned voff) {
        const char* st = smem + (t & (RING - 1)) * STAGE_BYTES + lane16;
        const char* sa = st + wr * 9 * 1024;
        const char* sb = st + MT * 1024 + wc * 4096;
        const unsigned so = (unsigned)(iss & (RING - 1)) * STAGE_BYTES;
        constexpr bool D = decltype(dma)::value;
        if constexpr (D) { glds16_sys(voff, sbase[0], ldst[0] + so); SB(); }
#pragma unroll
        for (int n = 0; n < 4; ++n) Bf[n] = *(const i32x4*)(sb + n * 1024);
        if constexpr (D) { SB(); glds16_sys(voff, sbase[1], ldst[1] + so); SB(); }
#pragma unroll
        for (int m = 0; m < 3; ++m) Af[m] = *(const i32x4*)(sa + m * 1024);
        if constexpr (D) { SB(); glds16_sys(voff, sbase[2], ldst[2] + so); SB(); }
#pragma unroll
        for (int m = 3; m < 6; ++m) Af[m] = *(const i32x4*)(sa + m * 1024);
        if constexpr (D) { SB(); glds16_sys(voff, sbase[3], ldst[3] + so); SB(); }
#pragma unroll
        for (int m = 6; m < 9; ++m) Af[m] = *(const i32x4*)(sa + m * 1024);
        if constexpr (D) { SB(); glds16_sys(voff, sbase[4], ldst[4] + so); }
        SB();
    };
    auto mma_rows = [&](int m0, int m1) {
#pragma unroll
        for (int m = m0; m < m1; ++m)
#pragma unroll
            for (int n = 0; n < 4; ++n) acc[m][n] = __builtin_amdgcn_mfma_i32_16x16x64_i8(Af[m], Bf[n], acc[m][n], 0, 0, 0);
    };
    auto mma_phase = [&](auto&& h1, auto&& h2, auto&& h3) {
        __builtin_amdgcn_s_setprio(1);
        mma_rows(0, 3);
        SB();
        h1();
        SB();
        mma_rows(3, 4);
        SB();
        h2();
        SB();
        mma_rows(4, 5);
        SB();
        h3();
        SB();
        mma_rows(5, 9);
        __builtin_amdgcn_s_setprio(0);
    };
    typedef BC<true> True_;
    typedef BC<false> False_;
    WG_BARRIER();
    if (poller) {
        poller_block(0, RING, false);
        WG_BARRIER();
        bool inflight = false;
#pragma nounroll
        for (int t = 0; t < 64; ++t) {
            DSTAMP(t);
            npub = __builtin_amdgcn_readfirstlane(npub);
            unsigned fv = 0;
            if (LIKELY(inflight)) { wait_vm<0>(); fv = (unsigned)pb_lds[lane]; }
            read_frags(t, False_{}, 0u);
            bool adv = false;
            __builtin_amdgcn_s_setprio(2);
            if (LIKELY(inflight)) adv = poll_eval(fv);
            __builtin_amdgcn_s_setprio(0);
            WG_BARRIER();
            SB();
            mma_phase([&]() { stream_advance(adv); stream_state(); },
                      [&]() { pnext = (unsigned)st_lds[pc + 1]; },
                      [&]() {
                          inflight = npub <= t + PAHEAD && npub < 64;
                          if (inflight) poll_issue();
                      });
            SB();
            if (UNLIKELY(npub <= t + 1 && t < 63)) { poller_block(t + 1, t + RING, inflight); inflight = false; }
            WG_BARRIER();
        }
        DSTAMP(64);
    } else if (!g1) {
        loader_block(0, RING);
        wait_landed(0);
        int lim = smin3(RING, __builtin_amdgcn_readfirstlane(cnt_lds[0]), 64);
        unsigned voff = voff_from((unsigned)qt_lds[iss]);
        WG_BARRIER();
        for (int t = 0; t < 64; ++t) {
            if (LIKELY(iss < lim)) { read_frags(t, True_{}, voff); ++iss; } else read_frags(t, False_{}, voff);
            WG_BARRIER();
            const int cv = cnt_lds[0];
            const unsigned qe = (unsigned)qt_lds[iss];
            SB();
            int cs;
            mma_phase([&]() { cs = __builtin_amdgcn_readfirstlane(cv); },
                      [&]() { lim = smin3(t + 1 + RING, cs, 64); },
                      [&]() { voff = voff_from(qe); asm volatile("" : "+v"(voff)); });
            SB();
            if (LIKELY(t < 63)) {
                if (UNLIKELY(iss <= t + 1)) {
                    loader_block(t + 1, t + RING);
                    lim = smin3(t + 1 + RING, __builtin_amdgcn_readfirstlane(cnt_lds[0]), 64);
                    voff = voff_from((unsigned)qt_lds[iss]);
                }
                wait_landed(t + 1);
            }
            WG_BARRIER();
        }
    } else {
        loader_block(0, RING);
        wait_landed(0);
        WG_BARRIER();
        int lim = smin3(RING, __builtin_amdgcn_readfirstlane(cnt_lds[0]), 64);
        unsigned voff = voff_from((unsigned)qt_lds[iss]);
        WG_BARRIER();
        for (int t = 0; t < 63; ++t) {
            if (LIKELY(iss < lim)) { read_frags(t, True_{}, voff); ++iss; } else read_frags(t, False_{}, voff);
            if (UNLIKELY(iss <= t + 1)) loader_block(t + 1, t + RING);
            wait_landed(t + 1);
            WG_BARRIER();
            const int cv = cnt_lds[0];
            const unsigned qe = (unsigned)qt_lds[iss];
            SB();
            int cs;
            mma_phase([&]() { cs = __builtin_amdgcn_readfirstlane(cv); },
                      [&]() { lim = smin3(t + 1 + RING, cs, 64); },
                      [&]() { voff = voff_from(qe); asm volatile("" : "+v"(voff)); });
            SB();
            WG_BARRIER();
        }
        read_frags(63, False_{}, 0u);
        WG_BARRIER();
        SB();
        mma_phase([]() {}, []() {}, []() {});
        SB();
    }

    const unsigned tv = __hip_atomic_load((gu32*)(myws + TMO_OFF), __ATOMIC_RELAXED, __HIP_MEMORY_SCOPE_AGENT);
    const bool bad = __builtin_amdgcn_readfirstlane(tv) == seq;
    const float scale = a.sx[0] * a.sw[0];
    float* outp = a.out + (size_t)(I * (MT * 16) + wr * 144 + (lane >> 4) * 4) * NN + J * 256 + wc * 64 + (lane & 15) * 4;
    auto put = [&](int m, int r, f32x4 o) {
        asm volatile("global_store_dwordx4 %0, %1, off sc1\n\ts_nop 1" :: "v"(outp + (size_t)(m * 16 + r) * NN), "v"(o) : "memory");
    };
    if (UNLIKELY(bad)) {
        const float qn = __builtin_nanf("");
        for (int m = 0; m < 9; ++m)
            if (MT * I + wr * 9 + m < 256)
                for (int r = 0; r < 4; ++r) put(m, r, (f32x4){qn, qn, qn, qn});
        return;
    }
    const float tscale = scale * -1.4426950408889634f, tmax = 60.0f * 1.4426950408889634f;
#pragma unroll
    for (int m = 0; m < 9; ++m) {
        if (MT * I + wr * 9 + m < 256) {
            f32x2 y[4][2], d[4][2];
#pragma unroll
            for (int r = 0; r < 4; ++r)
#pragma unroll
                for (int h = 0; h < 2; ++h) {
                    const f32x2 v = {(float)acc[m][2 * h][r], (float)acc[m][2 * h + 1][r]};
                    y[r][h] = v * scale;
                    d[r][h] = v * tscale;
                }
#pragma unroll
            for (int r = 0; r < 4; ++r)
#pragma unroll
                for (int h = 0; h < 2; ++h) {
                    d[r][h].x = __builtin_amdgcn_exp2f(__builtin_amdgcn_fmed3f(d[r][h].x, -tmax, tmax));
                    d[r][h].y = __builtin_amdgcn_exp2f(__builtin_amdgcn_fmed3f(d[r][h].y, -tmax, tmax));
                }
#pragma unroll
            for (int r = 0; r < 4; ++r)
#pragma unroll
                for (int h = 0; h < 2; ++h) {
                    d[r][h] = d[r][h] + 1.0f;
                    d[r][h].x = __builtin_amdgcn_rcpf(d[r][h].x);
                    d[r][h].y = __builtin_amdgcn_rcpf(d[r][h].y);
                }
#pragma unroll
            for (int r = 0; r < 4; ++r) {
                const f32x2 o0 = y[r][0] * d[r][0], o1 = y[r][1] * d[r][1];
                put(m, r, (f32x4){o0.x, o0.y, o1.x, o1.y});
            }
        }
    }
}

__global__ void __launch_bounds__(THREADS, 2) fused_gemm_ag(Args a) {
    extern __shared__ __attribute__((aligned(16))) char smem[];
    const int b = blockIdx.x;
    const int wave = __builtin_amdgcn_readfirstlane(threadIdx.x >> 6);
    const int lane = threadIdx.x & 63;
    if (b < NCOMM && wave != 0 && wave != 7) return;
    if (LIKELY(b < NCOMM)) {
        volatile lds_int* done = (volatile lds_int*)((lds_char*)smem);
        if (UNLIKELY(wave == 7)) {
            if (lane == 0) done[0] = 0;
            WG_BARRIER();
            flag_wave(a, b, lane, next_seq(a, b, wave, lane), done);
        } else {
            comm_wave(a, b, lane, done);
        }
    } else {
        gemm_wg(a, smem, b - NCOMM, wave, lane, b);
    }
}

}

extern "C" void kernel_launch(int rank, int nranks, ncclComm_t comm, hipStream_t stream,
                              void* const* d_in, const int* in_sizes, int n_in,
                              void* d_out, int out_size,
                              const int* in_dtypes, int out_dtype,
                              void* const* d_out_all,
                              void* const* d_ws_all, size_t ws_bytes) {
    (void)comm; (void)in_sizes; (void)n_in; (void)in_dtypes; (void)out_dtype; (void)d_out_all;
    if (nranks != NR || (unsigned)rank >= (unsigned)NR || ws_bytes < WS_NEEDED || out_size != MM * NN) {
        fprintf(stderr, "kernel_launch: unexpected configuration\n");
        return;
    }
    HIP_CHECK(hipFuncSetAttribute((const void*)fused_gemm_ag, hipFuncAttributeMaxDynamicSharedMemorySize, LDS_BYTES));
    Args a;
    memset(&a, 0, sizeof a);
    a.x = (const char*)d_in[0];
    a.w = (const char*)d_in[1];
    a.sx = (const float*)d_in[2];
    a.sw = (const float*)d_in[3];
    a.out = (float*)d_out;
    for (int j = 0; j < NR; ++j) a.ws[j] = (char*)d_ws_all[(rank + 1 + j) % NR];
    a.rank = rank;
    for (int g = 0; g < NGEMM; ++g) {
        int I, J;
        tile_of(g, I, J);
        const int u0 = (MT * I) >> 2;
        int u1 = MT * I + MT - 1; if (u1 > 255) u1 = 255; u1 >>= 2;
        int key[NR];
        for (int r = 0; r < NR; ++r) {
            int mx = 0;
            for (int u = u0; u <= u1; ++u) { int p = pos_of(u, r); if (p > mx) mx = p; }
            for (int u = 64 + 4 * J; u < 64 + 4 * J + 4; ++u) { int p = pos_of(u, r); if (p > mx) mx = p; }
            key[r] = mx * 8 + r;
        }
        unsigned ord = (unsigned)rank;
        int used = 1 << rank;
        for (int k = 1; k < 8; ++k) {
            int best = -1;
            for (int r = 0; r < NR; ++r) if (!((used >> r) & 1) && (best < 0 || key[r] < key[best])) best = r;
            used |= 1 << best;
            ord |= (unsigned)best << (4 * k);
        }
        a.order[g] = ord;
    }
    hipLaunchKernelGGL(fused_gemm_ag, dim3(GRID), dim3(THREADS), LDS_BYTES, stream, a);
    HIP_CHECK(hipGetLastError());
}
```
